```python
import numpy as np

kernel_name = "gpu_gemm_rs_m4096_k4096_n4096_f32_reluqe4m3"


ABSMAX_RTOL = 0.09

COMPUTE_DTYPE = "float32"
EPILOGUE = "relu_quant_e4m3"
IN_QUANT = None
OUT_QUANT = "e4m3"
IN_DTYPES = {"x": "float32", "w_mat": "float32"}
WORLD_SIZE = 8

def setup_inputs():

    x = np.random.randn(4096, 4096).astype(np.float32)
    w_mat = (np.random.randn(4096, 4096) / np.sqrt(4096)).astype(np.float32)
    return {"x": x, "w_mat": w_mat}

def _snap_fp8(x, mant_bits, max_abs, min_normal):
    a = np.abs(x.astype(np.float64))
    _m, _e = np.frexp(a)
    q = np.where(a >= min_normal, np.ldexp(1.0, _e - 1 - mant_bits),
                 min_normal * 2.0 ** -mant_bits)
    snapped = np.minimum(np.round(a / q) * q, max_abs)
    return (np.sign(x) * snapped).astype(np.float32)
def _quant_dequant(y, amax):
    scale = amax / 448.0
    q = _snap_fp8(y / scale, 3, 448.0, 2.0 ** -6)
    return (q * scale).astype(np.float32)
def _epilogue(y):
    y = np.maximum(y, 0.0)
    return _quant_dequant(y, np.abs(y).max())

def reference(x, w_mat):
    y = (x @ w_mat).astype(np.float32)
    return _epilogue(y)
```

```cpp
#include <hip/hip_runtime.h>
#include <rccl/rccl.h>
#include <cstdio>

#define HIP_CHECK(expr) do { hipError_t _e = (expr); if (_e != hipSuccess) { \
    fprintf(stderr, "HIP error %s at %s:%d\n", hipGetErrorString(_e), __FILE__, __LINE__); \
    return; } } while (0)

typedef unsigned u32;
typedef unsigned long long u64;
typedef u32 u32x2 __attribute__((ext_vector_type(2)));
typedef u32 u32x3 __attribute__((ext_vector_type(3)));
typedef u32 u32x4 __attribute__((ext_vector_type(4)));
typedef u32 u32x8 __attribute__((ext_vector_type(8)));
typedef u32 u32x16 __attribute__((ext_vector_type(16)));
typedef float f32x2 __attribute__((ext_vector_type(2)));
typedef float f32x4 __attribute__((ext_vector_type(4)));
typedef __bf16 bf16x2 __attribute__((ext_vector_type(2)));
typedef __bf16 bf16x8 __attribute__((ext_vector_type(8)));
typedef __attribute__((address_space(1))) u32 gu32;
typedef __attribute__((address_space(1))) u64 gu64;
typedef __amdgpu_buffer_rsrc_t rsrc_t;

#define A_ROWS 10
#define TEST_BASE_SHIFT 0

constexpr int NR = 8;
constexpr int NP = 24;
constexpr int NCOMM = 2 * NP;
constexpr int GRID = 512;
constexpr int NGEMM = GRID - NCOMM;
constexpr int KEEP = 3;
constexpr int N_I = 64 * A_ROWS;
constexpr int N_C = 64 * (16 - A_ROWS);
constexpr int HELD_I = N_I - NP * KEEP, HELD_C = N_C - NP * KEEP;
constexpr int NHOLD = (HELD_I + HELD_C) / 4;
constexpr u64 DEADLINE = 30000000ull;

constexpr u32 BLK = 3072;
constexpr u32 N_IN = 8 * 48 * 8, N_OUT = 16 * 4 * 64;
constexpr u32 O_SEQ = 0;
constexpr u32 O_TMO = O_SEQ + GRID * 4 * 4;
constexpr u32 O_AMAX = O_TMO + 1024;
constexpr u32 O_PROG = O_AMAX + 4096;
constexpr u32 O_INHDR = O_PROG + NCOMM * 128 + 1024;
constexpr u32 O_OUTHDR = O_INHDR + N_IN * 16;
constexpr u32 O_INDATA = ((O_OUTHDR + 512 * 128 + 4095) / 4096) * 4096;
constexpr u32 O_OUTDATA = O_INDATA + N_IN * BLK;
constexpr u32 O_Y = O_OUTDATA + N_OUT * BLK;
constexpr u32 O_INOVF = O_Y + 1024 * 8192;
constexpr u32 O_OUTOVF = O_INOVF + N_IN * 1024;
constexpr u32 O_DBG = O_OUTOVF + N_OUT * 1024;
constexpr u32 WS_USED = O_DBG + GRID * 4 * 64 * 8;
static_assert(O_AMAX % 128 == 0 && O_PROG % 128 == 0 && O_INHDR % 128 == 0 && O_OUTHDR % 128 == 0, "align");
static_assert(NR * NCOMM * 8 <= 4096 && 384 % NP == 0 && NCOMM % 8 == 0 && (HELD_I + HELD_C) % 4 == 0, "shape");

constexpr int SYS = 17;

struct Params {
    const float* x;
    const float* w;
    unsigned char* out;
    char* my;
    int rank;
    int pad;
    char* rel[NR];
};

#define STAMP(id) do { } while (0)
#define STAMPC(id) do { } while (0)
#define STAMP_INIT()
#define STAMP_DUMP() do { } while (0)
#define POLL_SLEEP 4
#define GAUX SYS

#define DEV __device__ __forceinline__
#define CFENCE() asm volatile("" ::: "memory")
#define WAIT_VM0() do { __builtin_amdgcn_s_waitcnt(0x0F70); asm volatile("s_waitcnt vmcnt(0)" ::: "memory"); } while (0)

DEV rsrc_t mk_rsrc(const void* base, u32 bytes) { return __builtin_amdgcn_make_buffer_rsrc((void*)base, 0, bytes, 0x00020000); }
DEV u32 rfl(u32 v) { return __builtin_amdgcn_readfirstlane(v); }
DEV u32 cvt2(float a, float b) { f32x2 v = {a, b}; return __builtin_bit_cast(u32, __builtin_convertvector(v, bf16x2)); }
DEV u32 umax(u32 a, u32 b) { return a > b ? a : b; }
DEV u32 wave_umax(u32 v) {
    v = umax(v, (u32)__builtin_amdgcn_update_dpp(0, (int)v, 0x111, 0xf, 0xf, true));
    v = umax(v, (u32)__builtin_amdgcn_update_dpp(0, (int)v, 0x112, 0xf, 0xf, true));
    v = umax(v, (u32)__builtin_amdgcn_update_dpp(0, (int)v, 0x114, 0xf, 0xf, true));
    v = umax(v, (u32)__builtin_amdgcn_update_dpp(0, (int)v, 0x118, 0xf, 0xf, true));
    v = umax(v, (u32)__builtin_amdgcn_update_dpp(0, (int)v, 0x142, 0xa, 0xf, false));
    v = umax(v, (u32)__builtin_amdgcn_update_dpp(0, (int)v, 0x143, 0xc, 0xf, false));
    return (u32)__builtin_amdgcn_readlane((int)v, 63);
}

struct Wait {
    u32 seq;
    u64 t0;
    gu32* tmo;
};
DEV bool expired(const Wait& wt) {
    const u64 dt = __builtin_amdgcn_s_memrealtime() - wt.t0;
    if (dt < 100000ull) return false;
    if (dt > DEADLINE || rfl(__hip_atomic_load(wt.tmo, __ATOMIC_RELAXED, __HIP_MEMORY_SCOPE_AGENT)) == wt.seq) {
        __hip_atomic_store(wt.tmo, wt.seq, __ATOMIC_RELAXED, __HIP_MEMORY_SCOPE_AGENT);
        return true;
    }
    return false;
}
DEV u64 ld_gran(const char* p) { return __hip_atomic_load((gu64*)p, __ATOMIC_RELAXED, __HIP_MEMORY_SCOPE_SYSTEM); }
DEV void st_gran(char* p, u32 payload, u32 seq) {
    __hip_atomic_store((gu64*)p, ((u64)seq << 32) | payload, __ATOMIC_RELAXED, __HIP_MEMORY_SCOPE_SYSTEM);
}
DEV bool poll_gran(const char* p, bool act, u64& v, const Wait& wt) {
    bool ok = true;
    for (;;) {
        const bool done = !act || (u32)(v >> 32) >= wt.seq;
        if (__all(done)) break;
        if (expired(wt)) { ok = false; break; }
        __builtin_amdgcn_s_sleep(POLL_SLEEP);
        if (!done) v = ld_gran(p);
    }
    CFENCE();
    return ok;
}

struct Packed {
    u32 L[8], N[4], X[4];
    u32 lo, hi;
};
DEV void pack(const u32 (&D)[16], float amax, Packed& o, int kind) {
    const u32 top = wave_umax(__builtin_bit_cast(u32, amax)) >> 24;
    int b = (int)(top & 0x7f) - 7 - ((TEST_BASE_SHIFT & kind) ? 1 : 0);
    b = b < 0 ? 0 : b;
    const u32 base4 = (u32)b * 0x01010101u;
    u32 H[8], d[8], chk = 0;
#pragma unroll
    for (int q = 0; q < 8; ++q) {
        o.L[q] = __builtin_amdgcn_perm(D[2 * q + 1], D[2 * q], 0x06040200u);
        H[q] = __builtin_amdgcn_perm(D[2 * q + 1], D[2 * q], 0x07050301u);
        d[q] = H[q] - base4;
        chk |= d[q];
    }
    u64 pay = (u64)b;
    u64 m = __ballot((chk & 0x78787878u) != 0);
    if (__builtin_expect(m != 0, 0)) {
        u32x8 dl;
        u32 esc = 0;
#pragma unroll
        for (int q = 0; q < 8; ++q) {
            d[q] = (((H[q] | 0x80808080u) - base4) & 0x07070707u) | (H[q] & 0x80808080u);
            dl[q] = (d[q] + base4) ^ H[q];
            esc |= ((dl[q] + 0x7f7f7f7fu) & 0x80808080u) >> (7 - q);
        }
        int cnt = 0;
        while (m != 0 && cnt <= 3) {
            const int l = __builtin_ctzll(m);
            m &= m - 1;
            u32 bits = (u32)__builtin_amdgcn_readlane((int)esc, l);
            while (bits != 0 && cnt <= 3) {
                const int pos = __builtin_ctz(bits);
                bits &= bits - 1;
                const u32 e = ((u32)__builtin_amdgcn_readlane((int)dl[pos & 7], l) >> (pos & 24)) & 0x7f;
                pay |= (u64)((u32)l | ((u32)(4 * (pos & 7) + (pos >> 3)) << 6) | (e << 11)) << (8 + 18 * cnt);
                ++cnt;
            }
        }
        if (cnt > 3) {
            pay = 0x80;
#pragma unroll
            for (int t = 0; t < 4; ++t) { d[2 * t] = H[2 * t]; d[2 * t + 1] = 0; }
        }
    }
#pragma unroll
    for (int t = 0; t < 4; ++t) {
        o.N[t] = d[2 * t] | __builtin_amdgcn_alignbit(d[2 * t + 1], d[2 * t + 1], 28);
        o.X[t] = H[2 * t + 1];
    }
    o.lo = (u32)pay;
    o.hi = (u32)(pay >> 32);
}
DEV void dec_h(u32 n, u32 base4, u32& ha, u32& hb) {
    ha = (n & 0x87878787u) + base4;
    hb = (__builtin_amdgcn_alignbit(n, n, 4) & 0x87878787u) + base4;
}
DEV u32x4 mk_frag(u32 la, u32 lb, u32 ha, u32 hb) {
    u32x4 r;
    r.x = __builtin_amdgcn_perm(ha, la, 0x05010400u);
    r.y = __builtin_amdgcn_perm(ha, la, 0x07030602u);
    r.z = __builtin_amdgcn_perm(hb, lb, 0x05010400u);
    r.w = __builtin_amdgcn_perm(hb, lb, 0x07030602u);
    return r;
}
struct Pk { u32x4 l0, l1, n; };
DEV Pk ld_in(rsrc_t rs, int lane, u32 slot) {
    Pk k;
    const u32 so = O_INDATA + slot * BLK;
    k.l0 = __builtin_amdgcn_raw_buffer_load_b128(rs, lane * 16, so, GAUX);
    k.l1 = __builtin_amdgcn_raw_buffer_load_b128(rs, lane * 16 + 1024, so, GAUX);
    k.n = __builtin_amdgcn_raw_buffer_load_b128(rs, lane * 16 + 2048, so, GAUX);
    return k;
}
DEV u32x16 cat4(u32x4 a, u32x4 b, u32x4 c, u32x4 d) {
    const u32x8 l = __builtin_shufflevector(a, b, 0, 1, 2, 3, 4, 5, 6, 7), h = __builtin_shufflevector(c, d, 0, 1, 2, 3, 4, 5, 6, 7);
    return __builtin_shufflevector(l, h, 0, 1, 2, 3, 4, 5, 6, 7, 8, 9, 10, 11, 12, 13, 14, 15);
}
#define FRAG(v, t) __builtin_shufflevector(v, v, 4 * (t), 4 * (t) + 1, 4 * (t) + 2, 4 * (t) + 3)
DEV u32x16 unpack_in(const Pk& k, u32 lo) {
    const u32 base4 = (lo & 0x7f) * 0x01010101u;
    u32 ha, hb;
    dec_h(k.n.x, base4, ha, hb); const u32x4 f0 = mk_frag(k.l0.x, k.l0.y, ha, hb);
    dec_h(k.n.y, base4, ha, hb); const u32x4 f1 = mk_frag(k.l0.z, k.l0.w, ha, hb);
    dec_h(k.n.z, base4, ha, hb); const u32x4 f2 = mk_frag(k.l1.x, k.l1.y, ha, hb);
    dec_h(k.n.w, base4, ha, hb); const u32x4 f3 = mk_frag(k.l1.z, k.l1.w, ha, hb);
    return cat4(f0, f1, f2, f3);
}
DEV u32x16 unpack_in_raw(const char* my, int lane, u32 slot) {
    const char* pd = my + O_INDATA + slot * BLK + lane * 16;
    const char* px = my + O_INOVF + slot * 1024 + lane * 16;
    u32x4 l0, l1, n, x;
    asm volatile("global_load_dwordx4 %0, %4, off sc0 sc1\n\tglobal_load_dwordx4 %1, %4, off offset:1024 sc0 sc1\n\t"
                 "global_load_dwordx4 %2, %4, off offset:2048 sc0 sc1\n\tglobal_load_dwordx4 %3, %5, off sc0 sc1\n\ts_waitcnt vmcnt(0)"
                 : "=&v"(l0), "=&v"(l1), "=&v"(n), "=&v"(x) : "v"(pd), "v"(px) : "memory");
    return cat4(mk_frag(l0.x, l0.y, n.x, x.x), mk_frag(l0.z, l0.w, n.y, x.y), mk_frag(l1.x, l1.y, n.z, x.z), mk_frag(l1.z, l1.w, n.w, x.w));
}
DEV void st128(u32x4 v, rsrc_t rs, u32 off) { __builtin_amdgcn_raw_buffer_store_b128(v, rs, off, 0, SYS); }
DEV void st96(u32x3 v, rsrc_t rs, u32 off) { __builtin_amdgcn_raw_buffer_store_b96(v, rs, off, 0, SYS); }
DEV void st_gran_b(rsrc_t rs, bool act, u32 off, u32 payload, u32 seq) {
    const u32x2 v = {payload, seq};
    __builtin_amdgcn_raw_buffer_store_b64(v, rs, act ? off : 0xfffffff0u, 0, SYS);
}

DEV int chunk_at(int pos) {
    if (pos < 2 * A_ROWS) return (pos & 1) ? 16 + (pos >> 1) : (pos >> 1);
    if (pos < A_ROWS + 16) return 16 + pos - A_ROWS;
    if (pos < 32) return pos - 16;
    return pos;
}
DEV void ld_src(const float* px, const float* pw, int lane, bool valid, int tg, int ks, int chunk, int blk, f32x4 (&F)[8]) {
    const int fr = lane & 15, fq = lane >> 4;
    const bool isx = chunk < 16;
    const rsrc_t rs = mk_rsrc(isx ? px : pw, valid ? 4096u * 512u * 4u : 0u);
    u32 so, s2, s1, s0, vo;
    if (isx) {
        const int owner = (chunk < A_ROWS ? tg : 1 - tg) * 4 + (blk >> 1);
        so = ((owner * 512 + chunk * 32) * 512 + ks * 128 + (blk & 1) * 64) * 4;
        s2 = 16 * 2048; s1 = 128; s0 = 64;
        vo = fr * 2048 + fq * 16;
    } else {
        so = ((ks * 128 + (blk >> 1) * 32) * 4096 + (chunk - 16) * 128 + (blk & 1) * 64) * 4;
        s2 = 16 * 16384; s1 = 2 * 16384; s0 = 16384;
        vo = fq * 4 * 16384 + fr * 16;
    }
#pragma unroll
    for (int u = 0; u < 8; ++u)
        F[u] = __builtin_bit_cast(f32x4, __builtin_amdgcn_raw_buffer_load_b128(
                   rs, vo, so + (u >> 2) * s2 + ((u >> 1) & 1) * s1 + (u & 1) * s0, 0));
}
DEV float cvt_src(const f32x4 (&F)[8], bool isx, u32 (&D)[16]) {
    if (isx) {
#pragma unroll
        for (int u = 0; u < 8; ++u) { D[2 * u] = cvt2(F[u].x, F[u].y); D[2 * u + 1] = cvt2(F[u].z, F[u].w); }
    } else {
#pragma unroll
        for (int f = 0; f < 4; ++f)
#pragma unroll
            for (int e = 0; e < 4; ++e) D[4 * f + e] = cvt2(F[2 * e][f], F[2 * e + 1][f]);
    }
    float m = 0.f;
#pragma unroll
    for (int u = 0; u < 8; ++u) {
        m = fmaxf(fmaxf(m, fabsf(F[u].x)), fabsf(F[u].y));
        m = fmaxf(fmaxf(m, fabsf(F[u].z)), fabsf(F[u].w));
    }
    return m;
}
DEV void st_in(rsrc_t rs, int lane, u32 slot, const Packed& k) {
    if (__builtin_expect(k.lo & 0x80, 0)) st128((u32x4){k.X[0], k.X[1], k.X[2], k.X[3]}, rs, O_INOVF + slot * 1024 + lane * 16);
    const u32 o = O_INDATA + slot * BLK + lane * 16;
    st128((u32x4){k.L[0], k.L[1], k.L[2], k.L[3]}, rs, o);
    st128((u32x4){k.L[4], k.L[5], k.L[6], k.L[7]}, rs, o + 1024);
    st128((u32x4){k.N[0], k.N[1], k.N[2], k.N[3]}, rs, o + 2048);
}

DEV void item_at(bool intra, int i, int& c, int& cg) {
    const int t = i >> 1;
    int n;
    if (intra) {
        if (t < A_ROWS * A_ROWS) {
            int s = 0;
            while ((s + 1) * (s + 1) <= t) ++s;
            const int r = t - s * s;
            if (r < s) { c = s; n = r; } else { c = r - s; n = s; }
        } else {
            n = t / A_ROWS;
            c = t - n * A_ROWS;
        }
    } else {
        n = t / (16 - A_ROWS);
        c = A_ROWS + t - n * (16 - A_ROWS);
    }
    cg = 2 * n + (i & 1);
}
DEV u32 q4(float a, float b, float c, float d, float inv) {
    int r = __builtin_amdgcn_cvt_pk_fp8_f32(a * inv, b * inv, 0, false);
    r = __builtin_amdgcn_cvt_pk_fp8_f32(c * inv, d * inv, r, true);
    return (u32)r;
}

#define XSLOT(s, kh) (xs0 + (s) * 384 + (kh))
#define WSLOT(s) (ws0 + (s) * 384)
DEV void fix_w(u32x4* wr, u32 lo, u32 hi, u32 slot, const char* my, int lane) {
    if (__builtin_expect(((lo >> 7) | hi) != 0, 0)) {
        if (lo & 0x80) {
            const u32x16 r = unpack_in_raw(my, lane, slot);
            wr[lane] = FRAG(r, 0); wr[64 + lane] = FRAG(r, 1); wr[128 + lane] = FRAG(r, 2); wr[192 + lane] = FRAG(r, 3);
        } else if (lane < 3) {
            const u32 e = (u32)((((u64)hi << 32) | lo) >> (8 + 18 * lane)) & 0x3ffff;
            const u32 idx = (e >> 6) & 31;
            atomicXor((u32*)wr + ((idx >> 3) * 64 + (e & 63)) * 4 + ((idx & 7) >> 1), (e >> 11) << (8 + 16 * (idx & 1)));
        }
    }
}
DEV void fix_x(u32x16& xn, u32 lo, u32 hi, u32 slot, const char* my, int lane) {
    asm volatile("" : "+v"(xn));
    if (__builtin_expect(((lo >> 7) | hi) != 0, 0)) {
        if (lo & 0x80) {
            xn = unpack_in_raw(my, lane, slot);
        } else {
            u64 pay = (((u64)hi << 32) | lo) >> 8;
            while (pay != 0) {
                const u32 e = (u32)pay & 0x3ffff, idx = (e >> 6) & 31;
                pay >>= 18;
                xn[idx >> 1] ^= (lane == (int)(e & 63)) ? (e >> 11) << (8 + 16 * (idx & 1)) : 0;
            }
        }
    }
}
DEV void mm16(f32x4 (&acc)[2][4], const u32x4* rd, const u32x16& xc, int lane) {
#pragma unroll
    for (int i = 0; i < 8; ++i) {
        const bf16x8 av = __builtin_bit_cast(bf16x8, rd[i * 64 + lane]);
        const u32x4 x0 = (i >> 2) ? FRAG(xc, 1) : FRAG(xc, 0), x1 = (i >> 2) ? FRAG(xc, 3) : FRAG(xc, 2);
        acc[0][i & 3] = __builtin_amdgcn_mfma_f32_16x16x32_bf16(av, __builtin_bit_cast(bf16x8, x0), acc[0][i & 3], 0, 0, 0);
        acc[1][i & 3] = __builtin_amdgcn_mfma_f32_16x16x32_bf16(av, __builtin_bit_cast(bf16x8, x1), acc[1][i & 3], 0, 0, 0);
    }
}
#define UPFRONT 2
template <int NV, int NL, int NW>
DEV void mm_order() {
#pragma unroll
    for (int i = 0; i < 4; ++i) {
        __builtin_amdgcn_sched_group_barrier(0x100, 1, 0);
        if (NV) __builtin_amdgcn_sched_group_barrier(0x002, 3, 0);
    }
#pragma unroll
    for (int i = 0; i < 16; ++i) {
        __builtin_amdgcn_sched_group_barrier(0x008, 1, 0);
        if (i < 8 && (i & 1) == 0) __builtin_amdgcn_sched_group_barrier(0x100, 1, 0);
        if (NV) __builtin_amdgcn_sched_group_barrier(0x002, NV, 0);
        if (NW && (i == 3 || i == 8)) __builtin_amdgcn_sched_group_barrier(0x200, 1, 0);
        if (i >= 16 - 2 * NL && (i & 1)) __builtin_amdgcn_sched_group_barrier(0x020, 1, 0);
    }
}
DEV void gemm_half(f32x4 (&acc)[2][4], u32x4* b0, u32x4* b1, u32x4* b2, u32x4* b3, u32 hd, u32 xs0, u32 ws0, rsrc_t rs, const char* my,
                   int lane, int wv) {
    Pk W[4], XA[4], XB[4];
#pragma unroll
    for (int s = 0; s < UPFRONT; ++s) {
        if (s == 0) W[0] = ld_in(rs, lane, WSLOT(0));
        __builtin_amdgcn_sched_barrier(0);
        XA[s] = ld_in(rs, lane, XSLOT(s, 0));
        __builtin_amdgcn_sched_barrier(0);
        if (s + 1 < UPFRONT) W[s + 1] = ld_in(rs, lane, WSLOT(s + 1));
        __builtin_amdgcn_sched_barrier(0);
        XB[s] = ld_in(rs, lane, XSLOT(s, 1));
        __builtin_amdgcn_sched_barrier(0);
    }
#define HD(i) ((u32)__builtin_amdgcn_readlane((int)hd, (i)))
    u32x16 xa, xb;
    {
        const u32x16 wf = unpack_in(W[0], HD(4));
        u32x4* const wr = b0 + wv * 256;
        wr[lane] = FRAG(wf, 0); wr[64 + lane] = FRAG(wf, 1); wr[128 + lane] = FRAG(wf, 2); wr[192 + lane] = FRAG(wf, 3);
        xa = unpack_in(XA[0], HD(0));
        fix_x(xa, HD(0), HD(1), XSLOT(0, 0), my, lane);
        fix_w(wr, HD(4), HD(5), WSLOT(0), my, lane);
        __syncthreads();
    }
#pragma unroll
    for (int s = 0; s < 4; ++s) {
        const u32x4* const rd = s == 0 ? b0 : s == 1 ? b1 : s == 2 ? b2 : b3;
        u32x4* const wr = (s == 0 ? b1 : s == 1 ? b2 : b3) + wv * 256;
        const bool ld = s + UPFRONT < 4;
        u32 ha, hb;
        const u32 wb4 = s < 3 ? (HD(6 * s + 10) & 0x7f) * 0x01010101u : 0;
        if (s < 3) {
            dec_h(W[(s + 1) & 3].n.x, wb4, ha, hb); wr[lane] = mk_frag(W[(s + 1) & 3].l0.x, W[(s + 1) & 3].l0.y, ha, hb);
            dec_h(W[(s + 1) & 3].n.y, wb4, ha, hb); wr[64 + lane] = mk_frag(W[(s + 1) & 3].l0.z, W[(s + 1) & 3].l0.w, ha, hb);
        }
        xb = unpack_in(XB[s], HD(6 * s + 2));
        if (ld) { W[(s + UPFRONT) & 3] = ld_in(rs, lane, WSLOT(s + UPFRONT)); XA[(s + UPFRONT) & 3] = ld_in(rs, lane, XSLOT(s + UPFRONT, 0)); }
        mm16(acc, rd, xa, lane);
        if (ld) mm_order<3, 6, 1>(); else if (s < 3) mm_order<3, 0, 1>(); else mm_order<2, 0, 0>();
        fix_x(xb, HD(6 * s + 2), HD(6 * s + 3), XSLOT(s, 1), my, lane);
        if (s < 3) {
            dec_h(W[(s + 1) & 3].n.z, wb4, ha, hb); wr[128 + lane] = mk_frag(W[(s + 1) & 3].l1.x, W[(s + 1) & 3].l1.y, ha, hb);
            dec_h(W[(s + 1) & 3].n.w, wb4, ha, hb); wr[192 + lane] = mk_frag(W[(s + 1) & 3].l1.z, W[(s + 1) & 3].l1.w, ha, hb);
            xa = unpack_in(XA[(s + 1) & 3], HD(6 * s + 6));
        }
        if (ld) XB[(s + UPFRONT) & 3] = ld_in(rs, lane, XSLOT(s + UPFRONT, 1));
        mm16(acc, rd + 512, xb, lane);
        if (ld) mm_order<3, 3, 1>(); else if (s < 3) mm_order<3, 0, 1>(); else mm_order<0, 0, 0>();
        if (s < 3) {
            fix_x(xa, HD(6 * s + 6), HD(6 * s + 7), XSLOT(s + 1, 0), my, lane);
            fix_w(wr, HD(6 * s + 10), HD(6 * s + 11), WSLOT(s + 1), my, lane);
            __syncthreads();
        }
    }
}

__global__ void __launch_bounds__(256, 2) fused(const Params p) {
    __shared__ u32x4 wb0[1024], wb1[1024], wb2[1024], wb3[1024];
    __shared__ u32 red[12];
    const int lane = threadIdx.x & 63;
    const int wv = (int)rfl(threadIdx.x >> 6);
    const int rank = p.rank & 7, g = rank >> 2, q = rank & 3;
    const int bid = blockIdx.x;
    char* const my = p.my;
    const float* const px = p.x;
    const float* const pw_ = p.w;
    char* const peer = p.rel[4 + wv];
    asm volatile("" :: "s"(my), "s"(px), "s"(pw_), "s"(peer), "s"(rank));
    const rsrc_t rsm = mk_rsrc(my, WS_USED);

    Wait wt;
    wt.t0 = __builtin_amdgcn_s_memrealtime();
    wt.tmo = (gu32*)(my + O_TMO);
    gu32* const seqp = (gu32*)(my + O_SEQ + (bid * 4 + wv) * 4);
    const u32 seqv = __hip_atomic_load(seqp, __ATOMIC_RELAXED, __HIP_MEMORY_SCOPE_AGENT);
    STAMP_INIT();
    STAMP(0);
    STAMPC(60);
    if (bid < NCOMM) __builtin_amdgcn_s_setprio(3);
    else if (bid < 256) __builtin_amdgcn_s_setprio(1);
    if (threadIdx.x == 0) red[4] = 0;

    {
        const bool push = bid < NP;
        const int gw = (bid - NCOMM) * 4 + wv;
        const int cnt = push ? 384 / NP : (bid >= NCOMM && gw < 1536 ? 1 : 0);
        const int tg = push ? 1 - g : g, ks = push ? wv : (gw >> 3) & 3, ksrc = push ? 4 + q : ks;
        int i = push ? bid : ((gw >> 5) << 3) | (gw & 7);
        const rsrc_t rsd = mk_rsrc(push ? peer : my, WS_USED);
        f32x4 F[8];
        u32 D[16];
        int ch = chunk_at(i >> 3);
        ld_src(px, pw_, lane, cnt > 0, tg, ks, ch, i & 7, F);
        STAMP(62);
        float m = cvt_src(F, ch < 16, D);
        asm volatile("" : "+v"(m));
        STAMP(63);
        wt.seq = rfl(seqv) + 1;
        u32 hoff = 0xfffffff0u, hpay = 0;
        for (int it = 0; it < cnt; ++it) {
            Packed k;
            const int kind = ch < 16 ? 1 : 2;
            const u32 slot = (ksrc * 48 + ch) * 8 + (i & 7);
            i += NP;
            ch = chunk_at(i >> 3);
            ld_src(px, pw_, lane, it + 1 < cnt, tg, ks, ch, i & 7, F);
            __builtin_amdgcn_sched_barrier(0);
            if (it == 9) STAMP(19);
            pack(D, m, k, kind);
            asm volatile("" : "+v"(k.L[0]), "+v"(k.L[1]), "+v"(k.L[2]), "+v"(k.L[3]), "+v"(k.L[4]), "+v"(k.L[5]), "+v"(k.L[6]), "+v"(k.L[7]),
                              "+v"(k.N[0]), "+v"(k.N[1]), "+v"(k.N[2]), "+v"(k.N[3]));
            STAMP(40 + it);
            asm volatile("s_waitcnt vmcnt(8)" ::: "memory");
            st_in(rsd, lane, slot, k);
            st_gran_b(rsd, true, hoff, hpay, wt.seq);
            __builtin_amdgcn_sched_barrier(0);
            hoff = lane < 2 ? O_INHDR + slot * 16 + lane * 8 : 0xfffffff0u;
            hpay = lane ? k.hi : k.lo;
            STAMP(2 + it);
            m = cvt_src(F, ch < 16, D);
            asm volatile("" : "+v"(m));
            if (it == 8) STAMP(18);
        }
        WAIT_VM0();
        st_gran_b(rsd, true, hoff, hpay, wt.seq);
        if (lane == 0) __hip_atomic_store(seqp, wt.seq, __ATOMIC_RELAXED, __HIP_MEMORY_SCOPE_AGENT);
        STAMP(1);
    }

    bool ok = true;
    float y[32];
    int oc[KEEP], ocg[KEEP];
    const bool puller = bid < NCOMM;

    if (puller) {
        const bool intra = bid >= NP;
        const int pw = intra ? bid - NP : bid;
        char* const amx = p.rel[lane & 7] + O_AMAX + (rank * NCOMM + bid) * 8;
        const int sg = intra ? 0 : 1;
        const int total = intra ? N_I : N_C;
        const int cnt = (total - pw + NP - 1) / NP;
        gu32* const prog = (gu32*)(my + O_PROG + bid * 128 + wv * 4);
        const char* const hbase = my + O_OUTHDR + ((lane >> 1) & 3) * 32 + (lane & 1) * 8;
        float mx = 0.f;
        int c0 = 0, cg0 = 0, c1, cg1, c2, cg2;
        u32x3 nxt[4];
        u64 h0 = 0, h1;
        item_at(intra, pw, c1, cg1);
        item_at(intra, pw + NP, c2, cg2);
        h1 = ld_gran(hbase + (c1 * 32 + (cg1 >> 1)) * 128 + (cg1 & 1) * 16);
#pragma unroll
        for (int s = 0; s < 4; ++s) nxt[s] = (u32x3){0, 0, 0};
#pragma unroll
        for (int i = 0; i < 32; ++i) y[i] = 0.f;
#pragma unroll
        for (int r = 0; r < KEEP; ++r) { oc[r] = 0; ocg[r] = 0; }
        for (int it = -1; it < cnt; ++it) {
            const bool more = it + 1 < cnt;
            const u32 so = O_OUTDATA + ((c1 * 4 + q) * 64 + cg1) * BLK + wv * 768;
            const u32 rb = more ? WS_USED : 0u;
            WAIT_VM0();
            u32x3 cur[4];
#pragma unroll
            for (int s = 0; s < 4; ++s) cur[s] = nxt[s];
            if (more && ok) ok = poll_gran(hbase + (c1 * 32 + (cg1 >> 1)) * 128 + (cg1 & 1) * 16, true, h1, wt);
#pragma unroll
            for (int s = 0; s < 4; ++s) nxt[s] = __builtin_amdgcn_raw_buffer_load_b96(mk_rsrc(p.rel[sg * 4 + s], rb), lane * 12, so, SYS);
            __builtin_amdgcn_sched_barrier(0);
            STAMP(20 + it + 1);
            const u32 hc = (u32)h0;
            h0 = h1;
            h1 = ld_gran(hbase + (c2 * 32 + (cg2 >> 1)) * 128 + (cg2 & 1) * 16);
            if (it >= 2 && it - 2 < cnt - KEEP && ok && lane == 0)
                __hip_atomic_store(prog, wt.seq * 64 + it - 1, __ATOMIC_RELAXED, __HIP_MEMORY_SCOPE_AGENT);
            if (it >= 1 && it - 1 < cnt - KEEP) {
                const u32 o = O_Y + (oc[KEEP - 1] * 64 + ocg[KEEP - 1]) * 8192 + wv * 2048 + lane * 16;
                const float* v = y + 8 * (KEEP - 1);
                st128(__builtin_bit_cast(u32x4, (f32x4){v[0], v[1], v[2], v[3]}), rsm, o);
                st128(__builtin_bit_cast(u32x4, (f32x4){v[4], v[5], v[6], v[7]}), rsm, o + 1024);
            }
            if (it >= 0) {
                float v[8];
#pragma unroll
                for (int s = 0; s < 4; ++s) {
                    const u32 lo = (u32)__builtin_amdgcn_readlane((int)hc, 2 * s);
                    const u32 hi = (u32)__builtin_amdgcn_readlane((int)hc, 2 * s + 1);
                    u32 ha, hb;
                    dec_h(cur[s].z, (lo & 0x7f) * 0x01010101u, ha, hb);
                    if (__builtin_expect(((lo >> 7) | hi) != 0, 0)) {
                        if (lo & 0x80) {
                            const char* px = p.rel[sg * 4 + s] + O_OUTOVF + ((c0 * 4 + q) * 64 + cg0) * 1024 + lane * 16 + wv * 4;
                            ha = cur[s].z;
                            asm volatile("global_load_dword %0, %1, off sc0 sc1\n\ts_waitcnt vmcnt(0)" : "=&v"(hb) : "v"(px) : "memory");
                        } else {
                            u64 pay = (((u64)hi << 32) | lo) >> 8;
                            while (pay != 0) {
                                const u32 e = (u32)pay & 0x3ffff, idx = (e >> 6) & 31;
                                pay >>= 18;
                                const u32 t = (int)(idx >> 3) == wv ? (e >> 11) << (8 * (idx & 3)) : 0;
                                const u32 ta = (idx & 4) ? 0 : t, tb = (idx & 4) ? t : 0;
                                const bool me = lane == (int)(e & 63);
                                ha ^= me ? ta : 0;
                                hb ^= me ? tb : 0;
                            }
                        }
                    }
#pragma unroll
                    for (int j = 0; j < 4; ++j) {
                        const u32 sel = ((4u + j) << 24) | ((u32)j << 16) | 0x0c0cu;
                        const float a = __builtin_bit_cast(float, __builtin_amdgcn_perm(ha, cur[s].x, sel));
                        const float b = __builtin_bit_cast(float, __builtin_amdgcn_perm(hb, cur[s].y, sel));
                        v[j] = s ? v[j] + a : a;
                        v[4 + j] = s ? v[4 + j] + b : b;
                    }
                }
#pragma unroll
                for (int r = 0; r + 1 < KEEP; ++r) {
                    oc[r] = oc[r + 1]; ocg[r] = ocg[r + 1];
#pragma unroll
                    for (int j = 0; j < 8; ++j) y[8 * r + j] = y[8 * r + 8 + j];
                }
                oc[KEEP - 1] = c0; ocg[KEEP - 1] = cg0;
#pragma unroll
                for (int j = 0; j < 8; ++j) { const float t = fmaxf(v[j], 0.f); mx = fmaxf(mx, t); y[8 * (KEEP - 1) + j] = t; }
            }
            c0 = c1; cg0 = cg1; c1 = c2; cg1 = cg2;
            if (it + 3 < cnt) item_at(intra, pw + (it + 3) * NP, c2, cg2);
        }
        const u32 wm = wave_umax(__builtin_bit_cast(u32, mx));
        if (lane == 0) red[wv] = ok ? wm : 0xffffffffu;
        __syncthreads();
        if (wv == 0) {
            const u32 m = umax(umax(red[0], red[1]), umax(red[2], red[3]));
            if (lane < 8 && m != 0xffffffffu) st_gran(amx, m, wt.seq);
        }
        STAMP(56);
    } else {
        const int gi = bid - NCOMM;
        __syncthreads();
        const int xcd = gi & 7;
        for (int lt = gi >> 3; lt < 128; lt += NGEMM / 8) {
            const int cb = xcd >> 2, ai = (A_ROWS - cb + 1) / 2, l2 = lt >> 1;
            int c2, n2;
            if (l2 < 4 * ai) { n2 = l2 / ai; c2 = l2 - n2 * ai; }
            else if (l2 < 32) { c2 = ai + ((l2 - 4 * ai) >> 2); n2 = (l2 - 4 * ai) & 3; }
            else { n2 = l2 >> 3; c2 = l2 & 7; }
            const int c = 2 * c2 + cb, n = 4 * n2 + (xcd & 3), h = lt & 1;
            f32x4 acc[2][4];
#pragma unroll
            for (int a = 0; a < 2; ++a)
#pragma unroll
                for (int f = 0; f < 4; ++f) acc[a][f] = (f32x4){0.f, 0.f, 0.f, 0.f};
#pragma nounroll
            for (int hf = 0; hf < 2; ++hf) {
                const u32 xs0 = (hf * 4 * 48 + c) * 8 + wv * 2, ws0 = (hf * 4 * 48 + 16 + n) * 8 + wv * 2 + h;
                const int hs = (lane / 6) & 3, hk = (lane % 6) >> 1;
                const char* hp = my + O_INHDR + (hk == 2 ? WSLOT(hs) : XSLOT(hs, hk)) * 16 + (lane & 1) * 8;
                u64 hv = lane < 24 ? ld_gran(hp) : 0;
                if (ok) ok = poll_gran(hp, lane < 24, hv, wt);
                if (!ok && lane == 0) red[4] = 1;
                STAMP((lt < NGEMM / 8 ? 10 : 30) + hf * 2);
                gemm_half(acc, wb0, wb1, wb2, wb3, (u32)hv, xs0, ws0, rsm, my, lane, wv);
                STAMP((lt < NGEMM / 8 ? 11 : 31) + hf * 2);
            }
            u32 D[16];
            float m = 0.f;
#pragma unroll
            for (int rf = 0; rf < 2; ++rf)
#pragma unroll
                for (int r = 0; r < 4; ++r) {
                    D[rf * 8 + r * 2] = cvt2(acc[rf][0][r], acc[rf][1][r]);
                    D[rf * 8 + r * 2 + 1] = cvt2(acc[rf][2][r], acc[rf][3][r]);
                    m = fmaxf(fmaxf(m, fabsf(acc[rf][0][r])), fabsf(acc[rf][1][r]));
                    m = fmaxf(fmaxf(m, fabsf(acc[rf][2][r])), fabsf(acc[rf][3][r]));
                }
            Packed k;
            pack(D, m, k, 4);
            const u32 ob = (c * 4 + wv) * 64 + 2 * n + h;
            if (__builtin_expect(k.lo & 0x80, 0)) st128((u32x4){k.X[0], k.X[1], k.X[2], k.X[3]}, rsm, O_OUTOVF + ob * 1024 + lane * 16);
#pragma unroll
            for (int t = 0; t < 4; ++t)
                st96((u32x3){k.L[2 * t], k.L[2 * t + 1], k.N[t]}, rsm, O_OUTDATA + ob * BLK + lane * 12 + t * 768);
            const rsrc_t rso = mk_rsrc(p.rel[(c < A_ROWS ? 0 : 4) + wv], WS_USED);
            ok = ok && red[4] == 0;
            WAIT_VM0();
            st_gran_b(rso, ok && lane < 2, O_OUTHDR + (c * 32 + n) * 128 + q * 32 + h * 16 + lane * 8, lane ? k.hi : k.lo, wt.seq);
            STAMP(lt < NGEMM / 8 ? 14 : 34);
        }
        if (gi < NGEMM - NHOLD) { STAMP_DUMP(); return; }
        int hi = (gi - (NGEMM - NHOLD)) * 4 + wv;
        const bool intra = hi < HELD_I;
        if (!intra) hi -= HELD_I;
        item_at(intra, hi, oc[0], ocg[0]);
        gu32* const prog = (gu32*)(my + O_PROG + ((intra ? NP : 0) + hi % NP) * 128 + (lane & 3) * 4);
        const u32 need = wt.seq * 64 + hi / NP + 1;
        while (ok) {
            const u32 v = __hip_atomic_load(prog, __ATOMIC_RELAXED, __HIP_MEMORY_SCOPE_AGENT);
            if (__all(v >= need)) break;
            if (expired(wt)) ok = false;
            __builtin_amdgcn_s_sleep(POLL_SLEEP);
        }
        CFENCE();
        const u32 so = O_Y + (oc[0] * 64 + ocg[0]) * 8192;
#pragma unroll
        for (int u = 0; u < 8; ++u) {
            const f32x4 a = __builtin_bit_cast(f32x4, __builtin_amdgcn_raw_buffer_load_b128(rsm, lane * 16, so + u * 1024, SYS));
            y[4 * u] = a.x; y[4 * u + 1] = a.y; y[4 * u + 2] = a.z; y[4 * u + 3] = a.w;
        }
        STAMP(15);
    }

    {
        const int t = threadIdx.x;
        const bool two = t + 256 < NR * NCOMM;
        const char* a0 = my + O_AMAX + t * 8;
        u64 v0 = ld_gran(a0), v1 = two ? ld_gran(a0 + 2048) : 0;
        if (ok) ok = poll_gran(a0, true, v0, wt);
        if (ok) ok = poll_gran(a0 + 2048, two, v1, wt);
        const u32 wm = wave_umax(umax((u32)v0, (u32)v1));
        if (lane == 0) red[8 + wv] = ok ? wm : 0xffffffffu;
        __syncthreads();
        const u32 m = umax(umax(red[8], red[9]), umax(red[10], red[11]));
        ok = m != 0xffffffffu;
        const float am = __builtin_bit_cast(float, m);
        const float inv = am > 0.f ? 1.0f / (am / 448.0f) : 0.f;
        STAMP(57);
        const int fr = lane & 15, fq = lane >> 4;
        if (puller) {
#pragma unroll
            for (int r = 0; r < KEEP; ++r) {
                u32x2 o = {q4(y[8 * r], y[8 * r + 1], y[8 * r + 2], y[8 * r + 3], inv),
                           q4(y[8 * r + 4], y[8 * r + 5], y[8 * r + 6], y[8 * r + 7], inv)};
                if (!ok) o = (u32x2){0x7f7f7f7fu, 0x7f7f7f7fu};
                *(u32x2*)(p.out + (oc[r] * 32 + (wv >> 1) * 16 + fr) * 4096 + ocg[r] * 64 + fq * 16 + (wv & 1) * 8) = o;
            }
        } else {
#pragma unroll
            for (int rf = 0; rf < 2; ++rf) {
                u32x4 o;
#pragma unroll
                for (int j = 0; j < 4; ++j)
                    o[j] = q4(y[16 * rf + 4 * j], y[16 * rf + 4 * j + 1], y[16 * rf + 4 * j + 2], y[16 * rf + 4 * j + 3], inv);
                if (!ok) o = (u32x4){0x7f7f7f7fu, 0x7f7f7f7fu, 0x7f7f7f7fu, 0x7f7f7f7fu};
                *(u32x4*)(p.out + (oc[0] * 32 + rf * 16 + fr) * 4096 + ocg[0] * 64 + fq * 16) = o;
            }
        }
        STAMP(58);
        STAMPC(61);
        STAMP_DUMP();
    }
}

extern "C" void kernel_launch(int rank, int nranks, ncclComm_t comm, hipStream_t stream,
                              void* const* d_in, const int* in_sizes, int n_in,
                              void* d_out, int out_size,
                              const int* in_dtypes, int out_dtype,
                              void* const* d_out_all,
                              void* const* d_ws_all, size_t ws_bytes) {
    (void)comm; (void)in_dtypes; (void)out_dtype; (void)d_out_all;
    if (nranks != NR || n_in != 2 || (unsigned)rank >= (unsigned)NR || ws_bytes < WS_USED ||
        in_sizes[0] != 4096 * 512 || in_sizes[1] != 512 * 4096 || out_size != 512 * 4096) {
        fprintf(stderr, "kernel_launch: unexpected problem shape\n");
        return;
    }
    Params p;
    p.x = (const float*)d_in[0];
    p.w = (const float*)d_in[1];
    p.out = (unsigned char*)d_out;
    p.my = (char*)d_ws_all[rank];
    for (int r = 0; r < NR; ++r) p.rel[r] = (char*)d_ws_all[r ^ (rank & 4)];
    p.rank = rank;
    p.pad = 0;
    hipLaunchKernelGGL(fused, dim3(GRID), dim3(256), 0, stream, p);
    HIP_CHECK(hipGetLastError());
}
```

```cpp
#include <hip/hip_runtime.h>
#include <rccl/rccl.h>
#include <cstdio>

#define HIP_CHECK(expr) do { hipError_t _e = (expr); if (_e != hipSuccess) { \
    fprintf(stderr, "HIP error %s at %s:%d\n", hipGetErrorString(_e), __FILE__, __LINE__); \
    return; } } while (0)

typedef unsigned u32;
typedef unsigned long long u64;
typedef u32 u32x2 __attribute__((ext_vector_type(2)));
typedef u32 u32x3 __attribute__((ext_vector_type(3)));
typedef u32 u32x4 __attribute__((ext_vector_type(4)));
typedef u32 u32x8 __attribute__((ext_vector_type(8)));
typedef u32 u32x16 __attribute__((ext_vector_type(16)));
typedef float f32x2 __attribute__((ext_vector_type(2)));
typedef float f32x4 __attribute__((ext_vector_type(4)));
typedef __bf16 bf16x2 __attribute__((ext_vector_type(2)));
typedef __bf16 bf16x8 __attribute__((ext_vector_type(8)));
typedef __attribute__((address_space(1))) u32 gu32;
typedef __attribute__((address_space(1))) u64 gu64;
typedef __amdgpu_buffer_rsrc_t rsrc_t;

#define HEAD 24
#define STAG 2
#define LAG 3
#define TRIG 64
#define NPI 26
#define STAG0 0
#define STAGP 12
#define AS 128u
#define HS 128u
#define HW 32
#define PACE0 0
#define PACE 0
#define TEST_BASE_SHIFT 0

constexpr int NR = 8;
#define NPO 24
#define NKEPT 60
constexpr int NP = NPO;
constexpr int NCOMM = 2 * NP;
constexpr int GRID = 512;
constexpr int NG1 = 256 - NCOMM;
constexpr int NG2 = 256 - NG1;
constexpr int B_IN = 256 + NG1;
constexpr int NIN = (6 * NPI + 3) / 4;
constexpr int KEEP = 4;
constexpr int HELD = 256 - NKEPT;
static_assert(HELD <= NG1 && NG1 % 8 == 0 && NG2 + NIN + 96 <= NG1 && B_IN + NIN <= GRID && NKEPT <= NP * KEEP && 2 * NP <= HELD, "roles");
constexpr u64 DEADLINE = 30000000ull;

constexpr u32 FS = 128;
constexpr u32 BLK = 3072;
constexpr u32 SL = 50 * 8;
constexpr u32 N_IN = 16 * SL, N_PK = 2048, N_OUT = 3072;
constexpr u32 O_SEQ = 0;
constexpr u32 O_TMO = O_SEQ + GRID * 4 * 4;
constexpr u32 O_TICK = O_TMO + 128;
constexpr u32 O_AMAX = O_TMO + 2048;
constexpr u32 O_PROG = O_AMAX + 512 * 128;
constexpr u32 O_PKHDR = O_PROG + 512 * 128;
constexpr u32 O_OUTHDR = O_PKHDR + 12 * 256 * HS;
constexpr u32 O_INFLAG = O_OUTHDR + 512 * 128;
constexpr u32 O_IN = ((O_INFLAG + N_IN * FS + 4095) / 4096) * 4096;
constexpr u32 O_PK = O_IN + N_IN * 4096;
constexpr u32 O_OUTDATA = O_PK + N_PK * BLK;
constexpr u32 O_Y = O_OUTDATA + N_OUT * BLK;
constexpr u32 O_PKOVF = O_Y + 1024 * 8192;
constexpr u32 O_OUTOVF = O_PKOVF + N_PK * 1024;
constexpr u32 O_DBG = O_OUTOVF + N_OUT * 1024;
constexpr u32 WS_USED = O_DBG + GRID * 4 * 64 * 8;
constexpr u32 OOB = 0xfffff000u;
static_assert(O_AMAX % 128 == 0 && O_PROG % 128 == 0 && O_INFLAG % 128 == 0 && O_PKHDR % 128 == 0 && O_OUTHDR % 128 == 0, "align");
static_assert(NR * NCOMM <= 512 && GRID * 4 == 2048, "shape");

constexpr int SYS = 17;

struct Params {
    const float* x;
    const float* w;
    unsigned char* out;
    char* my;
    int rank;
    int pad;
    char* rel[NR];
};

#define STAMP(id) do { } while (0)
#define STAMPC(id) do { } while (0)
#define STAMP_INIT() const u32 stb = 0;
#define STAMP_DUMP() do { } while (0)
#define POLL_SLEEP 4

#define DEV __device__ __forceinline__
#define CFENCE() asm volatile("" ::: "memory")
#define WAIT_VM0() do { __builtin_amdgcn_s_waitcnt(0x0F70); asm volatile("s_waitcnt vmcnt(0)" ::: "memory"); } while (0)

DEV rsrc_t mk_rsrc(const void* base, u32 bytes) { return __builtin_amdgcn_make_buffer_rsrc((void*)base, 0, bytes, 0x00020000); }
DEV u32 rfl(u32 v) { return __builtin_amdgcn_readfirstlane(v); }
DEV u32 cvt2(float a, float b) { f32x2 v = {a, b}; return __builtin_bit_cast(u32, __builtin_convertvector(v, bf16x2)); }
DEV u32 umax(u32 a, u32 b) { return a > b ? a : b; }
DEV u32 wave_umax(u32 v) {
    v = umax(v, (u32)__builtin_amdgcn_update_dpp(0, (int)v, 0x111, 0xf, 0xf, true));
    v = umax(v, (u32)__builtin_amdgcn_update_dpp(0, (int)v, 0x112, 0xf, 0xf, true));
    v = umax(v, (u32)__builtin_amdgcn_update_dpp(0, (int)v, 0x114, 0xf, 0xf, true));
    v = umax(v, (u32)__builtin_amdgcn_update_dpp(0, (int)v, 0x118, 0xf, 0xf, true));
    v = umax(v, (u32)__builtin_amdgcn_update_dpp(0, (int)v, 0x142, 0xa, 0xf, false));
    v = umax(v, (u32)__builtin_amdgcn_update_dpp(0, (int)v, 0x143, 0xc, 0xf, false));
    return (u32)__builtin_amdgcn_readlane((int)v, 63);
}

struct Wait {
    u32 seq;
    u64 t0;
    gu32* tmo;
};
DEV bool expired(const Wait& wt) {
    const u64 dt = __builtin_amdgcn_s_memrealtime() - wt.t0;
    if (dt < 100000ull) return false;
    if (dt > DEADLINE || rfl(__hip_atomic_load(wt.tmo, __ATOMIC_RELAXED, __HIP_MEMORY_SCOPE_AGENT)) == wt.seq) {
        __hip_atomic_store(wt.tmo, wt.seq, __ATOMIC_RELAXED, __HIP_MEMORY_SCOPE_AGENT);
        return true;
    }
    return false;
}
DEV u64 ld_gran(const char* p) { return __hip_atomic_load((gu64*)p, __ATOMIC_RELAXED, __HIP_MEMORY_SCOPE_SYSTEM); }
DEV void st_gran(char* p, u32 payload, u32 seq) {
    __hip_atomic_store((gu64*)p, ((u64)seq << 32) | payload, __ATOMIC_RELAXED, __HIP_MEMORY_SCOPE_SYSTEM);
}
DEV bool poll_gran(const char* p, bool act, u64& v, const Wait& wt) {
    bool ok = true;
    bool done = !act || (u32)(v >> 32) >= wt.seq;
    if (__builtin_expect(!__all(done), 0)) {
        for (;;) {
            if (!done) { v = ld_gran(p); done = (u32)(v >> 32) >= wt.seq; }
            if (__all(done)) break;
            if (expired(wt)) { ok = false; break; }
            __builtin_amdgcn_s_sleep(POLL_SLEEP);
        }
    }
    CFENCE();
    return ok;
}
DEV bool poll_word(const char* p, bool act, u32 need, const Wait& wt) {
    bool ok = true, done = !act;
    for (;;) {
        if (!done) done = __hip_atomic_load((gu32*)p, __ATOMIC_RELAXED, __HIP_MEMORY_SCOPE_SYSTEM) >= need;
        if (__all(done)) break;
        if (expired(wt)) { ok = false; break; }
        __builtin_amdgcn_s_sleep(POLL_SLEEP);
    }
    CFENCE();
    return ok;
}

DEV bool poll_part(const char* p, bool& done, u64 want, const Wait& wt) {
    bool ok = true;
    if ((__ballot(done) & want) != want) {
        for (;;) {
            if (!done) done = __hip_atomic_load((gu32*)p, __ATOMIC_RELAXED, __HIP_MEMORY_SCOPE_SYSTEM) >= wt.seq;
            if ((__ballot(done) & want) == want) break;
            if (expired(wt)) { ok = false; break; }
            __builtin_amdgcn_s_sleep(POLL_SLEEP);
        }
    }
    CFENCE();
    return ok;
}

struct Packed {
    u32 L[8], N[4], X[4];
    u32 lo, hi;
};
DEV void pack(const u32 (&D)[16], float amax, Packed& o, int kind) {
    const u32 top = wave_umax(__builtin_bit_cast(u32, amax)) >> 24;
    int b = (int)(top & 0x7f) - 7 - ((TEST_BASE_SHIFT & kind) ? 1 : 0);
    b = b < 0 ? 0 : b;
    const u32 base4 = (u32)b * 0x01010101u;
    u32 H[8], d[8], chk = 0;
#pragma unroll
    for (int q = 0; q < 8; ++q) {
        o.L[q] = __builtin_amdgcn_perm(D[2 * q + 1], D[2 * q], 0x06040200u);
        H[q] = __builtin_amdgcn_perm(D[2 * q + 1], D[2 * q], 0x07050301u);
        d[q] = H[q] - base4;
        chk |= d[q];
    }
    u64 pay = (u64)b;
    u64 m = __ballot((chk & 0x78787878u) != 0);
    if (__builtin_expect(m != 0, 0)) {
        u32x8 dl;
        u32 esc = 0;
#pragma unroll
        for (int q = 0; q < 8; ++q) {
            d[q] = (((H[q] | 0x80808080u) - base4) & 0x07070707u) | (H[q] & 0x80808080u);
            dl[q] = (d[q] + base4) ^ H[q];
            esc |= ((dl[q] + 0x7f7f7f7fu) & 0x80808080u) >> (7 - q);
        }
        int cnt = 0;
        while (m != 0 && cnt <= 3) {
            const int l = __builtin_ctzll(m);
            m &= m - 1;
            u32 bits = (u32)__builtin_amdgcn_readlane((int)esc, l);
            while (bits != 0 && cnt <= 3) {
                const int pos = __builtin_ctz(bits);
                bits &= bits - 1;
                const u32 e = ((u32)__builtin_amdgcn_readlane((int)dl[pos & 7], l) >> (pos & 24)) & 0x7f;
                pay |= (u64)((u32)l | ((u32)(4 * (pos & 7) + (pos >> 3)) << 6) | (e << 11)) << (8 + 18 * cnt);
                ++cnt;
            }
        }
        if (cnt > 3) {
            pay = 0x80;
#pragma unroll
            for (int t = 0; t < 4; ++t) { d[2 * t] = H[2 * t]; d[2 * t + 1] = 0; }
        }
    }
#pragma unroll
    for (int t = 0; t < 4; ++t) {
        o.N[t] = d[2 * t] | __builtin_amdgcn_alignbit(d[2 * t + 1], d[2 * t + 1], 28);
        o.X[t] = H[2 * t + 1];
    }
    o.lo = (u32)pay;
    o.hi = (u32)(pay >> 32);
}
DEV void dec_h(u32 n, u32 base4, u32& ha, u32& hb) {
    ha = (n & 0x87878787u) + base4;
    hb = (__builtin_amdgcn_alignbit(n, n, 4) & 0x87878787u) + base4;
}
DEV u32x4 mk_frag(u32 la, u32 lb, u32 ha, u32 hb) {
    u32x4 r;
    r.x = __builtin_amdgcn_perm(ha, la, 0x05010400u);
    r.y = __builtin_amdgcn_perm(ha, la, 0x07030602u);
    r.z = __builtin_amdgcn_perm(hb, lb, 0x05010400u);
    r.w = __builtin_amdgcn_perm(hb, lb, 0x07030602u);
    return r;
}
struct Pk { u32x4 l0, l1, n; };
DEV u32x16 cat4(u32x4 a, u32x4 b, u32x4 c, u32x4 d) {
    const u32x8 l = __builtin_shufflevector(a, b, 0, 1, 2, 3, 4, 5, 6, 7), h = __builtin_shufflevector(c, d, 0, 1, 2, 3, 4, 5, 6, 7);
    return __builtin_shufflevector(l, h, 0, 1, 2, 3, 4, 5, 6, 7, 8, 9, 10, 11, 12, 13, 14, 15);
}
#define FRAG(v, t) __builtin_shufflevector(v, v, 4 * (t), 4 * (t) + 1, 4 * (t) + 2, 4 * (t) + 3)
DEV u32x16 unpack(const Pk& k, u32 lo, u32 hi, const char* ovf, int lane) {
    const u32 base4 = (lo & 0x7f) * 0x01010101u;
    u32x4 ha, hb;
#pragma unroll
    for (int t = 0; t < 4; ++t) { u32 a, b; dec_h(k.n[t], base4, a, b); ha[t] = a; hb[t] = b; }
    if (__builtin_expect(((lo >> 7) | hi) != 0, 0)) {
        if (lo & 0x80) {
            ha = k.n;
            asm volatile("global_load_dwordx4 %0, %1, off sc0 sc1\n\ts_waitcnt vmcnt(0)" : "=&v"(hb) : "v"(ovf) : "memory");
        } else {
            u64 pay = (((u64)hi << 32) | lo) >> 8;
            while (pay != 0) {
                const u32 e = (u32)pay & 0x3ffff, idx = (e >> 6) & 31;
                pay >>= 18;
                const u32 t = lane == (int)(e & 63) ? (e >> 11) << (8 * (idx & 3)) : 0;
#pragma unroll
                for (u32 z = 0; z < 4; ++z) {
                    ha[z] ^= (idx >> 2) == 2 * z ? t : 0;
                    hb[z] ^= (idx >> 2) == 2 * z + 1 ? t : 0;
                }
            }
        }
    }
    return cat4(mk_frag(k.l0.x, k.l0.y, ha.x, hb.x), mk_frag(k.l0.z, k.l0.w, ha.y, hb.y),
                mk_frag(k.l1.x, k.l1.y, ha.z, hb.z), mk_frag(k.l1.z, k.l1.w, ha.w, hb.w));
}
DEV void st128(u32x4 v, rsrc_t rs, u32 off) { __builtin_amdgcn_raw_buffer_store_b128(v, rs, off, 0, SYS); }
DEV void st96(u32x3 v, rsrc_t rs, u32 off) { __builtin_amdgcn_raw_buffer_store_b96(v, rs, off, 0, SYS); }
DEV void st_gran_b(rsrc_t rs, bool act, u32 off, u32 payload, u32 seq) {
    const u32x2 v = {payload, seq};
    __builtin_amdgcn_raw_buffer_store_b64(v, rs, act ? off : OOB, 0, SYS);
}

DEV void st_word_b(rsrc_t rs, bool act, u32 off, u32 v) { __builtin_amdgcn_raw_buffer_store_b32(v, rs, act ? off : OOB, 0, SYS); }

DEV int shell_of(int e) {
    const int k = e - LAG;
    return e < LAG ? e : e >= 16 - LAG ? e : (k & 1) ? LAG + (k >> 1) : 8 + (k >> 1);
}
DEV int sto(int ch) { return ch < 16 ? (ch & 3) * 4 + (ch >> 2) : 16 + (ch & 1) * 8 + ((ch - 16) >> 1); }
DEV int item_at(int i, int src) {
    const int e = i / 30, ps = shell_of(e);
    int r = i - e * 30;
    if ((r >> 1) >= src) r += 2;
    return (ps >> 3) * 256 + sto((r >> 4) * 16 + (ps & 7) * 2 + ((r >> 3) & 1)) * 8 + (r & 7);
}
DEV void ld_src(const float* px, const float* pw, int lane, int cls, int ks, int chunk, int blk, f32x4 (&F)[8]) {
    const int fr = lane & 15, fq = lane >> 4;
    const bool isx = chunk < 16;
    const rsrc_t rs = mk_rsrc(isx ? px : pw, 4096u * 512u * 4u);
    u32 so, s2, s1, s0, vo;
    if (isx) {
        const int owner = (chunk & 1) * 4 + (blk >> 1);
        so = ((owner * 512 + (cls * 8 + (chunk >> 1)) * 32) * 512 + ks * 128 + (blk & 1) * 64) * 4;
        s2 = 16 * 2048; s1 = 128; s0 = 64;
        vo = fr * 2048 + fq * 16;
    } else {
        so = ((ks * 128 + (blk >> 1) * 32) * 4096 + cls * 2048 + (chunk - 16) * 128 + (blk & 1) * 64) * 4;
        s2 = 16 * 16384; s1 = 2 * 16384; s0 = 16384;
        vo = fq * 4 * 16384 + fr * 16;
    }
#pragma unroll
    for (int u = 0; u < 8; ++u)
        F[u] = __builtin_bit_cast(f32x4, __builtin_amdgcn_raw_buffer_load_b128(
                   rs, vo, so + (u >> 2) * s2 + ((u >> 1) & 1) * s1 + (u & 1) * s0, 0));
}
DEV float cvt_src(const f32x4 (&F)[8], bool isx, u32 (&D)[16]) {
    if (isx) {
#pragma unroll
        for (int u = 0; u < 8; ++u) { D[2 * u] = cvt2(F[u].x, F[u].y); D[2 * u + 1] = cvt2(F[u].z, F[u].w); }
    } else {
#pragma unroll
        for (int f = 0; f < 4; ++f)
#pragma unroll
            for (int e = 0; e < 4; ++e) D[4 * f + e] = cvt2(F[2 * e][f], F[2 * e + 1][f]);
    }
    float m = 0.f;
#pragma unroll
    for (int u = 0; u < 8; ++u) {
        m = fmaxf(fmaxf(m, fabsf(F[u].x)), fabsf(F[u].y));
        m = fmaxf(fmaxf(m, fabsf(F[u].z)), fabsf(F[u].w));
    }
    return m;
}
DEV void st_plain(const u32 (&D)[16], rsrc_t rs, u32 off) {
#pragma unroll
    for (int t = 0; t < 4; ++t) st128((u32x4){D[4 * t], D[4 * t + 1], D[4 * t + 2], D[4 * t + 3]}, rs, off + t * 1024);
}
DEV void shell_at(int t, int& a, int& b) {
    int s = 0;
    while (2 * (s + 1) * (s + 1) <= t) ++s;
    int r = t - 2 * s * s;
    if (r < 2 * s) { a = s; b = r; }
    else { r -= 2 * s; const int up = r > s; a = r - up * (s + 1); b = 2 * s + up; }
}
DEV u32 q4(float a, float b, float c, float d, float inv) {
    int r = __builtin_amdgcn_cvt_pk_fp8_f32(a * inv, b * inv, 0, false);
    r = __builtin_amdgcn_cvt_pk_fp8_f32(c * inv, d * inv, r, true);
    return (u32)r;
}

#define XSLOT(s) (xs0 + ((s) >> 1) * SL + ((s) & 1))
#define WSLOT(s) (ws0 + ((s) >> 1) * SL + ((s) & 1) * 4)
#define SB() __builtin_amdgcn_sched_barrier(0)
template <int F>
DEV void ld_x(u32x4& d, u32x4 sd, u32 vo, u32 so) {
    asm volatile("buffer_load_dwordx4 %0, %1, %2, %3 offen offset:%4 sc0 sc1" : "+v"(d) : "v"(vo), "s"(sd), "s"(so), "i"(F * 1024));
}
template <int N>
DEV void wait_step(u32x4 (&x)[4]) {
    __builtin_amdgcn_s_waitcnt(0xC07F);
    asm volatile("s_waitcnt vmcnt(%4) lgkmcnt(0)\n\ts_barrier" : "+v"(x[0]), "+v"(x[1]), "+v"(x[2]), "+v"(x[3]) : "i"(N) : "memory");
}
DEV void mma(f32x4& c, u32x4 a, u32x4 b) { asm volatile("v_mfma_f32_16x16x32_bf16 %0, %1, %2, %0" : "+v"(c) : "v"(a), "v"(b)); }
template <int F>
DEV void mma_ldw(f32x4& c, u32x4 a, u32x4 b, u32 lds, u32x4 sd, u32 vo, u32 so) {
    asm volatile("s_mov_b32 m0, %3\n\tv_mfma_f32_16x16x32_bf16 %0, %1, %2, %0\n\tbuffer_load_dwordx4 %4, %5, %6 offen offset:%7 sc0 sc1 lds"
                 : "+v"(c) : "v"(a), "v"(b), "s"(lds), "v"(vo), "s"(sd), "s"(so), "i"(F * 1024) : "memory");
}
template <int S, int G>
DEV void gemm_steps(f32x4 (&acc)[2][8], u32x4 (&X)[4][4], u32x4 (&a)[8], const u32x4* ring, u32 lds0, u32x4 sd, u32 xs0, u32 ws0,
                    int skip, int lane, u32 stb) {
    const u32 vo = lane * 16;
    u32x4 b[8];
    const u32 vx = ((S + 3) >> 1) == skip ? 0x80000000u : vo;
    const u32 sx = O_IN + XSLOT(S + 3) * 4096, sw = O_IN + WSLOT(S + 4) * 4096, l = lds0 + S * 16384;
    constexpr bool LX = G < 2 || S == 0, LW = G < 2, END = G == 2 && S == 3;
#pragma unroll
    for (int i = 0; i < 8; ++i) {
        mma(acc[0][i], a[i], X[S][0]);
        SB();
        if (i < 6) b[i < 4 ? i + (i + 1) / 2 : i + 2] = ring[S * 1024 + (8 + (i < 4 ? i + (i + 1) / 2 : i + 2)) * 64 + lane];
        SB();
        mma(acc[1][i], a[i], X[S][2]);
        SB();
        if (i == 0 || i == 2) b[i + i / 2 + 1] = ring[S * 1024 + (8 + i + i / 2 + 1) * 64 + lane];
        if (LX && i == 1) ld_x<0>(X[(S + 3) & 3][0], sd, vx, sx);
        if (LX && i == 3) ld_x<1>(X[(S + 3) & 3][1], sd, vx, sx);
        if (LX && i == 4) ld_x<2>(X[(S + 3) & 3][2], sd, vx, sx);
        if (LX && i == 6) ld_x<3>(X[(S + 3) & 3][3], sd, vx, sx);
        SB();
    }
    if (!END) wait_step<(G == 0 && S == 0 ? 8 : G == 0 && S == 1 ? 12 : G < 2 || S == 0 ? 16 : S == 1 ? 8 : 0)>(X[(S + 1) & 3]);
    SB();
#pragma unroll
    for (int i = 0; i < 8; ++i) {
        mma(acc[0][i], b[i], X[S][1]);
        SB();
        if (!END) a[i] = ring[((S + 1) & 3) * 1024 + i * 64 + lane];
        SB();
        if (LW && i == 0) mma_ldw<0>(acc[1][i], b[i], X[S][3], l, sd, vo, sw);
        else if (LW && i == 2) mma_ldw<1>(acc[1][i], b[i], X[S][3], l, sd, vo, sw);
        else if (LW && i == 4) mma_ldw<2>(acc[1][i], b[i], X[S][3], l, sd, vo, sw);
        else if (LW && i == 6) mma_ldw<3>(acc[1][i], b[i], X[S][3], l, sd, vo, sw);
        else mma(acc[1][i], b[i], X[S][3]);
        SB();
    }
    if constexpr (S < 3) gemm_steps<S + 1, G>(acc, X, a, ring, lds0, sd, xs0, ws0, skip, lane, stb);
}
DEV void first_w(u32 lds0, u32x4 sd, u32 ws0, int lane) {
#pragma unroll
    for (int s = 0; s < 4; ++s) {
        const u32 sw = O_IN + WSLOT(s) * 4096, l = lds0 + s * 16384, vo = lane * 16;
        asm volatile("s_mov_b32 m0, %0\n\ts_nop 0\n\tbuffer_load_dwordx4 %1, %2, %3 offen sc0 sc1 lds\n\t"
                     "buffer_load_dwordx4 %1, %2, %3 offen offset:1024 sc0 sc1 lds\n\tbuffer_load_dwordx4 %1, %2, %3 offen offset:2048 sc0 sc1 lds\n\t"
                     "buffer_load_dwordx4 %1, %2, %3 offen offset:3072 sc0 sc1 lds" :: "s"(l), "v"(vo), "s"(sd), "s"(sw) : "memory");
    }
}
DEV bool gemm_seg(f32x4 (&acc)[2][8], u32x4 (&X)[4][4], const u32x4* ring, u32 lds0, u32x4 sd, u32 xs0, u32 ws0, int n, int skip,
                  bool wfirst, bool ok, const char* my, const Wait& wt, u32* fail, int lane, u32 stb) {
    u32x4 a[8];
    const char* const fp = my + O_INFLAG + ((lane & 16) ? WSLOT(lane & 15) : XSLOT(lane & 15)) * FS;
    bool done = lane >= 32 || (lane & 15) >= n || (lane >> 1) == skip;
    asm volatile("s_waitcnt lgkmcnt(0)\n\ts_barrier" ::: "memory");
    if (wfirst) {
        if (ok) ok = poll_part(fp, done, 0xffff0000ull, wt);
        first_w(lds0, sd, ws0, lane);
    }
    if (ok) ok = poll_part(fp, done, 0xffffull, wt);
    STAMP(38);
#pragma unroll
    for (int s = 0; s < 3; ++s) {
        const u32 sx = O_IN + XSLOT(s) * 4096, vx = (s >> 1) == skip ? 0x80000000u : lane * 16;
        ld_x<0>(X[s][0], sd, vx, sx); ld_x<1>(X[s][1], sd, vx, sx); ld_x<2>(X[s][2], sd, vx, sx); ld_x<3>(X[s][3], sd, vx, sx);
    }
    if (!wfirst) {
        if (ok) ok = poll_part(fp, done, 0xffff0000ull, wt);
        STAMP(38);
        first_w(lds0, sd, ws0, lane);
    } else {
        asm volatile("s_waitcnt vmcnt(8)" ::: "memory");
    }
    if (!ok && lane == 0) *fail = 1;
    STAMP(40);
    wait_step<12>(X[0]);
    STAMP(41);
    SB();
#pragma unroll
    for (int i = 0; i < 8; ++i) a[i] = ring[i * 64 + lane];
    SB();
    gemm_steps<0, 0>(acc, X, a, ring, lds0, sd, xs0, ws0, skip, lane, stb);
    xs0 += 2 * SL; ws0 += 2 * SL; skip -= 2;
    if (n > 8) {
        gemm_steps<0, 1>(acc, X, a, ring, lds0, sd, xs0, ws0, skip, lane, stb);
        xs0 += 2 * SL; ws0 += 2 * SL; skip -= 2;
    }
    gemm_steps<0, 2>(acc, X, a, ring, lds0, sd, xs0, ws0, skip, lane, stb);
    return ok;
}

__global__ void __launch_bounds__(256, 2) fused(const Params p) {
    __shared__ u32x4 ring[4096];
    __shared__ u32 red[12];
    const int lane = threadIdx.x & 63;
    const int wv = (int)rfl(threadIdx.x >> 6);
    const int rank = p.rank & 7, rm = rank & 1, ri = (rank >> 1) & 1, rj = rank >> 2;
    const int bid = blockIdx.x;
    char* const my = p.my;
    const float* const px = p.x;
    const float* const pw_ = p.w;
    asm volatile("" :: "s"(my), "s"(px), "s"(pw_), "s"(rank));
    const rsrc_t rsm = mk_rsrc(my, WS_USED);

    Wait wt;
    wt.t0 = __builtin_amdgcn_s_memrealtime();
    wt.tmo = (gu32*)(my + O_TMO);
    gu32* const seqp = (gu32*)(my + O_SEQ + (bid * 4 + wv) * 4);
    const u32 seqv = __hip_atomic_load(seqp, __ATOMIC_RELAXED, __HIP_MEMORY_SCOPE_AGENT);
    STAMP_INIT();
    STAMP(0);
    STAMPC(60);
    if (threadIdx.x == 0) red[4] = 0;

    const bool inpull = bid >= B_IN && bid < B_IN + NIN;
    wt.seq = 0;
#pragma nounroll
    for (int wg = inpull ? GRID : bid < HW ? bid : bid < NG1 ? bid + NCOMM : bid < 256 ? bid - NG1 + HW : bid < B_IN ? bid : bid - NIN; wg < GRID;
         wg = (wg >= 256 + NG2 && wg < 256 + NG2 + NIN) ? wg + GRID - NIN - 256 - NG2 : GRID) {
        const int u = wg * 4 + wv;
        const int pos = u >> 5, dm = (u >> 4) & 1, cls = (u >> 3) & 1, blk = u & 7;
        const int ps = shell_of(pos >> 2), ch = (pos & 2) * 8 + (ps & 7) * 2 + (pos & 1), pass = ps >> 3, ks = 2 * dm + pass;
        for (int z = wg < GRID - NIN ? pos * STAG + (pos >= STAGP ? STAG0 : 0) : 0; z > 0; --z) __builtin_amdgcn_s_sleep(1);
        f32x4 F[8];
        u32 D[16];
        ld_src(px, pw_, lane, cls, ks, ch, blk, F);
        STAMP(62);
        const bool isx = ch < 16, own = isx && (ch & 1) * 4 + (blk >> 1) == rank, same = cls == (isx ? ri : rj);
        const int dc = same ? 0 : (isx ? 2 : 4), d0 = (dc ^ (isx ? 4 : 2)) | (dm ^ rm), d1 = dc | (dm ^ rm);
        const rsrc_t rs0 = mk_rsrc(p.rel[d0], own ? 0u : WS_USED), rs1 = mk_rsrc(p.rel[d1], own || same ? 0u : WS_USED);
        const u32 cb8 = sto(ch) * 8 + blk;
        const u32 r0 = pass * 6 + d0 - 2, r1 = same ? 0 : pass * 6 + d1 - 2;
        const u32 ls = ks * SL + (same ? cb8 : isx ? 256 + (ch >> 1) * 2 + (blk & 1) : 144 + cb8);
        const int r32 = (pos & 3) * 8 + blk;
        const int item = (pos >> 2) * 30 + r32 - ((r32 >> 1) > rank ? 2 : 0);
        const bool local = same || own || !isx, head = item < HEAD;
        const float m = cvt_src(F, isx, D);
        STAMP(63);
        u32 hpay = 0;
        if (head) {
            if (PACE) while (__builtin_amdgcn_s_memrealtime() - wt.t0 < (u64)(PACE0 + item * PACE / 16)) __builtin_amdgcn_s_sleep(1);
            st_plain(D, rs0, O_IN + ((4 + r0) * SL + cb8) * 4096 + lane * 16);
            st_plain(D, rs1, O_IN + ((4 + r1) * SL + cb8) * 4096 + lane * 16);
        } else if (!own) {
            Packed k;
            pack(D, m, k, isx ? 1 : 2);
            const u32 pk = (ks * 2 + cls) * 256 + cb8, o = O_PK + pk * BLK + lane * 16;
            if (__builtin_expect(k.lo & 0x80, 0)) st128((u32x4){k.X[0], k.X[1], k.X[2], k.X[3]}, rsm, O_PKOVF + pk * 1024 + lane * 16);
            st128((u32x4){k.L[0], k.L[1], k.L[2], k.L[3]}, rsm, o);
            st128((u32x4){k.L[4], k.L[5], k.L[6], k.L[7]}, rsm, o + 1024);
            st128((u32x4){k.N[0], k.N[1], k.N[2], k.N[3]}, rsm, o + 2048);
            hpay = lane ? k.hi : k.lo;
        }
        if (local) st_plain(D, rsm, O_IN + ls * 4096 + lane * 16);
        wt.seq = rfl(seqv) + 1;
        STAMP(61);
        WAIT_VM0();
        if (head) {
            st_word_b(rs0, lane == 0, O_INFLAG + ((4 + r0) * SL + cb8) * FS, wt.seq);
            st_word_b(rs1, lane == 0, O_INFLAG + ((4 + r1) * SL + cb8) * FS, wt.seq);
        } else {
            st_gran_b(rs0, lane < 2, O_PKHDR + (r0 * 256 + cb8) * HS + lane * 8, hpay, wt.seq);
            st_gran_b(rs1, lane < 2, O_PKHDR + (r1 * 256 + cb8) * HS + lane * 8, hpay, wt.seq);
        }
        st_word_b(rsm, local && lane == 0, O_INFLAG + ls * FS, wt.seq);
        STAMP(1);
    }
    wt.seq = rfl(seqv) + 1;
    if (lane == 0) __hip_atomic_store(seqp, wt.seq, __ATOMIC_RELAXED, __HIP_MEMORY_SCOPE_AGENT);

    bool ok = true;
    float y[16 * KEEP];
    int oi[KEEP], oc[KEEP], ocg[KEEP];

    if (bid >= B_IN) {
        const int iw = (bid - B_IN) * 4 + wv, l = iw % 6, ip = iw / 6;
        if (ip >= NPI) { STAMP_DUMP(); return; }
        const int src = rank ^ (l + 2);
        char* const peer = p.rel[l + 2];
        const rsrc_t rs = mk_rsrc(peer, WS_USED);
        const char* const hb = my + O_PKHDR + l * 256 * HS + (lane & 1) * 8;
        constexpr int D0 = HEAD + 2 * NPI, ND = 480 - D0;
        const u32 tbase = (wt.seq - 1) * (ND + NPI) - D0;
        int cntr = 2;
#define HOFF(b) (((b) & 255) * HS + (((b) >> 8) & 1) * (6 * 256 * HS))
#define PKIDX(b) (((2 * rm + (((b) >> 8) & 1)) * 2 + (((b) & 128) ? rj : ri)) * 256 + ((b) & 255))
        int b0 = item_at(HEAD + ip, src);
        u32 tk = tbase + HEAD + NPI + ip, fo = OOB;
        u64 h0 = ld_gran(hb + HOFF(b0));
        WAIT_VM0();
        do {
            const u32 so = O_PK + PKIDX(b0) * BLK;
            if (ok) ok = poll_gran(hb + HOFF(b0), true, h0, wt);
            Pk k;
            k.l0 = __builtin_amdgcn_raw_buffer_load_b128(rs, lane * 16, so, SYS);
            k.l1 = __builtin_amdgcn_raw_buffer_load_b128(rs, lane * 16 + 1024, so, SYS);
            k.n = __builtin_amdgcn_raw_buffer_load_b128(rs, lane * 16 + 2048, so, SYS);
            SB();
            STAMP(cntr); ++cntr;
            const u32 lo = (u32)__builtin_amdgcn_readlane((int)(u32)h0, 0), hi = (u32)__builtin_amdgcn_readlane((int)(u32)h0, 1);
            const u32 i1 = rfl(tk) - tbase;
            const int b1 = i1 < 480u ? item_at(i1, src) : -1;
            h0 = ld_gran(hb + HOFF(b1));
            tk = __builtin_amdgcn_raw_ptr_buffer_atomic_add_i32(1, rsm, (b1 >= 0 && lane == 0) ? O_TICK + l * 128 : OOB, 0, 16);
            asm volatile("s_waitcnt vmcnt(5)" ::: "memory");
            st_word_b(rsm, ok, fo, wt.seq);
            const u32 slot = (4 + (b0 >> 8) * 6 + l) * SL + (b0 & 255), o = O_IN + slot * 4096 + lane * 16;
            fo = lane == 0 ? O_INFLAG + slot * FS : OOB;
            const u32x16 d = unpack(k, lo, hi, peer + O_PKOVF + PKIDX(b0) * 1024 + lane * 16, lane);
            asm volatile("" : "+v"(h0), "+v"(tk));
            st128(FRAG(d, 0), rsm, o); st128(FRAG(d, 1), rsm, o + 1024); st128(FRAG(d, 2), rsm, o + 2048); st128(FRAG(d, 3), rsm, o + 3072);
            SB();
            b0 = b1;
        } while (b0 >= 0);
        WAIT_VM0();
        st_word_b(rsm, ok, fo, wt.seq);
        STAMP(30);
        STAMP_DUMP();
        return;
    }
    const bool puller = bid >= NG1 && bid < 256;
    const int hi = bid < 256 ? bid - NG2 : bid - 256 + NG1 - NG2;
    if (!puller) {
        const bool mainT = bid < 256 + NG2;
        if (!mainT && bid < B_IN - 96) { STAMP_DUMP(); return; }
        const int xcd = bid & 7, k8 = (bid & 255) >> 3;
        __syncthreads();
        const u32 lds0 = (u32)(size_t)(__attribute__((address_space(3))) u32x4*)ring + wv * 4096;
        const u32x4 sd = {(u32)(size_t)my, (u32)((size_t)my >> 32) & 0xffffu, WS_USED, 0x00020000u};
        u32 xs0, ws0, ob, ho;
        int od;
        bool wfirst = false;
        if (mainT) {
            int a, b;
            shell_at(bid < 256 ? k8 : 31 - k8, a, b);
            const int cx = 4 * a + (xcd >> 1), n = 2 * b + (xcd & 1);
            xs0 = ((xcd >> 1) * 4 + a) * 8 + wv * 2;
            ws0 = (16 + (xcd & 1) * 8 + b) * 8 + wv;
            ob = (cx * 4 + wv) * 32 + 2 * n;
            ho = ((ri * 8 + (cx >> 1)) * 32 + rj * 16 + n) * 128 + rm * 32;
            od = ((cx & 1) * 4 + wv) ^ rank;
            wfirst = (cx >> 1) > (n >> 1);
        } else {
            const int e = xcd * 12 + k8 - (NG1 - 96) / 8, dq = (e >> 5) + 1, c = ((e >> 4) & 1) * 4 + wv, n = e & 15;
            xs0 = (dq & 1) ? 256 + c * 2 : sto(2 * c + rj) * 8 + (rank & 3) * 2;
            ws0 = ((dq & 2) ? 144 : 0) + sto(16 + n) * 8 + wv;
            const int rg = (ri ^ (dq & 1)) * 8 + c, n32 = (rj ^ (dq >> 1)) * 16 + n;
            ob = 2048 + rg * 64 + 2 * n32;
            ho = (rg * 32 + n32) * 128 + 64;
            od = 0;
        }
        f32x4 acc[2][8];
#pragma unroll
        for (int a = 0; a < 2; ++a)
#pragma unroll
            for (int f = 0; f < 8; ++f) acc[a][f] = (f32x4){0.f, 0.f, 0.f, 0.f};
        u32x4 X[4][4];
#pragma unroll
        for (int i = 0; i < 16; ++i) X[i >> 2][i & 3] = (u32x4){0, 0, 0, 0};
#pragma nounroll
        for (int seg = 0; seg < (mainT ? 3 : 1); ++seg) {
            ok = gemm_seg(acc, X, ring, lds0, sd, xs0, ws0, seg ? 12 : 8, seg ? od - 2 : -1, wfirst, ok, my, wt, &red[4], lane, stb);
            STAMP(11 + 2 * seg);
            xs0 += (seg ? 6 : 4) * SL; ws0 += (seg ? 6 : 4) * SL;
        }
        asm volatile("s_nop 15" ::: "memory");
        SB();
        u32 hpay = 0;
#pragma nounroll
        for (int h = 0; h < 2; ++h) {
            u32 D[16];
            float m = 0.f;
#pragma unroll
            for (int rf = 0; rf < 2; ++rf)
#pragma unroll
                for (int r = 0; r < 4; ++r) {
                    D[rf * 8 + r * 2] = cvt2(acc[rf][0][r], acc[rf][1][r]);
                    D[rf * 8 + r * 2 + 1] = cvt2(acc[rf][2][r], acc[rf][3][r]);
                    m = fmaxf(fmaxf(m, fabsf(acc[rf][0][r])), fabsf(acc[rf][1][r]));
                    m = fmaxf(fmaxf(m, fabsf(acc[rf][2][r])), fabsf(acc[rf][3][r]));
                }
            Packed k;
            pack(D, m, k, 4);
            if (__builtin_expect(k.lo & 0x80, 0)) st128((u32x4){k.X[0], k.X[1], k.X[2], k.X[3]}, rsm, O_OUTOVF + (ob + h) * 1024 + lane * 16);
#pragma unroll
            for (int t = 0; t < 4; ++t)
                st96((u32x3){k.L[2 * t], k.L[2 * t + 1], k.N[t]}, rsm, O_OUTDATA + (ob + h) * BLK + lane * 12 + t * 768);
            hpay = (lane >> 1) == h ? ((lane & 1) ? k.hi : k.lo) : hpay;
#pragma unroll
            for (int rf = 0; rf < 2; ++rf)
#pragma unroll
                for (int f = 0; f < 4; ++f) acc[rf][f] = acc[rf][4 + f];
        }
        const rsrc_t rso = mk_rsrc(p.rel[od & 7], WS_USED);
        ok = ok && red[4] == 0;
        WAIT_VM0();
        st_gran_b(rso, ok && lane < 4, O_OUTHDR + ho + lane * 8, hpay, wt.seq);
        STAMP(16);
        if (hi < 0 || hi >= HELD) { STAMP_DUMP(); return; }
    }

    u32 owed = OOB;
    int rgA, cgA, rgB, cgB;
    if (puller) {
        const int pid = bid - NG1, grp = pid >= NP, pw = pid - grp * NP;
        const int dqa = grp ? 3 : 1, dqb = grp ? 0 : 2;
        rgA = (ri ^ (dqa & 1)) * 8; cgA = (rj ^ (dqa >> 1)) * 32; rgB = (ri ^ (dqb & 1)) * 8; cgB = (rj ^ (dqb >> 1)) * 32;
        char* const amx = p.rel[lane & 7] + O_AMAX + (rank * NCOMM + pid) * AS;
        const u32 tko = O_TICK + (6 + grp * 4 + wv) * 128, tbase = (wt.seq - 1) * (256 - NP) - 2 * NP;
        const bool hact = lane < (grp ? 10 : 12);
        const int hl = lane < 12 ? lane : 0;
        const char* const hbase = my + O_OUTHDR + (hl < 6 ? rgA * 32 + cgA / 2 : rgB * 32 + cgB / 2) * 128 + ((hl % 6) >> 1) * 32 + (hl & 1) * 8;
        const char* sp[6];
        sp[0] = p.rel[dqa * 2 + rm]; sp[1] = p.rel[dqa * 2 + (rm ^ 1)]; sp[2] = my;
        sp[3] = p.rel[dqb * 2 + rm]; sp[4] = p.rel[dqb * 2 + (rm ^ 1)]; sp[5] = my;
#define OBLK(s, c, cg) ((s) % 3 < 2 ? (u32)((((c) * 2 + rj) * 4 + (rank & 3)) * 32 + (cg)) \
                                    : (u32)(2048 + (((s) < 3 ? rgA : rgB) + (c)) * 64 + ((s) < 3 ? cgA : cgB) + (cg)))
#define OHDR(c, cg) (((c) * 32 + ((cg) >> 1)) * 128 + ((cg) & 1) * 16)
        {
            const int l = (grp * 4 + wv) % 6, b = item_at(480 - TRIG + pw, rank ^ (l + 2));
            if (ok) ok = poll_word(my + O_INFLAG + ((4 + (b >> 8) * 6 + l) * SL + (b & 255)) * FS, lane == 0, wt.seq, wt);
        }
        float mx = 0.f;
        int i0 = -1, i1 = pw, c0 = 0, cg0 = 0, c1, cg1, nkeep = 0;
        u32x3 nxt[6];
        u64 h0 = 0, h1;
        u32 tk = tbase + NP + pw, fo = OOB;
        shell_at(pw >> 1, c1, cg1); cg1 = 2 * cg1 + (pw & 1);
        h1 = ld_gran(hbase + OHDR(c1, cg1));
#pragma unroll
        for (int s = 0; s < 6; ++s) nxt[s] = (u32x3){0, 0, 0};
#pragma unroll
        for (int i = 0; i < 16 * KEEP; ++i) y[i] = 0.f;
#pragma unroll
        for (int r = 0; r < KEEP; ++r) { oi[r] = -1; oc[r] = 0; ocg[r] = 0; }
        auto hand_over = [&]() {
            if (oi[KEEP - 1] >= 0 && oi[KEEP - 1] < HELD) {
                const u32 o = O_Y + (oc[KEEP - 1] * 64 + ocg[KEEP - 1]) * 8192 + wv * 2048 + lane * 16;
                const u32 oa = o + (rgA * 64 + cgA) * 8192, ob = o + (rgB * 64 + cgB) * 8192;
                const float* v = y + 16 * (KEEP - 1);
                st128(__builtin_bit_cast(u32x4, (f32x4){v[0], v[1], v[2], v[3]}), rsm, oa);
                st128(__builtin_bit_cast(u32x4, (f32x4){v[4], v[5], v[6], v[7]}), rsm, oa + 1024);
                st128(__builtin_bit_cast(u32x4, (f32x4){v[8], v[9], v[10], v[11]}), rsm, ob);
                st128(__builtin_bit_cast(u32x4, (f32x4){v[12], v[13], v[14], v[15]}), rsm, ob + 1024);
                fo = lane == 0 ? O_PROG + (grp * 256 + oi[KEEP - 1]) * 128 + wv * 4 : OOB;
            }
        };
        int cntr = 20;
        do {
            const bool more = i1 >= 0;
            const u32 rb = more ? WS_USED : 0u;
            WAIT_VM0();
            u32x3 cur[6];
#pragma unroll
            for (int s = 0; s < 6; ++s) cur[s] = nxt[s];
            if (more && ok) ok = poll_gran(hbase + OHDR(c1, cg1), hact, h1, wt);
#pragma unroll
            for (int z = 0; z < 6; ++z) {
                const int s = z < 2 ? z : z < 4 ? z + 1 : z == 4 ? 2 : 5;
                nxt[s] = __builtin_amdgcn_raw_buffer_load_b96(mk_rsrc(sp[s], (s == 5 && grp) ? 0u : rb), lane * 12,
                                                              O_OUTDATA + OBLK(s, c1, cg1) * BLK + wv * 768, SYS);
            }
            SB();
            STAMP(cntr); ++cntr;
            st_word_b(rsm, ok, fo, wt.seq);
            fo = OOB;
            hand_over();
            const u32 hc = hact ? (u32)h0 : 0u;
            h0 = h1;
            int i2 = -1, c2 = 0, cg2 = 0;
            if (more && nkeep < KEEP) {
                const u32 t = rfl(tk) - tbase;
                if (t < 256u) {
                    i2 = t;
                    shell_at(i2 >> 1, c2, cg2); cg2 = 2 * cg2 + (i2 & 1);
                    nkeep += i2 >= HELD;
                    h1 = ld_gran(hbase + OHDR(c2, cg2));
                    tk = __builtin_amdgcn_raw_ptr_buffer_atomic_add_i32(1, rsm, (nkeep < KEEP && lane == 0) ? tko : OOB, 0, 16);
                }
            }
            if (i0 >= 0) {
                float v[16];
#pragma unroll
                for (int s = 0; s < 6; ++s) {
                    const u32 lo = (u32)__builtin_amdgcn_readlane((int)hc, 2 * s);
                    const u32 hi = (u32)__builtin_amdgcn_readlane((int)hc, 2 * s + 1);
                    u32 ha, hb;
                    dec_h(cur[s].z, (lo & 0x7f) * 0x01010101u, ha, hb);
                    if (__builtin_expect(((lo >> 7) | hi) != 0, 0)) {
                        if (lo & 0x80) {
                            const char* pv = sp[s] + O_OUTOVF + OBLK(s, c0, cg0) * 1024 + lane * 16 + wv * 4;
                            ha = cur[s].z;
                            asm volatile("global_load_dword %0, %1, off sc0 sc1\n\ts_waitcnt vmcnt(0)" : "=&v"(hb) : "v"(pv) : "memory");
                        } else {
                            u64 pay = (((u64)hi << 32) | lo) >> 8;
                            while (pay != 0) {
                                const u32 e = (u32)pay & 0x3ffff, idx = (e >> 6) & 31;
                                pay >>= 18;
                                const u32 t = (int)(idx >> 3) == wv ? (e >> 11) << (8 * (idx & 3)) : 0;
                                const u32 ta = (idx & 4) ? 0 : t, tb = (idx & 4) ? t : 0;
                                const bool me = lane == (int)(e & 63);
                                ha ^= me ? ta : 0;
                                hb ^= me ? tb : 0;
                            }
                        }
                    }
                    float* const vv = v + (s / 3) * 8;
#pragma unroll
                    for (int j = 0; j < 4; ++j) {
                        const u32 sel = ((4u + j) << 24) | ((u32)j << 16) | 0x0c0cu;
                        const float a = __builtin_bit_cast(float, __builtin_amdgcn_perm(ha, cur[s].x, sel));
                        const float b = __builtin_bit_cast(float, __builtin_amdgcn_perm(hb, cur[s].y, sel));
                        vv[j] = s % 3 ? vv[j] + a : a;
                        vv[4 + j] = s % 3 ? vv[4 + j] + b : b;
                    }
                }
#pragma unroll
                for (int r = 0; r + 1 < KEEP; ++r) {
                    oi[r] = oi[r + 1]; oc[r] = oc[r + 1]; ocg[r] = ocg[r + 1];
#pragma unroll
                    for (int j = 0; j < 16; ++j) y[16 * r + j] = y[16 * r + 16 + j];
                }
                oi[KEEP - 1] = i0; oc[KEEP - 1] = c0; ocg[KEEP - 1] = cg0;
#pragma unroll
                for (int j = 0; j < 16; ++j) { const float t = fmaxf(v[j], 0.f); mx = fmaxf(mx, t); y[16 * (KEEP - 1) + j] = t; }
            }
            i0 = i1; c0 = c1; cg0 = cg1; i1 = i2; c1 = c2; cg1 = cg2;
        } while (i0 >= 0);
        const u32 wm = wave_umax(__builtin_bit_cast(u32, mx));
        if (lane == 0) red[wv] = ok ? wm : 0xffffffffu;
        __syncthreads();
        if (wv == 0) {
            const u32 m = umax(umax(red[0], red[1]), umax(red[2], red[3]));
            if (lane < 8 && m != 0xffffffffu) st_gran(amx, m, wt.seq);
        }
        STAMP(56);
        if (nkeep == KEEP && lane == 0) owed = tko;
        if (oi[KEEP - 2] < HELD) {
            WAIT_VM0();
            st_word_b(rsm, ok, fo, wt.seq);
            fo = OOB;
            hand_over();
            WAIT_VM0();
            st_word_b(rsm, ok, fo, wt.seq);
        }
    } else {
        rgA = (ri ^ (wv & 1)) * 8; cgA = (rj ^ (wv >> 1)) * 32; rgB = 0; cgB = 0;
        shell_at(hi >> 1, oc[0], ocg[0]);
        ocg[0] = 2 * ocg[0] + (hi & 1);
        if (ok) ok = poll_word(my + O_PROG + ((wv == 0 || wv == 3 ? 256 : 0) + hi) * 128 + (lane & 3) * 4, lane < 4, wt.seq, wt);
        const u32 so = O_Y + ((rgA + oc[0]) * 64 + cgA + ocg[0]) * 8192;
#pragma unroll
        for (int u = 0; u < 8; ++u) {
            const f32x4 a = __builtin_bit_cast(f32x4, __builtin_amdgcn_raw_buffer_load_b128(rsm, lane * 16, so + u * 1024, SYS));
            y[4 * u] = a.x; y[4 * u + 1] = a.y; y[4 * u + 2] = a.z; y[4 * u + 3] = a.w;
        }
        STAMP(17);
        if (ok) ok = poll_word(my + O_PROG + (HELD - 1) * 128 + wv * 4, lane == 0, wt.seq, wt);
    }

    {
        const int t = threadIdx.x;
        const bool two = t + 256 < NR * NCOMM;
        const char* a0 = my + O_AMAX + t * AS;
        u64 v0 = 0, v1 = 0;
        bool d0 = false, d1 = !two;
        while (ok) {
            if (!d0) v0 = ld_gran(a0);
            if (!d1) v1 = ld_gran(a0 + 256 * AS);
            d0 = (u32)(v0 >> 32) >= wt.seq;
            d1 = !two || (u32)(v1 >> 32) >= wt.seq;
            if (__all(d0 && d1)) break;
            if (expired(wt)) ok = false;
            __builtin_amdgcn_s_sleep(1);
        }
        const u32 wm = wave_umax(umax((u32)v0, (u32)v1));
        if (lane == 0) red[8 + wv] = ok ? wm : 0xffffffffu;
        __syncthreads();
        const u32 m = umax(umax(red[8], red[9]), umax(red[10], red[11]));
        ok = m != 0xffffffffu;
        const float am = __builtin_bit_cast(float, m);
        const float inv = am > 0.f ? 1.0f / (am / 448.0f) : 0.f;
        STAMP(57);
        const int fr = lane & 15, fq = lane >> 4;
        if (puller) {
            __builtin_amdgcn_raw_ptr_buffer_atomic_add_i32(1, rsm, owed, 0, 16);
#pragma unroll
            for (int r = 0; r < 2 * KEEP; ++r) {
                if (oi[r >> 1] < HELD) continue;
                u32x2 o = {q4(y[8 * r], y[8 * r + 1], y[8 * r + 2], y[8 * r + 3], inv),
                           q4(y[8 * r + 4], y[8 * r + 5], y[8 * r + 6], y[8 * r + 7], inv)};
                if (!ok) o = (u32x2){0x7f7f7f7fu, 0x7f7f7f7fu};
                const int rg = ((r & 1) ? rgB : rgA) + oc[r >> 1], cg = ((r & 1) ? cgB : cgA) + ocg[r >> 1];
                *(u32x2*)(p.out + (rg * 32 + (wv >> 1) * 16 + fr) * 4096 + cg * 64 + fq * 16 + (wv & 1) * 8) = o;
            }
        } else {
#pragma unroll
            for (int rf = 0; rf < 2; ++rf) {
                u32x4 o;
#pragma unroll
                for (int j = 0; j < 4; ++j)
                    o[j] = q4(y[16 * rf + 4 * j], y[16 * rf + 4 * j + 1], y[16 * rf + 4 * j + 2], y[16 * rf + 4 * j + 3], inv);
                if (!ok) o = (u32x4){0x7f7f7f7fu, 0x7f7f7f7fu, 0x7f7f7f7fu, 0x7f7f7f7fu};
                *(u32x4*)(p.out + ((rgA + oc[0]) * 32 + rf * 16 + fr) * 4096 + (cgA + ocg[0]) * 64 + fq * 16) = o;
            }
        }
        STAMP(58);
        STAMPC(59);
        STAMP_DUMP();
    }
}

extern "C" void kernel_launch(int rank, int nranks, ncclComm_t comm, hipStream_t stream,
                              void* const* d_in, const int* in_sizes, int n_in,
                              void* d_out, int out_size,
                              const int* in_dtypes, int out_dtype,
                              void* const* d_out_all,
                              void* const* d_ws_all, size_t ws_bytes) {
    (void)comm; (void)in_dtypes; (void)out_dtype; (void)d_out_all;
    if (nranks != NR || n_in != 2 || (unsigned)rank >= (unsigned)NR || ws_bytes < WS_USED ||
        in_sizes[0] != 4096 * 512 || in_sizes[1] != 512 * 4096 || out_size != 512 * 4096) {
        fprintf(stderr, "kernel_launch: unexpected problem shape\n");
        return;
    }
    Params p;
    p.x = (const float*)d_in[0];
    p.w = (const float*)d_in[1];
    p.out = (unsigned char*)d_out;
    p.my = (char*)d_ws_all[rank];
    for (int r = 0; r < NR; ++r) p.rel[r] = (char*)d_ws_all[r ^ rank];
    p.rank = rank;
    p.pad = 0;
    hipLaunchKernelGGL(fused, dim3(GRID), dim3(256), 0, stream, p);
    HIP_CHECK(hipGetLastError());
}
```

```cpp
#include <hip/hip_runtime.h>
#include <rccl/rccl.h>
#include <cstdio>

#define HIP_CHECK(expr) do { hipError_t _e = (expr); if (_e != hipSuccess) { \
    fprintf(stderr, "HIP error %s at %s:%d\n", hipGetErrorString(_e), __FILE__, __LINE__); \
    return; } } while (0)

typedef unsigned u32;
typedef unsigned long long u64;
typedef u32 u32x2 __attribute__((ext_vector_type(2)));
typedef u32 u32x3 __attribute__((ext_vector_type(3)));
typedef u32 u32x4 __attribute__((ext_vector_type(4)));
typedef u32 u32x8 __attribute__((ext_vector_type(8)));
typedef u32 u32x16 __attribute__((ext_vector_type(16)));
typedef float f32x2 __attribute__((ext_vector_type(2)));
typedef float f32x4 __attribute__((ext_vector_type(4)));
typedef __bf16 bf16x2 __attribute__((ext_vector_type(2)));
typedef __bf16 bf16x8 __attribute__((ext_vector_type(8)));
typedef __attribute__((address_space(1))) u32 gu32;
typedef __attribute__((address_space(1))) u64 gu64;
typedef __amdgpu_buffer_rsrc_t rsrc_t;

#define HEAD 30
#define STAG 3
#define LAG 3
#define TRIG 64
#define NPI 26
#define STAG0 0
#define STAGP 10
#define AS 128u
#define AGG 1
#define HW 32
#define SRC_AUX 2
#define OUT_AUX 0
#define BARP 1
#define TEST_BASE_SHIFT 0

constexpr int NR = 8;
#define NPO 24
#define NKEPT 52
constexpr int NP = NPO;
constexpr int NCOMM = 2 * NP;
constexpr int GRID = 512;
constexpr int NG1 = 256 - NCOMM;
constexpr int NG2 = 256 - NG1;
constexpr int B_IN = 256 + NG1;
constexpr int NIN = (6 * NPI + 3) / 4;
constexpr int KEEP = 4;
constexpr int HELD = 256 - NKEPT;
static_assert(HELD <= NG1 && NG1 % 8 == 0 && NG2 + NIN + 96 <= NG1 && B_IN + NIN <= GRID && NKEPT <= NP * KEEP && 2 * NP <= HELD, "roles");
constexpr u64 DEADLINE = 30000000ull;

constexpr u32 FS = 128;
constexpr u32 BLK = 3072;
constexpr u32 SL = 50 * 8;
constexpr u32 N_IN = 16 * SL, N_PK = 2048, N_OUT = 3072;
constexpr u32 O_SEQ = 0;
constexpr u32 O_TMO = O_SEQ + GRID * 4 * 4;
constexpr u32 O_TICK = O_TMO + 128;
constexpr u32 O_AMAX = O_TMO + 2048;
constexpr u32 O_PROG = O_AMAX + 512 * 128;
constexpr u32 O_PKHDR = O_PROG + 512 * 128;
constexpr u32 O_OUTHDR = O_PKHDR + 12 * 64 * 128;
constexpr u32 O_INFLAG = O_OUTHDR + 512 * 128;
constexpr u32 O_IN = ((O_INFLAG + N_IN * FS + 4095) / 4096) * 4096;
constexpr u32 O_PK = O_IN + N_IN * 4096;
constexpr u32 O_OUTDATA = O_PK + N_PK * BLK;
constexpr u32 O_Y = O_OUTDATA + N_OUT * BLK;
constexpr u32 O_PKOVF = O_Y + 1024 * 8192;
constexpr u32 O_OUTOVF = O_PKOVF + N_PK * 1024;
constexpr u32 O_DBG = O_OUTOVF + N_OUT * 1024;
constexpr u32 WS_USED = O_DBG + GRID * 4 * 64 * 8;
constexpr u32 OOB = 0xfffff000u;
static_assert(O_AMAX % 128 == 0 && O_PROG % 128 == 0 && O_INFLAG % 128 == 0 && O_PKHDR % 128 == 0 && O_OUTHDR % 128 == 0, "align");
static_assert(NR * NCOMM <= 384 && GRID * 4 == 2048, "shape");

constexpr int SYS = 17;

struct Params {
    const float* x;
    const float* w;
    unsigned char* out;
    char* my;
    int rank;
    int pad;
    char* rel[NR];
};

#define STAMP(id) do { } while (0)
#define STAMPC(id) do { } while (0)
#define STAMP_INIT() const u32 stb = 0;
#define STAMP_DUMP() do { } while (0)
#define POLL_SLEEP 4

#define DEV __device__ __forceinline__
#define CFENCE() asm volatile("" ::: "memory")
#define WAIT_VM0() do { __builtin_amdgcn_s_waitcnt(0x0F70); asm volatile("s_waitcnt vmcnt(0)" ::: "memory"); } while (0)

DEV rsrc_t mk_rsrc(const void* base, u32 bytes) { return __builtin_amdgcn_make_buffer_rsrc((void*)base, 0, bytes, 0x00020000); }
DEV u32 rfl(u32 v) { return __builtin_amdgcn_readfirstlane(v); }
DEV u32 cvt2(float a, float b) { f32x2 v = {a, b}; return __builtin_bit_cast(u32, __builtin_convertvector(v, bf16x2)); }
DEV u32 umax(u32 a, u32 b) { return a > b ? a : b; }
DEV u32 wave_umax(u32 v) {
    v = umax(v, (u32)__builtin_amdgcn_update_dpp(0, (int)v, 0x111, 0xf, 0xf, true));
    v = umax(v, (u32)__builtin_amdgcn_update_dpp(0, (int)v, 0x112, 0xf, 0xf, true));
    v = umax(v, (u32)__builtin_amdgcn_update_dpp(0, (int)v, 0x114, 0xf, 0xf, true));
    v = umax(v, (u32)__builtin_amdgcn_update_dpp(0, (int)v, 0x118, 0xf, 0xf, true));
    v = umax(v, (u32)__builtin_amdgcn_update_dpp(0, (int)v, 0x142, 0xa, 0xf, false));
    v = umax(v, (u32)__builtin_amdgcn_update_dpp(0, (int)v, 0x143, 0xc, 0xf, false));
    return (u32)__builtin_amdgcn_readlane((int)v, 63);
}

struct Wait {
    u32 seq;
    u64 t0;
    gu32* tmo;
};
DEV bool expired(const Wait& wt) {
    const u64 dt = __builtin_amdgcn_s_memrealtime() - wt.t0;
    if (dt < 100000ull) return false;
    if (dt > DEADLINE || rfl(__hip_atomic_load(wt.tmo, __ATOMIC_RELAXED, __HIP_MEMORY_SCOPE_AGENT)) == wt.seq) {
        __hip_atomic_store(wt.tmo, wt.seq, __ATOMIC_RELAXED, __HIP_MEMORY_SCOPE_AGENT);
        return true;
    }
    return false;
}
DEV u64 ld_gran(const char* p) { return __hip_atomic_load((gu64*)p, __ATOMIC_RELAXED, __HIP_MEMORY_SCOPE_SYSTEM); }
DEV void st_gran(char* p, u32 payload, u32 seq) {
    __hip_atomic_store((gu64*)p, ((u64)seq << 32) | payload, __ATOMIC_RELAXED, __HIP_MEMORY_SCOPE_SYSTEM);
}
DEV bool poll_gran(const char* p, bool act, u64& v, const Wait& wt) {
    bool ok = true;
    bool done = !act || (u32)(v >> 32) >= wt.seq;
    if (__builtin_expect(!__all(done), 0)) {
        for (;;) {
            if (!done) { v = ld_gran(p); done = (u32)(v >> 32) >= wt.seq; }
            if (__all(done)) break;
            if (expired(wt)) { ok = false; break; }
            __builtin_amdgcn_s_sleep(POLL_SLEEP);
        }
    }
    CFENCE();
    return ok;
}
DEV bool poll_word(const char* p, bool act, u32 need, const Wait& wt) {
    bool ok = true, done = !act;
    for (;;) {
        if (!done) done = __hip_atomic_load((gu32*)p, __ATOMIC_RELAXED, __HIP_MEMORY_SCOPE_SYSTEM) >= need;
        if (__all(done)) break;
        if (expired(wt)) { ok = false; break; }
        __builtin_amdgcn_s_sleep(POLL_SLEEP);
    }
    CFENCE();
    return ok;
}

DEV bool poll_part(const char* p, bool& done, u64 want, const Wait& wt) {
    bool ok = true;
    if ((__ballot(done) & want) != want) {
        for (;;) {
            if (!done) done = __hip_atomic_load((gu32*)p, __ATOMIC_RELAXED, __HIP_MEMORY_SCOPE_SYSTEM) >= wt.seq;
            if ((__ballot(done) & want) == want) break;
            if (expired(wt)) { ok = false; break; }
            __builtin_amdgcn_s_sleep(POLL_SLEEP);
        }
    }
    CFENCE();
    return ok;
}

struct Packed {
    u32 L[8], N[4], X[4];
    u32 lo, hi;
};
DEV void pack(const u32 (&D)[16], float amax, Packed& o, int kind) {
    const u32 top = wave_umax(__builtin_bit_cast(u32, amax)) >> 24;
    int b = (int)(top & 0x7f) - 7 - ((TEST_BASE_SHIFT & kind) ? 1 : 0);
    b = b < 0 ? 0 : b;
    const u32 base4 = (u32)b * 0x01010101u;
    u32 H[8], d[8], chk = 0;
#pragma unroll
    for (int q = 0; q < 8; ++q) {
        o.L[q] = __builtin_amdgcn_perm(D[2 * q + 1], D[2 * q], 0x06040200u);
        H[q] = __builtin_amdgcn_perm(D[2 * q + 1], D[2 * q], 0x07050301u);
        d[q] = H[q] - base4;
        chk |= d[q];
    }
    u64 pay = (u64)b;
    u64 m = __ballot((chk & 0x78787878u) != 0);
    if (__builtin_expect(m != 0, 0)) {
        u32x8 dl;
        u32 esc = 0;
#pragma unroll
        for (int q = 0; q < 8; ++q) {
            d[q] = (((H[q] | 0x80808080u) - base4) & 0x07070707u) | (H[q] & 0x80808080u);
            dl[q] = (d[q] + base4) ^ H[q];
            esc |= ((dl[q] + 0x7f7f7f7fu) & 0x80808080u) >> (7 - q);
        }
        int cnt = 0;
        while (m != 0 && cnt <= 3) {
            const int l = __builtin_ctzll(m);
            m &= m - 1;
            u32 bits = (u32)__builtin_amdgcn_readlane((int)esc, l);
            while (bits != 0 && cnt <= 3) {
                const int pos = __builtin_ctz(bits);
                bits &= bits - 1;
                const u32 e = ((u32)__builtin_amdgcn_readlane((int)dl[pos & 7], l) >> (pos & 24)) & 0x7f;
                pay |= (u64)((u32)l | ((u32)(4 * (pos & 7) + (pos >> 3)) << 6) | (e << 11)) << (8 + 18 * cnt);
                ++cnt;
            }
        }
        if (cnt > 3) {
            pay = 0x80;
#pragma unroll
            for (int t = 0; t < 4; ++t) { d[2 * t] = H[2 * t]; d[2 * t + 1] = 0; }
        }
    }
#pragma unroll
    for (int t = 0; t < 4; ++t) {
        o.N[t] = d[2 * t] | __builtin_amdgcn_alignbit(d[2 * t + 1], d[2 * t + 1], 28);
        o.X[t] = H[2 * t + 1];
    }
    o.lo = (u32)pay;
    o.hi = (u32)(pay >> 32);
}
DEV void dec_h(u32 n, u32 base4, u32& ha, u32& hb) {
    ha = (n & 0x87878787u) + base4;
    hb = (__builtin_amdgcn_alignbit(n, n, 4) & 0x87878787u) + base4;
}
DEV u32x4 mk_frag(u32 la, u32 lb, u32 ha, u32 hb) {
    u32x4 r;
    r.x = __builtin_amdgcn_perm(ha, la, 0x05010400u);
    r.y = __builtin_amdgcn_perm(ha, la, 0x07030602u);
    r.z = __builtin_amdgcn_perm(hb, lb, 0x05010400u);
    r.w = __builtin_amdgcn_perm(hb, lb, 0x07030602u);
    return r;
}
struct Pk { u32x4 l0, l1, n; };
DEV u32x16 cat4(u32x4 a, u32x4 b, u32x4 c, u32x4 d) {
    const u32x8 l = __builtin_shufflevector(a, b, 0, 1, 2, 3, 4, 5, 6, 7), h = __builtin_shufflevector(c, d, 0, 1, 2, 3, 4, 5, 6, 7);
    return __builtin_shufflevector(l, h, 0, 1, 2, 3, 4, 5, 6, 7, 8, 9, 10, 11, 12, 13, 14, 15);
}
#define FRAG(v, t) __builtin_shufflevector(v, v, 4 * (t), 4 * (t) + 1, 4 * (t) + 2, 4 * (t) + 3)
DEV u32x16 unpack(const Pk& k, u32 lo, u32 hi, const char* ovf, int lane) {
    const u32 base4 = (lo & 0x7f) * 0x01010101u;
    u32x4 ha, hb;
#pragma unroll
    for (int t = 0; t < 4; ++t) { u32 a, b; dec_h(k.n[t], base4, a, b); ha[t] = a; hb[t] = b; }
    if (__builtin_expect(((lo >> 7) | hi) != 0, 0)) {
        if (lo & 0x80) {
            ha = k.n;
            asm volatile("global_load_dwordx4 %0, %1, off sc0 sc1\n\ts_waitcnt vmcnt(0)" : "=&v"(hb) : "v"(ovf) : "memory");
        } else {
            u64 pay = (((u64)hi << 32) | lo) >> 8;
            while (pay != 0) {
                const u32 e = (u32)pay & 0x3ffff, idx = (e >> 6) & 31;
                pay >>= 18;
                const u32 t = lane == (int)(e & 63) ? (e >> 11) << (8 * (idx & 3)) : 0;
#pragma unroll
                for (u32 z = 0; z < 4; ++z) {
                    ha[z] ^= (idx >> 2) == 2 * z ? t : 0;
                    hb[z] ^= (idx >> 2) == 2 * z + 1 ? t : 0;
                }
            }
        }
    }
    return cat4(mk_frag(k.l0.x, k.l0.y, ha.x, hb.x), mk_frag(k.l0.z, k.l0.w, ha.y, hb.y),
                mk_frag(k.l1.x, k.l1.y, ha.z, hb.z), mk_frag(k.l1.z, k.l1.w, ha.w, hb.w));
}
DEV void st128(u32x4 v, rsrc_t rs, u32 off) { __builtin_amdgcn_raw_buffer_store_b128(v, rs, off, 0, SYS); }
DEV void st96(u32x3 v, rsrc_t rs, u32 off) { __builtin_amdgcn_raw_buffer_store_b96(v, rs, off, 0, SYS); }
DEV void st_gran_b(rsrc_t rs, bool act, u32 off, u32 payload, u32 seq) {
    const u32x2 v = {payload, seq};
    __builtin_amdgcn_raw_buffer_store_b64(v, rs, act ? off : OOB, 0, SYS);
}

DEV void st_word_b(rsrc_t rs, bool act, u32 off, u32 v) { __builtin_amdgcn_raw_buffer_store_b32(v, rs, act ? off : OOB, 0, SYS); }

DEV int shell_of(int e) {
    const int k = e - LAG;
    return e < LAG ? e : e >= 16 - LAG ? e : (k & 1) ? LAG + (k >> 1) : 8 + (k >> 1);
}
DEV int sto(int ch) { return ch < 16 ? (ch & 3) * 4 + (ch >> 2) : 16 + (ch & 1) * 8 + ((ch - 16) >> 1); }
DEV int item_at(int i, int src) {
    const int e = i / 30, ps = shell_of(e);
    int r = i - e * 30;
    if ((r >> 1) >= src) r += 2;
    return (ps >> 3) * 256 + sto((r >> 4) * 16 + (ps & 7) * 2 + ((r >> 3) & 1)) * 8 + (r & 7);
}
DEV void ld_src(const float* px, const float* pw, int lane, int cls, int ks, int chunk, int blk, f32x4 (&F)[8]) {
    const int fr = lane & 15, fq = lane >> 4;
    const bool isx = chunk < 16;
    const rsrc_t rs = mk_rsrc(isx ? px : pw, 4096u * 512u * 4u);
    u32 so, s2, s1, s0, vo;
    if (isx) {
        const int owner = (chunk & 1) * 4 + (blk >> 1);
        so = ((owner * 512 + (cls * 8 + (chunk >> 1)) * 32) * 512 + ks * 128 + (blk & 1) * 64) * 4;
        s2 = 16 * 2048; s1 = 128; s0 = 64;
        vo = fr * 2048 + fq * 16;
    } else {
        so = ((ks * 128 + (blk >> 1) * 32) * 4096 + cls * 2048 + (chunk - 16) * 128 + (blk & 1) * 64) * 4;
        s2 = 16 * 16384; s1 = 2 * 16384; s0 = 16384;
        vo = fq * 4 * 16384 + fr * 16;
    }
#pragma unroll
    for (int u = 0; u < 8; ++u)
        F[u] = __builtin_bit_cast(f32x4, __builtin_amdgcn_raw_buffer_load_b128(
                   rs, vo, so + (u >> 2) * s2 + ((u >> 1) & 1) * s1 + (u & 1) * s0, SRC_AUX));
}
DEV float cvt_src(const f32x4 (&F)[8], bool isx, u32 (&D)[16]) {
    if (isx) {
#pragma unroll
        for (int u = 0; u < 8; ++u) { D[2 * u] = cvt2(F[u].x, F[u].y); D[2 * u + 1] = cvt2(F[u].z, F[u].w); }
    } else {
#pragma unroll
        for (int f = 0; f < 4; ++f)
#pragma unroll
            for (int e = 0; e < 4; ++e) D[4 * f + e] = cvt2(F[2 * e][f], F[2 * e + 1][f]);
    }
    float m = 0.f;
#pragma unroll
    for (int u = 0; u < 8; ++u) {
        m = fmaxf(fmaxf(m, fabsf(F[u].x)), fabsf(F[u].y));
        m = fmaxf(fmaxf(m, fabsf(F[u].z)), fabsf(F[u].w));
    }
    return m;
}
DEV void st_plain(const u32 (&D)[16], rsrc_t rs, u32 off) {
#pragma unroll
    for (int t = 0; t < 4; ++t) st128((u32x4){D[4 * t], D[4 * t + 1], D[4 * t + 2], D[4 * t + 3]}, rs, off + t * 1024);
}
DEV void shell_at(int t, int& a, int& b) {
    int s = 0;
    while (2 * (s + 1) * (s + 1) <= t) ++s;
    int r = t - 2 * s * s;
    if (r < 2 * s) { a = s; b = r; }
    else { r -= 2 * s; const int up = r > s; a = r - up * (s + 1); b = 2 * s + up; }
}
DEV u32x4 relu4(f32x4 v) {
    typedef int i32x4 __attribute__((ext_vector_type(4)));
    return __builtin_bit_cast(u32x4, __builtin_elementwise_max(__builtin_bit_cast(i32x4, v), (i32x4){0, 0, 0, 0}));
}
DEV u32 q4(float a, float b, float c, float d, float inv) {
    int r = __builtin_amdgcn_cvt_pk_fp8_f32(a * inv, b * inv, 0, false);
    r = __builtin_amdgcn_cvt_pk_fp8_f32(c * inv, d * inv, r, true);
    return (u32)r;
}

#define XSLOT(s) (xs0 + ((s) >> 1) * SL + ((s) & 1))
#define WSLOT(s) (ws0 + ((s) >> 1) * SL + ((s) & 1) * 4)
#define SB() __builtin_amdgcn_sched_barrier(0)
template <int F>
DEV void ld_x(u32x4& d, u32x4 sd, u32 vo, u32 so) {
    asm volatile("buffer_load_dwordx4 %0, %1, %2, %3 offen offset:%4 sc0 sc1" : "+v"(d) : "v"(vo), "s"(sd), "s"(so), "i"(F * 1024));
}
template <int N>
DEV void wait_step(u32x4 (&x)[4]) {
    __builtin_amdgcn_s_waitcnt(0xC07F);
    asm volatile("s_waitcnt vmcnt(%4) lgkmcnt(0)\n\ts_barrier" : "+v"(x[0]), "+v"(x[1]), "+v"(x[2]), "+v"(x[3]) : "i"(N) : "memory");
}
DEV void mma(f32x4& c, u32x4 a, u32x4 b) { asm volatile("v_mfma_f32_16x16x32_bf16 %0, %1, %2, %0" : "+v"(c) : "v"(a), "v"(b)); }
template <int F>
DEV void mma_ldw(f32x4& c, u32x4 a, u32x4 b, u32 lds, u32x4 sd, u32 vo, u32 so) {
    asm volatile("s_mov_b32 m0, %3\n\tv_mfma_f32_16x16x32_bf16 %0, %1, %2, %0\n\tbuffer_load_dwordx4 %4, %5, %6 offen offset:%7 sc0 sc1 lds"
                 : "+v"(c) : "v"(a), "v"(b), "s"(lds), "v"(vo), "s"(sd), "s"(so), "i"(F * 1024) : "memory");
}
template <int S, int G>
DEV bool gemm_steps(f32x4 (&acc)[2][8], u32x4 (&X)[4][4], u32x4 (&a)[8], const u32x4* ring, u32 lds0, u32x4 sd, u32 xs0, u32 ws0,
                    int skip, int lane, u32 fo, u32& fv, u32 seq, u32* vote) {
    const u32 vo = lane * 16;
    u32x4 b[8];
    const int sl = (S + 3) >> 1;
    const u32 vx = (sl == skip || sl == skip + 6) ? 0x80000000u : vo;
    const u32 sx = O_IN + XSLOT(S + 3) * 4096, sw = O_IN + WSLOT(S + 4) * 4096, l = lds0 + S * 16384;
    constexpr bool LX = G < 2 || S == 0, LW = G < 2, END = G == 2 && S == 3;
    u32 vt = 0;
    if (G < 2 && S == 0) asm volatile("buffer_load_dword %0, %1, %2, 0 offen sc0 sc1" : "+v"(fv) : "v"(fo), "s"(sd));
#pragma unroll
    for (int i = 0; i < 8; ++i) {
        mma(acc[0][i], a[i], X[S][0]);
        SB();
        if (i < 6) b[i < 4 ? i + (i + 1) / 2 : i + 2] = ring[S * 1024 + (8 + (i < 4 ? i + (i + 1) / 2 : i + 2)) * 64 + lane];
        if (G < 2 && S == 3 && i == 6) vote[(fo != OOB && fv < seq) ? 0 : 8] = 1;
        SB();
        mma(acc[1][i], a[i], X[S][2]);
        SB();
        if (i == 0 || i == 2) b[i + i / 2 + 1] = ring[S * 1024 + (8 + i + i / 2 + 1) * 64 + lane];
        if (LX && i == 1) ld_x<0>(X[(S + 3) & 3][0], sd, vx, sx);
        if (LX && i == 3) ld_x<1>(X[(S + 3) & 3][1], sd, vx, sx);
        if (LX && i == 4) ld_x<2>(X[(S + 3) & 3][2], sd, vx, sx);
        if (LX && i == 6) ld_x<3>(X[(S + 3) & 3][3], sd, vx, sx);
        SB();
    }
    if (!END) wait_step<(G == 0 && S == 0 ? 8 : G == 0 && S == 1 ? 12 : G < 2 || S == 0 ? 16 : S == 1 ? 8 : 0)>(X[(S + 1) & 3]);
    if (G < 2 && S == 2) asm volatile("" : "+v"(fv));
    if (G < 2 && S == 3) vt = *vote;
    SB();
#pragma unroll
    for (int i = 0; i < 8; ++i) {
        mma(acc[0][i], b[i], X[S][1]);
        SB();
        if (!END) a[i] = ring[((S + 1) & 3) * 1024 + i * 64 + lane];
        SB();
        if (LW && i == 0) mma_ldw<0>(acc[1][i], b[i], X[S][3], l, sd, vo, sw);
        else if (LW && i == 2) mma_ldw<1>(acc[1][i], b[i], X[S][3], l, sd, vo, sw);
        else if (LW && i == 4) mma_ldw<2>(acc[1][i], b[i], X[S][3], l, sd, vo, sw);
        else if (LW && i == 6) mma_ldw<3>(acc[1][i], b[i], X[S][3], l, sd, vo, sw);
        else mma(acc[1][i], b[i], X[S][3]);
        SB();
    }
    if constexpr (S < 3) return gemm_steps<S + 1, G>(acc, X, a, ring, lds0, sd, xs0, ws0, skip, lane, fo, fv, seq, vote);
    else return rfl(vt) == 0;
}
DEV void first_w(u32 lds0, u32x4 sd, u32 ws0, int lane) {
#pragma unroll
    for (int s = 0; s < 4; ++s) {
        const u32 sw = O_IN + WSLOT(s) * 4096, l = lds0 + s * 16384, vo = lane * 16;
        asm volatile("s_mov_b32 m0, %0\n\ts_nop 0\n\tbuffer_load_dwordx4 %1, %2, %3 offen sc0 sc1 lds\n\t"
                     "buffer_load_dwordx4 %1, %2, %3 offen offset:1024 sc0 sc1 lds\n\tbuffer_load_dwordx4 %1, %2, %3 offen offset:2048 sc0 sc1 lds\n\t"
                     "buffer_load_dwordx4 %1, %2, %3 offen offset:3072 sc0 sc1 lds" :: "s"(l), "v"(vo), "s"(sd), "s"(sw) : "memory");
    }
}
DEV bool gemm_tile(f32x4 (&acc)[2][8], const u32x4* ring, u32 lds0, u32x4 sd, u32 xs0, u32 ws0, int gend, int skip, bool wfirst,
                   const char* my, const Wait& wt, u32* fail, u32* votes, int lane, u32 stb) {
    u32x4 X[4][4], a[8];
#pragma unroll
    for (int i = 0; i < 16; ++i) X[i >> 2][i & 3] = (u32x4){0, 0, 0, 0};
    const u32 f0 = O_INFLAG + ((lane & 16) ? WSLOT(lane & 15) : XSLOT(lane & 15)) * FS;
    const bool none = lane >= 32 || (lane & 15) >= 12 || (lane >> 1) == skip - 4;
    bool ok = true;
    u32 fv = 0;
    int g = 0;
    do {
        bool done = g ? none : lane >= 32 || (lane & 15) >= 8;
        const char* const fp = my + f0 + g * (2 * SL * FS);
        asm volatile("s_waitcnt lgkmcnt(0)\n\ts_barrier" ::: "memory");
        if (wfirst) {
            if (ok) ok = poll_part(fp, done, 0xffff0000ull, wt);
            first_w(lds0, sd, ws0, lane);
        }
        if (ok) ok = poll_part(fp, done, 0xffffull, wt);
        STAMP(20 + g);
#pragma unroll
        for (int s = 0; s < 3; ++s) {
            const u32 sx = O_IN + XSLOT(s) * 4096, vx = ((s >> 1) == skip || (s >> 1) == skip + 6) ? 0x80000000u : lane * 16;
            ld_x<0>(X[s][0], sd, vx, sx); ld_x<1>(X[s][1], sd, vx, sx); ld_x<2>(X[s][2], sd, vx, sx); ld_x<3>(X[s][3], sd, vx, sx);
        }
        if (!wfirst) {
            if (ok) ok = poll_part(fp, done, 0xffff0000ull, wt);
            STAMP(20 + g);
            first_w(lds0, sd, ws0, lane);
        } else {
            asm volatile("s_waitcnt vmcnt(8)" ::: "memory");
        }
        if (!ok && lane == 0) *fail = 1;
        wait_step<12>(X[0]);
        SB();
#pragma unroll
        for (int i = 0; i < 8; ++i) a[i] = ring[i * 64 + lane];
        SB();
#define NEXT_FO ((g == 0 || g == 3) && !none ? f0 + (g + 2) * (2 * SL * FS) : OOB)
#define ADVANCE() do { STAMP(30 + g); ++g; xs0 += 2 * SL; ws0 += 2 * SL; skip -= 2; } while (0)
        bool go = gemm_steps<0, 0>(acc, X, a, ring, lds0, sd, xs0, ws0, skip, lane, NEXT_FO, fv, wt.seq, votes + g);
        ADVANCE();
        while (g + 1 < gend && go) {
            go = gemm_steps<0, 1>(acc, X, a, ring, lds0, sd, xs0, ws0, skip, lane, NEXT_FO, fv, wt.seq, votes + g);
            ADVANCE();
        }
        gemm_steps<0, 2>(acc, X, a, ring, lds0, sd, xs0, ws0, skip, lane, OOB, fv, wt.seq, votes);
        ADVANCE();
    } while (g < gend);
    return ok;
}

__global__ void __launch_bounds__(256, 2) fused(const Params p) {
    __shared__ u32x4 ring[4096];
    __shared__ __attribute__((aligned(16))) u32 red[48];
    const int lane = threadIdx.x & 63;
    const int wv = (int)rfl(threadIdx.x >> 6);
    const int rank = p.rank & 7, rm = rank & 1, ri = (rank >> 1) & 1, rj = rank >> 2;
    const int bid = blockIdx.x;
    char* const my = p.my;
    const float* const px = p.x;
    const float* const pw_ = p.w;
    asm volatile("" :: "s"(my), "s"(px), "s"(pw_), "s"(rank));
    const rsrc_t rsm = mk_rsrc(my, WS_USED);

    Wait wt;
    wt.t0 = __builtin_amdgcn_s_memrealtime();
    wt.tmo = (gu32*)(my + O_TMO);
    gu32* const seqp = (gu32*)(my + O_SEQ + (bid * 4 + wv) * 4);
    const u32 seqv = __hip_atomic_load(seqp, __ATOMIC_RELAXED, __HIP_MEMORY_SCOPE_AGENT);
    STAMP_INIT();
    STAMP(0);
    STAMPC(60);
    if (threadIdx.x < 32) red[threadIdx.x] = 0;

    const bool inpull = bid >= B_IN && bid < B_IN + NIN;
    wt.seq = 0;
    u32* hx = red + 32;
#pragma nounroll
    for (int wg = inpull ? GRID : bid < HW ? bid : bid < NG1 ? bid + NCOMM : bid < 256 ? bid - NG1 + HW : bid < B_IN ? bid : bid - NIN; wg < GRID;
         wg = (wg >= 256 + NG2 && wg < 256 + NG2 + NIN) ? wg + GRID - NIN - 256 - NG2 : GRID) {
        const int u = wg * 4 + wv;
        const int pos = u >> 5, dm = (u >> 4) & 1, cls = (u >> 3) & 1, blk = u & 7;
        const int ps = shell_of(pos >> 2), ch = (pos & 2) * 8 + (ps & 7) * 2 + (pos & 1), pass = ps >> 3, ks = 2 * dm + pass;
        for (int z = (wg < GRID - NIN && pos >= STAGP) ? pos * STAG + STAG0 : 0; z > 0; --z) __builtin_amdgcn_s_sleep(1);
        f32x4 F[8];
        u32 D[16];
        ld_src(px, pw_, lane, cls, ks, ch, blk, F);
        STAMP(62);
        const bool isx = ch < 16, own = isx && (ch & 1) * 4 + (blk >> 1) == rank, same = cls == (isx ? ri : rj);
        const int dc = same ? 0 : (isx ? 2 : 4), d0 = (dc ^ (isx ? 4 : 2)) | (dm ^ rm), d1 = dc | (dm ^ rm);
        const rsrc_t rs0 = mk_rsrc(p.rel[d0], WS_USED), rs1 = mk_rsrc(p.rel[d1], same ? 0u : WS_USED);
        const u32 cb8 = sto(ch) * 8 + blk;
        const u32 r0 = pass * 6 + d0 - 2, r1 = same ? 0 : pass * 6 + d1 - 2;
        const u32 ls = ks * SL + (same ? cb8 : isx ? 256 + (ch >> 1) * 2 + (blk & 1) : 144 + cb8);
        const int r32 = (pos & 3) * 8 + blk;
        const int item = (pos >> 2) * 30 + r32 - ((r32 >> 1) > rank ? 2 : 0);
        const bool local = same || own || !isx, head = !own && item < HEAD;
        const float m = cvt_src(F, isx, D);
        STAMP(63);
        u32 hpay = 0;
        if (head) {
            st_plain(D, rs0, O_IN + ((4 + r0) * SL + cb8) * 4096 + lane * 16);
            st_plain(D, rs1, O_IN + ((4 + r1) * SL + cb8) * 4096 + lane * 16);
        } else if (!own) {
            Packed k;
            pack(D, m, k, isx ? 1 : 2);
            const u32 pk = (ks * 2 + cls) * 256 + cb8, o = O_PK + pk * BLK + lane * 16;
            if (__builtin_expect(k.lo & 0x80, 0)) st128((u32x4){k.X[0], k.X[1], k.X[2], k.X[3]}, rsm, O_PKOVF + pk * 1024 + lane * 16);
            st128((u32x4){k.L[0], k.L[1], k.L[2], k.L[3]}, rsm, o);
            st128((u32x4){k.L[4], k.L[5], k.L[6], k.L[7]}, rsm, o + 1024);
            st128((u32x4){k.N[0], k.N[1], k.N[2], k.N[3]}, rsm, o + 2048);
            hpay = lane ? k.hi : k.lo;
        }
        if (local) st_plain(D, rsm, O_IN + ls * 4096 + lane * 16);
        wt.seq = rfl(seqv) + 1;
        STAMP(61);
        WAIT_VM0();
        if (head) {
            st_word_b(rs0, lane == 0, O_INFLAG + ((4 + r0) * SL + cb8) * FS, wt.seq);
            st_word_b(rs1, lane == 0, O_INFLAG + ((4 + r1) * SL + cb8) * FS, wt.seq);
        }
        st_word_b(rsm, local && lane == 0, O_INFLAG + ls * FS, wt.seq);
        if (AGG) {
            if (lane < 2) hx[wv * 2 + lane] = hpay;
            __syncthreads();
            if (wv == 3 && item >= HEAD) {
                const u32 v = hx[lane & 7];
                st_gran_b(rs0, lane < 8, O_PKHDR + (r0 * 64 + (cb8 >> 2)) * 128 + lane * 8, v, wt.seq);
                st_gran_b(rs1, lane < 8, O_PKHDR + (r1 * 64 + (cb8 >> 2)) * 128 + lane * 8, v, wt.seq);
            }
            hx += 8;
        } else if (!head && !own) {
            st_gran_b(rs0, lane < 2, O_PKHDR + (r0 * 64 + (cb8 >> 2)) * 128 + wv * 16 + lane * 8, hpay, wt.seq);
            st_gran_b(rs1, lane < 2, O_PKHDR + (r1 * 64 + (cb8 >> 2)) * 128 + wv * 16 + lane * 8, hpay, wt.seq);
        }
        STAMP(1);
    }
    wt.seq = rfl(seqv) + 1;
    if (lane == 0) __hip_atomic_store(seqp, wt.seq, __ATOMIC_RELAXED, __HIP_MEMORY_SCOPE_AGENT);

    bool ok = true;
    float y[16 * KEEP];
    int nk = 0, kept = 0;

    if (bid >= B_IN) {
        const int iw = (bid - B_IN) * 4 + wv, l = iw % 6, ip = iw / 6;
        if (ip >= NPI) { STAMP_DUMP(); return; }
        const int src = rank ^ (l + 2);
        char* const peer = p.rel[l + 2];
        const rsrc_t rs = mk_rsrc(peer, WS_USED);
        const char* const hb = my + O_PKHDR + l * 64 * 128 + (lane & 1) * 8;
        constexpr int D0 = HEAD + 2 * NPI, ND = 480 - D0;
        const u32 tbase = (wt.seq - 1) * (ND + NPI) - D0;
        int cntr = 2;
#define HOFF(b) (((b) & 252) * 32 + ((b) & 3) * 16 + (((b) >> 8) & 1) * (6 * 64 * 128))
#define PKIDX(b) (((2 * rm + (((b) >> 8) & 1)) * 2 + (((b) & 128) ? rj : ri)) * 256 + ((b) & 255))
        int b0 = item_at(HEAD + ip, src);
        u32 tk = tbase + HEAD + NPI + ip, fo = OOB;
        u64 h0 = ld_gran(hb + HOFF(b0));
        WAIT_VM0();
        do {
            const u32 so = O_PK + PKIDX(b0) * BLK;
            if (ok) ok = poll_gran(hb + HOFF(b0), true, h0, wt);
            Pk k;
            k.l0 = __builtin_amdgcn_raw_buffer_load_b128(rs, lane * 16, so, SYS);
            k.l1 = __builtin_amdgcn_raw_buffer_load_b128(rs, lane * 16 + 1024, so, SYS);
            k.n = __builtin_amdgcn_raw_buffer_load_b128(rs, lane * 16 + 2048, so, SYS);
            SB();
            STAMP(cntr); ++cntr;
            const u32 lo = (u32)__builtin_amdgcn_readlane((int)(u32)h0, 0), hi = (u32)__builtin_amdgcn_readlane((int)(u32)h0, 1);
            const u32 i1 = rfl(tk) - tbase;
            const int b1 = i1 < 480u ? item_at(i1, src) : -1;
            h0 = ld_gran(hb + HOFF(b1));
            tk = __builtin_amdgcn_raw_ptr_buffer_atomic_add_i32(1, rsm, (b1 >= 0 && lane == 0) ? O_TICK + l * 128 : OOB, 0, 16);
            asm volatile("s_waitcnt vmcnt(5)" ::: "memory");
            st_word_b(rsm, ok, fo, wt.seq);
            const u32 slot = (4 + (b0 >> 8) * 6 + l) * SL + (b0 & 255), o = O_IN + slot * 4096 + lane * 16;
            fo = lane == 0 ? O_INFLAG + slot * FS : OOB;
            const u32x16 d = unpack(k, lo, hi, peer + O_PKOVF + PKIDX(b0) * 1024 + lane * 16, lane);
            asm volatile("" : "+v"(h0), "+v"(tk));
            st128(FRAG(d, 0), rsm, o); st128(FRAG(d, 1), rsm, o + 1024); st128(FRAG(d, 2), rsm, o + 2048); st128(FRAG(d, 3), rsm, o + 3072);
            SB();
            b0 = b1;
        } while (b0 >= 0);
        WAIT_VM0();
        st_word_b(rsm, ok, fo, wt.seq);
        STAMP(30);
        STAMP_DUMP();
        return;
    }
    const bool puller = bid >= NG1 && bid < 256;
    const int hi = bid < 256 ? bid - NG2 : bid - 256 + NG1 - NG2;
    if (!puller) {
        const bool mainT = bid < 256 + NG2;
        if (!mainT && bid < B_IN - 96) { STAMP_DUMP(); return; }
        const int xcd = bid & 7, k8 = (bid & 255) >> 3;
        __syncthreads();
        const u32 lds0 = (u32)(size_t)(__attribute__((address_space(3))) u32x4*)ring + wv * 4096;
        const u32x4 sd = {(u32)(size_t)my, (u32)((size_t)my >> 32) & 0xffffu, WS_USED, 0x00020000u};
        u32 xs0, ws0, ob, ho;
        int od;
        bool wfirst = false;
        if (mainT) {
            int a, b;
            shell_at(bid < 256 ? k8 : 31 - k8, a, b);
            const int cx = 4 * a + (xcd >> 1), n = 2 * b + (xcd & 1);
            xs0 = ((xcd >> 1) * 4 + a) * 8 + wv * 2;
            ws0 = (16 + (xcd & 1) * 8 + b) * 8 + wv;
            ob = (cx * 4 + wv) * 32 + 2 * n;
            ho = ((ri * 8 + (cx >> 1)) * 32 + rj * 16 + n) * 128 + rm * 32;
            od = ((cx & 1) * 4 + wv) ^ rank;
            wfirst = (cx >> 1) > (n >> 1);
        } else {
            const int e = k8 - (NG1 - 96) / 8, dq = (e >> 2) + 1, c = ((e >> 1) & 1) * 4 + wv, n = ((xcd >> 1) * 2 + (e & 1)) * 2 + (xcd & 1);
            xs0 = (dq & 1) ? 256 + c * 2 : sto(2 * c + rj) * 8 + (rank & 3) * 2;
            ws0 = ((dq & 2) ? 144 : 0) + sto(16 + n) * 8 + wv;
            const int rg = (ri ^ (dq & 1)) * 8 + c, n32 = (rj ^ (dq >> 1)) * 16 + n;
            ob = 2048 + rg * 64 + 2 * n32;
            ho = (rg * 32 + n32) * 128 + 64;
            od = 0;
        }
        f32x4 acc[2][8];
#pragma unroll
        for (int a = 0; a < 2; ++a)
#pragma unroll
            for (int f = 0; f < 8; ++f) acc[a][f] = (f32x4){0.f, 0.f, 0.f, 0.f};
        ok = gemm_tile(acc, ring, lds0, sd, xs0, ws0, mainT ? 8 : 2, od >= 2 ? od + 2 : -64, wfirst, my, wt, &red[4], &red[16], lane, stb);
        asm volatile("s_nop 15" ::: "memory");
        SB();
        u32 hpay = 0;
#pragma nounroll
        for (int h = 0; h < 2; ++h) {
            u32 D[16];
            float m = 0.f;
#pragma unroll
            for (int rf = 0; rf < 2; ++rf)
#pragma unroll
                for (int r = 0; r < 4; ++r) {
                    D[rf * 8 + r * 2] = cvt2(acc[rf][0][r], acc[rf][1][r]);
                    D[rf * 8 + r * 2 + 1] = cvt2(acc[rf][2][r], acc[rf][3][r]);
                    m = fmaxf(fmaxf(m, fabsf(acc[rf][0][r])), fabsf(acc[rf][1][r]));
                    m = fmaxf(fmaxf(m, fabsf(acc[rf][2][r])), fabsf(acc[rf][3][r]));
                }
            Packed k;
            pack(D, m, k, 4);
            if (__builtin_expect(k.lo & 0x80, 0)) st128((u32x4){k.X[0], k.X[1], k.X[2], k.X[3]}, rsm, O_OUTOVF + (ob + h) * 1024 + lane * 16);
#pragma unroll
            for (int t = 0; t < 4; ++t)
                st96((u32x3){k.L[2 * t], k.L[2 * t + 1], k.N[t]}, rsm, O_OUTDATA + (ob + h) * BLK + lane * 12 + t * 768);
            hpay = (lane >> 1) == h ? ((lane & 1) ? k.hi : k.lo) : hpay;
#pragma unroll
            for (int rf = 0; rf < 2; ++rf)
#pragma unroll
                for (int f = 0; f < 4; ++f) acc[rf][f] = acc[rf][4 + f];
        }
        const rsrc_t rso = mk_rsrc(p.rel[od & 7], WS_USED);
        ok = ok && red[4] == 0;
        WAIT_VM0();
        st_gran_b(rso, ok && lane < 4, O_OUTHDR + ho + lane * 8, hpay, wt.seq);
        STAMP(16);
        if (hi < 0 || hi >= HELD) { STAMP_DUMP(); return; }
    }

    u32 owed = OOB;
    int part = wv;
    int rgA = 0, cgA = 0, rgB = 0, cgB = 0;
#pragma unroll
    for (int i = 0; i < 16 * KEEP; ++i) asm volatile("" : "=v"(y[i]));
#pragma nounroll
    for (int pass = 0; pass < 2; ++pass) {
    int bid = blockIdx.x, rank = p.rank & 7, wv = (int)rfl(threadIdx.x >> 6), lane = threadIdx.x & 63;
    asm volatile("" : "+s"(bid), "+s"(rank), "+s"(wv), "+v"(lane));
    const int rm = rank & 1, ri = (rank >> 1) & 1, rj = rank >> 2;
    if (puller) {
        const int pid = bid - NG1, grp = pid >= NP, pw = pid - grp * NP;
        const int dqa = grp ? 3 : 1, dqb = grp ? 0 : 2;
        rgA = (ri ^ (dqa & 1)) * 8; cgA = (rj ^ (dqa >> 1)) * 32; rgB = (ri ^ (dqb & 1)) * 8; cgB = (rj ^ (dqb >> 1)) * 32;
        char* const amx = p.rel[lane & 7] + O_AMAX + (rank * NCOMM + pid) * AS;
        const u32 tko = O_TICK + (6 + grp * 4 + wv) * 128, tbase = (wt.seq - 1) * (256 - NP) - 2 * NP;
        const bool hact = lane < (grp ? 10 : 12);
        const int hl = lane < 12 ? lane : 0;
        const char* const hbase = my + O_OUTHDR + (hl < 6 ? rgA * 32 + cgA / 2 : rgB * 32 + cgB / 2) * 128 + ((hl % 6) >> 1) * 32 + (hl & 1) * 8;
        const char* sp[6];
        sp[0] = p.rel[dqa * 2 + rm]; sp[1] = p.rel[dqa * 2 + (rm ^ 1)]; sp[2] = my;
        sp[3] = p.rel[dqb * 2 + rm]; sp[4] = p.rel[dqb * 2 + (rm ^ 1)]; sp[5] = my;
#define OBLK(s, c, cg) ((s) % 3 < 2 ? (u32)((((c) * 2 + rj) * 4 + (rank & 3)) * 32 + (cg)) \
                                    : (u32)(2048 + (((s) < 3 ? rgA : rgB) + (c)) * 64 + ((s) < 3 ? cgA : cgB) + (cg)))
#define OHDR(c, cg) (((c) * 32 + ((cg) >> 1)) * 128 + ((cg) & 1) * 16)
        if (pass == 0) __syncthreads();
        float mx = 0.f;
        auto decode = [&](u32 t) -> int { int c, n; shell_at(t >> 1, c, n); return t * 256 + c * 32 + 2 * n + (t & 1); };
#define IC(i) (((i) >> 5) & 7)
#define IG(i) ((i) & 31)
        int ib = -1, ic = decode(pw), nkeep = 0;
        u32x3 nxt[6];
        f32x4 v[4];
        u32 b4[6], so[3], hcur = 0, tk = tbase + NP + pw, fo = OOB;
        u64 fix = 0, hn = 0;
#pragma unroll
        for (int s = 0; s < 6; ++s) { nxt[s] = (u32x3){0, 0, 0}; b4[s] = 0; }
        auto hand_over = [&]() -> u32 {
            const bool mem = ib >= 0 && ib < HELD * 256;
            const u32 o = O_Y + (IC(ib) * 64 + IG(ib)) * 8192 + wv * 2048 + lane * 16;
            const u32 oa = mem ? o + (rgA * 64 + cgA) * 8192 : OOB, ob = mem ? o + (rgB * 64 + cgB) * 8192 : OOB;
            const u32x4 r0 = relu4(v[0]), r1 = relu4(v[1]), r2 = relu4(v[2]), r3 = relu4(v[3]);
            st128(r0, rsm, oa); st128(r1, rsm, oa + 1024); st128(r2, rsm, ob); st128(r3, rsm, ob + 1024);
            if (ib >= HELD * 256) {
                u32x4* const d = ring + wv * 1024 + nk * 256 + lane;
                d[0] = r0; d[64] = r1; d[128] = r2; d[192] = r3;
                kept = lane == nk ? ib : kept;
                ++nk;
            }
            return (mem && lane == 0) ? O_PROG + (grp * 256 + (ib >> 8)) * 128 + wv * 4 : OOB;
        };
        auto process = [&](const u32x3 (&cur)[6]) {
            u32 ha[6], hb[6];
#pragma unroll
            for (int s = 0; s < 6; ++s) dec_h(cur[s].z, b4[s], ha[s], hb[s]);
            if (__builtin_expect(fix != 0, 0)) {
#pragma unroll
                for (int s = 0; s < 6; ++s) {
                    const u32 lo = (u32)__builtin_amdgcn_readlane((int)hcur, 2 * s);
                    const u32 hi = (u32)__builtin_amdgcn_readlane((int)hcur, 2 * s + 1);
                    if (lo & 0x80) {
                        const char* pv = sp[s] + O_OUTOVF + OBLK(s, IC(ib), IG(ib)) * 1024 + lane * 16 + wv * 4;
                        ha[s] = cur[s].z;
                        asm volatile("global_load_dword %0, %1, off sc0 sc1\n\ts_waitcnt vmcnt(0)" : "=&v"(hb[s]) : "v"(pv) : "memory");
                    } else {
                        u64 pay = (((u64)hi << 32) | lo) >> 8;
                        while (pay != 0) {
                            const u32 e = (u32)pay & 0x3ffff, idx = (e >> 6) & 31;
                            pay >>= 18;
                            const u32 t = (int)(idx >> 3) == wv ? (e >> 11) << (8 * (idx & 3)) : 0;
                            const u32 ta = (idx & 4) ? 0 : t, tb = (idx & 4) ? t : 0;
                            const bool me = lane == (int)(e & 63);
                            ha[s] ^= me ? ta : 0;
                            hb[s] ^= me ? tb : 0;
                        }
                    }
                }
            }
#pragma unroll
            for (int s = 0; s < 6; ++s) {
                f32x4 a, b;
#pragma unroll
                for (int j = 0; j < 4; ++j) {
                    const u32 sel = ((4u + j) << 24) | ((u32)j << 16) | 0x0c0cu;
                    a[j] = __builtin_bit_cast(float, __builtin_amdgcn_perm(ha[s], cur[s].x, sel));
                    b[j] = __builtin_bit_cast(float, __builtin_amdgcn_perm(hb[s], cur[s].y, sel));
                }
                v[(s / 3) * 2] = s % 3 ? v[(s / 3) * 2] + a : a;
                v[(s / 3) * 2 + 1] = s % 3 ? v[(s / 3) * 2 + 1] + b : b;
            }
#pragma unroll
            for (int j = 0; j < 4; ++j) mx = fmaxf(fmaxf(fmaxf(fmaxf(mx, v[j].x), v[j].y), v[j].z), v[j].w);
            asm volatile("" : "+v"(mx));
        };
        int cntr = 20;
        auto offsets = [&]() {
#pragma unroll
            for (int z = 0; z < 3; ++z) so[z] = O_OUTDATA + OBLK(z ? 3 * z - 1 : 0, IC(ic), IG(ic)) * BLK + wv * 768;
            asm volatile("" : "+s"(so[0]), "+s"(so[1]), "+s"(so[2]));
        };
        offsets();
        if (pass) {
            const int l = (grp * 4 + wv) % 6, b = item_at(480 - TRIG + pw, rank ^ (l + 2));
            if (ok) ok = poll_word(my + O_INFLAG + ((4 + (b >> 8) * 6 + l) * SL + (b & 255)) * FS, lane == 0, wt.seq, wt);
        }
        if (pass) do {
            asm volatile("" : "+v"(nxt[0]), "+v"(nxt[1]), "+v"(nxt[2]), "+v"(nxt[3]), "+v"(nxt[4]), "+v"(nxt[5]), "+v"(hn));
            u32x3 cur[6];
#pragma unroll
            for (int s = 0; s < 6; ++s) cur[s] = nxt[s];
            int id = -1;
            u32 hc;
            {
                if (ok) ok = poll_gran(hbase + OHDR(IC(ic), IG(ic)), hact, hn, wt);
#pragma unroll
                for (int z = 0; z < 6; ++z) {
                    const int s = z < 2 ? z : z < 4 ? z + 1 : z == 4 ? 2 : 5;
                    nxt[s] = __builtin_amdgcn_raw_buffer_load_b96(mk_rsrc(sp[s], (s == 5 && grp) ? 0u : WS_USED), lane * 12,
                                                                  so[z < 4 ? 0 : z - 3], SYS);
                }
                SB();
                hc = hact ? (u32)hn : 0u;
                asm volatile("" : "+v"(hc));
                const u32 t = rfl(tk) - tbase;
                if (nkeep < KEEP && t < 256u) {
                    id = decode(t);
                    nkeep += t >= HELD;
                }
                hn = ld_gran(hbase + OHDR(IC(id), IG(id)));
                tk = __builtin_amdgcn_raw_ptr_buffer_atomic_add_i32(1, rsm, (id >= 0 && nkeep < KEEP && lane == 0) ? tko : OOB, 0, 16);
            }
            STAMP(cntr); ++cntr;
            process(cur);
            {
                const u32 fn = hand_over();
                asm volatile("s_waitcnt vmcnt(12)" ::: "memory");
                st_word_b(rsm, ok, fo, wt.seq);
                fo = fn;
                ib = ic; ic = id;
                hcur = hc;
                const u32 bv = (hc & 0x7f) * 0x01010101u;
#pragma unroll
                for (int s = 0; s < 6; ++s) b4[s] = (u32)__builtin_amdgcn_readlane((int)bv, 2 * s);
                const u32 oth = (u32)__builtin_amdgcn_update_dpp(0, (int)hc, 0xB1, 0xf, 0xf, false);
                const u64 pay = ((u64)oth << 32) | hc;
                bool need = (hc & 0x80) != 0;
#pragma unroll
                for (int z = 0; z < 3; ++z) {
                    const u32 e = (u32)(pay >> (8 + 18 * z));
                    need |= ((e & 0x3f800) != 0) & ((int)((e >> 9) & 3) == wv);
                }
                fix = __ballot(need & !(lane & 1));
                offsets();
            }
        } while (ic >= 0);
        WAIT_VM0();
        STAMP(cntr);
        process(nxt);
        const u32 wm = wave_umax(__builtin_bit_cast(u32, mx));
        u32 arrived = 0, m = 0;
        if (lane == 0) {
            __hip_atomic_fetch_max(&red[2 * pass], ok ? wm : 0xffffffffu, __ATOMIC_RELAXED, __HIP_MEMORY_SCOPE_WORKGROUP);
            CFENCE();
            arrived = __hip_atomic_fetch_add(&red[2 * pass + 1], 1u, __ATOMIC_RELAXED, __HIP_MEMORY_SCOPE_WORKGROUP);
            CFENCE();
            m = __hip_atomic_load(&red[2 * pass], __ATOMIC_RELAXED, __HIP_MEMORY_SCOPE_WORKGROUP);
        }
        m = rfl(m);
        part = rfl(arrived);
        if (part == 3 && lane < 8 * pass && m != 0xffffffffu) st_gran(amx, m, wt.seq);
        STAMP(56);
        if (BARP && part == 3) __builtin_amdgcn_s_barrier();
        if (nkeep == KEEP && lane == 0) owed = tko;
        st_word_b(rsm, ok, fo, wt.seq);
        fo = hand_over();
        if (ib < HELD * 256) {
            WAIT_VM0();
            st_word_b(rsm, ok, fo, wt.seq);
        }
#pragma unroll
        for (int j = 0; j < 4 * KEEP; ++j) {
            const f32x4 a = __builtin_bit_cast(f32x4, ring[wv * 1024 + j * 64 + lane]);
            y[4 * j] = a.x; y[4 * j + 1] = a.y; y[4 * j + 2] = a.z; y[4 * j + 3] = a.w;
        }
        if (BARP && part != 3) __builtin_amdgcn_s_barrier();
    } else if (pass) {
        rgA = (ri ^ (wv & 1)) * 8; cgA = (rj ^ (wv >> 1)) * 32; rgB = 0; cgB = 0;
        { int c, n; shell_at(hi >> 1, c, n); rgA += c; cgA += 2 * n + (hi & 1); }
        if (ok) ok = poll_word(my + O_PROG + ((wv == 0 || wv == 3 ? 256 : 0) + hi) * 128 + (lane & 3) * 4, lane < 4, wt.seq, wt);
        const u32 so = O_Y + (rgA * 64 + cgA) * 8192;
#pragma unroll
        for (int u = 0; u < 8; ++u) {
            const f32x4 a = __builtin_bit_cast(f32x4, __builtin_amdgcn_raw_buffer_load_b128(rsm, lane * 16, so + u * 1024, SYS));
            y[4 * u] = a.x; y[4 * u + 1] = a.y; y[4 * u + 2] = a.z; y[4 * u + 3] = a.w;
        }
        WAIT_VM0();
        STAMP(17);
    }

    {
        const char* a0 = my + O_AMAX + (part * 128 + lane) * AS;
        u64 v0 = 0, v1 = 0;
        bool d0 = part * 128 + lane >= NR * NCOMM, d1 = part * 128 + lane + 64 >= NR * NCOMM;
        if (pass && !puller && ok && part < 3 && hi < HELD - 2 * NP) ok = poll_word(my + O_PROG + (HELD - 1) * 128 + wv * 4, lane == 0, wt.seq, wt);
        while (pass && ok && part < 3) {
            if (!d0) v0 = ld_gran(a0);
            if (!d1) v1 = ld_gran(a0 + 64 * AS);
            d0 = d0 || (u32)(v0 >> 32) >= wt.seq;
            d1 = d1 || (u32)(v1 >> 32) >= wt.seq;
            if (__all(d0 && d1)) break;
            if (expired(wt)) ok = false;
            __builtin_amdgcn_s_sleep(1);
        }
        const u32 wm = wave_umax(umax((u32)v0, (u32)v1));
        if (lane == 0) red[8 + pass * 4 + wv] = ok ? wm : 0xffffffffu;
        __syncthreads();
        const u32x4 r4 = *(const u32x4*)&red[8 + pass * 4];
        const float am = __builtin_bit_cast(float, umax(umax(r4.x, r4.y), umax(r4.z, r4.w)));
        const float inv = am == 0.f ? 0.f : 448.0f * __builtin_amdgcn_rcpf(am);
        STAMP(57);
        const rsrc_t ro = mk_rsrc(p.out, pass ? 512u * 4096u : 0u);
        const int fr = lane & 15, fq = lane >> 4;
        if (puller) {
#pragma unroll
            for (int r = 0; r < 2 * KEEP; ++r) {
                if (pass && (r >> 1) >= nk) break;
                const u32x2 o = {q4(y[8 * r], y[8 * r + 1], y[8 * r + 2], y[8 * r + 3], inv),
                                 q4(y[8 * r + 4], y[8 * r + 5], y[8 * r + 6], y[8 * r + 7], inv)};
                const int it = __builtin_amdgcn_readlane(kept, r >> 1);
                const int rg = ((r & 1) ? rgB : rgA) + IC(it), cg = ((r & 1) ? cgB : cgA) + IG(it);
                __builtin_amdgcn_raw_buffer_store_b64(o, ro, ((wv >> 1) * 16 + fr) * 4096 + fq * 16 + (wv & 1) * 8, rg * 131072 + cg * 64, OUT_AUX);
            }
            if (pass) __builtin_amdgcn_raw_ptr_buffer_atomic_add_i32(1, rsm, owed, 0, 16);
        } else {
#pragma unroll
            for (int rf = 0; rf < 2; ++rf) {
                u32x4 o;
#pragma unroll
                for (int j = 0; j < 4; ++j)
                    o[j] = q4(y[16 * rf + 4 * j], y[16 * rf + 4 * j + 1], y[16 * rf + 4 * j + 2], y[16 * rf + 4 * j + 3], inv);
                __builtin_amdgcn_raw_buffer_store_b128(o, ro, (rgA * 32 + rf * 16 + fr) * 4096 + cgA * 64 + fq * 16, 0, OUT_AUX);
            }
        }
    }
    }
    STAMP(58);
    STAMPC(59);
    STAMP_DUMP();
}

extern "C" void kernel_launch(int rank, int nranks, ncclComm_t comm, hipStream_t stream,
                              void* const* d_in, const int* in_sizes, int n_in,
                              void* d_out, int out_size,
                              const int* in_dtypes, int out_dtype,
                              void* const* d_out_all,
                              void* const* d_ws_all, size_t ws_bytes) {
    (void)comm; (void)in_dtypes; (void)out_dtype; (void)d_out_all;
    if (nranks != NR || n_in != 2 || (unsigned)rank >= (unsigned)NR || ws_bytes < WS_USED ||
        in_sizes[0] != 4096 * 512 || in_sizes[1] != 512 * 4096 || out_size != 512 * 4096) {
        fprintf(stderr, "kernel_launch: unexpected problem shape\n");
        return;
    }
    Params p;
    p.x = (const float*)d_in[0];
    p.w = (const float*)d_in[1];
    p.out = (unsigned char*)d_out;
    p.my = (char*)d_ws_all[rank];
    for (int r = 0; r < NR; ++r) p.rel[r] = (char*)d_ws_all[r ^ rank];
    p.rank = rank;
    p.pad = 0;
    hipLaunchKernelGGL(fused, dim3(GRID), dim3(256), 0, stream, p);
    HIP_CHECK(hipGetLastError());
}
```

```cpp
#include <hip/hip_runtime.h>
#include <rccl/rccl.h>
#include <cstdio>

#define HIP_CHECK(expr) do { hipError_t _e = (expr); if (_e != hipSuccess) { \
    fprintf(stderr, "HIP error %s at %s:%d\n", hipGetErrorString(_e), __FILE__, __LINE__); \
    return; } } while (0)

typedef unsigned u32;
typedef unsigned long long u64;
typedef u32 u32x2 __attribute__((ext_vector_type(2)));
typedef u32 u32x3 __attribute__((ext_vector_type(3)));
typedef u32 u32x4 __attribute__((ext_vector_type(4)));
typedef u32 u32x8 __attribute__((ext_vector_type(8)));
typedef u32 u32x16 __attribute__((ext_vector_type(16)));
typedef float f32x2 __attribute__((ext_vector_type(2)));
typedef float f32x4 __attribute__((ext_vector_type(4)));
typedef __bf16 bf16x2 __attribute__((ext_vector_type(2)));
typedef __bf16 bf16x8 __attribute__((ext_vector_type(8)));
typedef __attribute__((address_space(1))) u32 gu32;
typedef __attribute__((address_space(1))) u64 gu64;
typedef __amdgpu_buffer_rsrc_t rsrc_t;

#define HEAD 36
#define STAG 3
#define ORDER 0x0128394a5b6c7defull
#define TRIG 64
#define NPI 26
#define STAG0 0
#define STAGP 11
#define AS 128u
#define HW 40
#define SRC_AUX 2
#define OUT_AUX SYS
#define TEST_BASE_SHIFT 0

constexpr int NR = 8;
#define NPO 24
#define NKEPT 64
constexpr int NP = NPO;
constexpr int NCOMM = 2 * NP;
constexpr int GRID = 512;
constexpr int NG1 = 256 - NCOMM;
constexpr int NG2 = 256 - NG1;
constexpr int B_IN = 256 + NG1;
constexpr int NIN = (6 * NPI + 3) / 4;
constexpr int KEEP = 4;
constexpr int HELD = 256 - NKEPT;
static_assert(HELD <= NG1 && NG1 % 8 == 0 && NG2 + NIN + 96 <= NG1 && B_IN + NIN <= GRID && NKEPT <= NP * KEEP && 2 * NP <= HELD && HEAD <= 64 && NCOMM * 4 % 6 == 0, "roles");
constexpr u64 DEADLINE = 30000000ull;

constexpr u32 FS = 128;
constexpr u32 BLK = 3072;
constexpr u32 SL = 50 * 8;
constexpr u32 N_IN = 16 * SL, N_PK = 2048, N_OUT = 3072;
constexpr u32 O_SEQ = 0;
constexpr u32 O_TMO = O_SEQ + GRID * 4 * 4;
constexpr u32 O_TICK = O_TMO + 128;
constexpr u32 O_AMAX = O_TMO + 2048;
constexpr u32 O_PROG = O_AMAX + 512 * 128;
constexpr u32 O_PKHDR = O_PROG + 1024 * 128;
constexpr u32 O_OUTHDR = O_PKHDR + 12 * 64 * 128;
constexpr u32 O_INFLAG = O_OUTHDR + 512 * 128;
constexpr u32 O_IN = ((O_INFLAG + N_IN * FS + 4095) / 4096) * 4096;
constexpr u32 O_PK = O_IN + N_IN * 4096;
constexpr u32 O_PKL = O_PK + N_PK * BLK;
constexpr u32 O_OUTDATA = O_PKL + 6 * 64 * BLK;
constexpr u32 O_Y = O_OUTDATA + N_OUT * BLK;
constexpr u32 O_PKOVF = O_Y + 1024 * 8192;
constexpr u32 O_OUTOVF = O_PKOVF + N_PK * 1024;
constexpr u32 O_DBG = O_OUTOVF + N_OUT * 1024;
constexpr u32 WS_USED = O_DBG + GRID * 4 * 64 * 8;
constexpr u32 OOB = 0xfffff000u;
static_assert(O_AMAX % 128 == 0 && O_PROG % 128 == 0 && O_INFLAG % 128 == 0 && O_PKHDR % 128 == 0 && O_OUTHDR % 128 == 0, "align");
static_assert(NR * NCOMM <= 384 && GRID * 4 == 2048, "shape");

constexpr int SYS = 17;

struct Params {
    const float* x;
    const float* w;
    unsigned char* out;
    char* my;
    int rank;
    int pad;
    char* rel[NR];
};

#define STAMP(id) do { } while (0)
#define STAMPC(id) do { } while (0)
#define STAMP_INIT() const u32 stb = 0;
#define STAMP_DUMP() do { } while (0)
#define POLL_SLEEP 4

#define DEV __device__ __forceinline__
#define CFENCE() asm volatile("" ::: "memory")
#define WAIT_VM0() do { __builtin_amdgcn_s_waitcnt(0x0F70); asm volatile("s_waitcnt vmcnt(0)" ::: "memory"); } while (0)

DEV rsrc_t mk_rsrc(const void* base, u32 bytes) { return __builtin_amdgcn_make_buffer_rsrc((void*)base, 0, bytes, 0x00020000); }
DEV u32 rfl(u32 v) { return __builtin_amdgcn_readfirstlane(v); }
DEV u32 cvt2(float a, float b) { f32x2 v = {a, b}; return __builtin_bit_cast(u32, __builtin_convertvector(v, bf16x2)); }
DEV u32 umax(u32 a, u32 b) { return a > b ? a : b; }
DEV u32 wave_umax(u32 v) {
    v = umax(v, (u32)__builtin_amdgcn_update_dpp(0, (int)v, 0x111, 0xf, 0xf, true));
    v = umax(v, (u32)__builtin_amdgcn_update_dpp(0, (int)v, 0x112, 0xf, 0xf, true));
    v = umax(v, (u32)__builtin_amdgcn_update_dpp(0, (int)v, 0x114, 0xf, 0xf, true));
    v = umax(v, (u32)__builtin_amdgcn_update_dpp(0, (int)v, 0x118, 0xf, 0xf, true));
    v = umax(v, (u32)__builtin_amdgcn_update_dpp(0, (int)v, 0x142, 0xa, 0xf, false));
    v = umax(v, (u32)__builtin_amdgcn_update_dpp(0, (int)v, 0x143, 0xc, 0xf, false));
    return (u32)__builtin_amdgcn_readlane((int)v, 63);
}

struct Wait {
    u32 seq;
    u64 t0;
    gu32* tmo;
};
DEV bool expired(const Wait& wt) {
    const u64 dt = __builtin_amdgcn_s_memrealtime() - wt.t0;
    if (dt < 100000ull) return false;
    if (dt > DEADLINE || rfl(__hip_atomic_load(wt.tmo, __ATOMIC_RELAXED, __HIP_MEMORY_SCOPE_AGENT)) == wt.seq) {
        __hip_atomic_store(wt.tmo, wt.seq, __ATOMIC_RELAXED, __HIP_MEMORY_SCOPE_AGENT);
        return true;
    }
    return false;
}
DEV u64 ld_gran(const char* p) { return __hip_atomic_load((gu64*)p, __ATOMIC_RELAXED, __HIP_MEMORY_SCOPE_SYSTEM); }
DEV void st_gran(char* p, u32 payload, u32 seq) {
    __hip_atomic_store((gu64*)p, ((u64)seq << 32) | payload, __ATOMIC_RELAXED, __HIP_MEMORY_SCOPE_SYSTEM);
}
DEV bool poll_gran(const char* p, bool act, u64& v, const Wait& wt) {
    bool ok = true;
    bool done = !act || (u32)(v >> 32) >= wt.seq;
    if (__builtin_expect(!__all(done), 0)) {
        for (;;) {
            if (!done) { v = ld_gran(p); done = (u32)(v >> 32) >= wt.seq; }
            if (__all(done)) break;
            if (expired(wt)) { ok = false; break; }
            __builtin_amdgcn_s_sleep(POLL_SLEEP);
        }
    }
    CFENCE();
    return ok;
}
DEV bool poll_word(const char* p, bool act, u32 need, const Wait& wt) {
    bool ok = true, done = !act;
    for (;;) {
        if (!done) done = __hip_atomic_load((gu32*)p, __ATOMIC_RELAXED, __HIP_MEMORY_SCOPE_SYSTEM) >= need;
        if (__all(done)) break;
        if (expired(wt)) { ok = false; break; }
        __builtin_amdgcn_s_sleep(POLL_SLEEP);
    }
    CFENCE();
    return ok;
}

DEV bool poll_part(const char* p, bool& done, u64 want, const Wait& wt) {
    bool ok = true;
    if ((__ballot(done) & want) != want) {
        for (;;) {
            if (!done) done = __hip_atomic_load((gu32*)p, __ATOMIC_RELAXED, __HIP_MEMORY_SCOPE_SYSTEM) >= wt.seq;
            if ((__ballot(done) & want) == want) break;
            if (expired(wt)) { ok = false; break; }
            __builtin_amdgcn_s_sleep(POLL_SLEEP);
        }
    }
    CFENCE();
    return ok;
}

struct Packed {
    u32 L[8], N[4], X[4];
    u32 lo, hi;
};
DEV void pack(const u32 (&D)[16], float amax, Packed& o, int kind) {
    const u32 top = wave_umax(__builtin_bit_cast(u32, amax)) >> 24;
    int b = (int)(top & 0x7f) - 7 - ((TEST_BASE_SHIFT & kind) ? 1 : 0);
    b = b < 0 ? 0 : b;
    const u32 base4 = (u32)b * 0x01010101u;
    u32 H[8], d[8], chk = 0;
#pragma unroll
    for (int q = 0; q < 8; ++q) {
        o.L[q] = __builtin_amdgcn_perm(D[2 * q + 1], D[2 * q], 0x06040200u);
        H[q] = __builtin_amdgcn_perm(D[2 * q + 1], D[2 * q], 0x07050301u);
        d[q] = H[q] - base4;
        chk |= d[q];
    }
    u64 pay = (u64)b;
    u64 m = __ballot((chk & 0x78787878u) != 0);
    if (__builtin_expect(m != 0, 0)) {
        u32x8 dl;
        u32 esc = 0;
#pragma unroll
        for (int q = 0; q < 8; ++q) {
            d[q] = (((H[q] | 0x80808080u) - base4) & 0x07070707u) | (H[q] & 0x80808080u);
            dl[q] = (d[q] + base4) ^ H[q];
            esc |= ((dl[q] + 0x7f7f7f7fu) & 0x80808080u) >> (7 - q);
        }
        int cnt = 0;
        while (m != 0 && cnt <= 3) {
            const int l = __builtin_ctzll(m);
            m &= m - 1;
            u32 bits = (u32)__builtin_amdgcn_readlane((int)esc, l);
            while (bits != 0 && cnt <= 3) {
                const int pos = __builtin_ctz(bits);
                bits &= bits - 1;
                const u32 e = ((u32)__builtin_amdgcn_readlane((int)dl[pos & 7], l) >> (pos & 24)) & 0x7f;
                pay |= (u64)((u32)l | ((u32)(4 * (pos & 7) + (pos >> 3)) << 6) | (e << 11)) << (8 + 18 * cnt);
                ++cnt;
            }
        }
        if (cnt > 3) {
            pay = 0x80;
#pragma unroll
            for (int t = 0; t < 4; ++t) { d[2 * t] = H[2 * t]; d[2 * t + 1] = 0; }
        }
    }
#pragma unroll
    for (int t = 0; t < 4; ++t) {
        o.N[t] = d[2 * t] | __builtin_amdgcn_alignbit(d[2 * t + 1], d[2 * t + 1], 28);
        o.X[t] = H[2 * t + 1];
    }
    o.lo = (u32)pay;
    o.hi = (u32)(pay >> 32);
}
DEV void dec_h(u32 n, u32 base4, u32& ha, u32& hb) {
    ha = (n & 0x87878787u) + base4;
    hb = (__builtin_amdgcn_alignbit(n, n, 4) & 0x87878787u) + base4;
}
DEV u32x4 mk_frag(u32 la, u32 lb, u32 ha, u32 hb) {
    u32x4 r;
    r.x = __builtin_amdgcn_perm(ha, la, 0x05010400u);
    r.y = __builtin_amdgcn_perm(ha, la, 0x07030602u);
    r.z = __builtin_amdgcn_perm(hb, lb, 0x05010400u);
    r.w = __builtin_amdgcn_perm(hb, lb, 0x07030602u);
    return r;
}
struct Pk { u32x4 l0, l1, n; };
DEV u32x16 cat4(u32x4 a, u32x4 b, u32x4 c, u32x4 d) {
    const u32x8 l = __builtin_shufflevector(a, b, 0, 1, 2, 3, 4, 5, 6, 7), h = __builtin_shufflevector(c, d, 0, 1, 2, 3, 4, 5, 6, 7);
    return __builtin_shufflevector(l, h, 0, 1, 2, 3, 4, 5, 6, 7, 8, 9, 10, 11, 12, 13, 14, 15);
}
#define FRAG(v, t) __builtin_shufflevector(v, v, 4 * (t), 4 * (t) + 1, 4 * (t) + 2, 4 * (t) + 3)
DEV u32x16 unpack(const Pk& k, u32 lo, u32 hi, const char* ovf, int lane) {
    const u32 base4 = (lo & 0x7f) * 0x01010101u;
    u32x4 ha, hb;
#pragma unroll
    for (int t = 0; t < 4; ++t) { u32 a, b; dec_h(k.n[t], base4, a, b); ha[t] = a; hb[t] = b; }
    if (__builtin_expect(((lo >> 7) | hi) != 0, 0)) {
        if (lo & 0x80) {
            ha = k.n;
            asm volatile("global_load_dwordx4 %0, %1, off sc0 sc1\n\ts_waitcnt vmcnt(0)" : "=&v"(hb) : "v"(ovf) : "memory");
        } else {
            u64 pay = (((u64)hi << 32) | lo) >> 8;
            while (pay != 0) {
                const u32 e = (u32)pay & 0x3ffff, idx = (e >> 6) & 31;
                pay >>= 18;
                const u32 t = lane == (int)(e & 63) ? (e >> 11) << (8 * (idx & 3)) : 0;
#pragma unroll
                for (u32 z = 0; z < 4; ++z) {
                    ha[z] ^= (idx >> 2) == 2 * z ? t : 0;
                    hb[z] ^= (idx >> 2) == 2 * z + 1 ? t : 0;
                }
            }
        }
    }
    return cat4(mk_frag(k.l0.x, k.l0.y, ha.x, hb.x), mk_frag(k.l0.z, k.l0.w, ha.y, hb.y),
                mk_frag(k.l1.x, k.l1.y, ha.z, hb.z), mk_frag(k.l1.z, k.l1.w, ha.w, hb.w));
}
DEV void st128(u32x4 v, rsrc_t rs, u32 off) { __builtin_amdgcn_raw_buffer_store_b128(v, rs, off, 0, SYS); }
DEV void st96(u32x3 v, rsrc_t rs, u32 off) { __builtin_amdgcn_raw_buffer_store_b96(v, rs, off, 0, SYS); }
DEV void st_gran_b(rsrc_t rs, bool act, u32 off, u32 payload, u32 seq) {
    const u32x2 v = {payload, seq};
    __builtin_amdgcn_raw_buffer_store_b64(v, rs, act ? off : OOB, 0, SYS);
}

DEV void st_word_b(rsrc_t rs, bool act, u32 off, u32 v) { __builtin_amdgcn_raw_buffer_store_b32(v, rs, act ? off : OOB, 0, SYS); }

DEV int shell_of(int e) { return (int)(ORDER >> (60 - 4 * e)) & 15; }
DEV int sto(int ch) { return ch < 16 ? (ch & 3) * 4 + (ch >> 2) : 16 + (ch & 1) * 8 + ((ch - 16) >> 1); }
DEV int item_at(int i, int src) {
    const int e = i / 30, ps = shell_of(e);
    int r = i - e * 30;
    if ((r >> 1) >= src) r += 2;
    return (ps >> 3) * 256 + sto((r >> 4) * 16 + (ps & 7) * 2 + ((r >> 3) & 1)) * 8 + (r & 7);
}
struct Cv { int pos, dm, cls, blk, ch, pass, ks; };
DEV Cv cv_of(int u) {
    Cv c;
    c.pos = u >> 5; c.dm = (u >> 4) & 1; c.cls = (u >> 3) & 1; c.blk = u & 7;
    const int ps = shell_of(c.pos >> 2);
    c.ch = (c.pos & 2) * 8 + (ps & 7) * 2 + (c.pos & 1); c.pass = ps >> 3; c.ks = 2 * c.dm + c.pass;
    return c;
}
DEV void ld_src(const Params& p, int lane, int u, f32x4 (&F)[8]) {
    const Cv c = cv_of(u);
    const int fr = lane & 15, fq = lane >> 4;
    const bool isx = c.ch < 16;
    const int owner = (c.ch & 1) * 4 + (c.blk >> 1);
    u32 so = isx ? ((owner * 512 + (c.cls * 8 + (c.ch >> 1)) * 32) * 512 + c.ks * 128 + (c.blk & 1) * 64) * 4
                 : ((c.ks * 128 + (c.blk >> 1) * 32) * 4096 + c.cls * 2048 + (c.ch - 16) * 128 + (c.blk & 1) * 64) * 4;
    u32 s2 = isx ? 16 * 2048 : 16 * 16384, s1 = isx ? 128 : 2 * 16384, s0 = isx ? 64 : 16384;
    u32 vo = isx ? fr * 2048 + fq * 16 : fq * 4 * 16384 + fr * 16;
    asm volatile("" : "+s"(so), "+s"(s2), "+s"(s1), "+s"(s0), "+v"(vo));
    __builtin_amdgcn_sched_barrier(0);
    const rsrc_t rs = mk_rsrc(isx ? p.x : p.w, 4096u * 512u * 4u);
#pragma unroll
    for (int i = 0; i < 8; ++i)
        F[i] = __builtin_bit_cast(f32x4, __builtin_amdgcn_raw_buffer_load_b128(
                   rs, vo, so + (i >> 2) * s2 + ((i >> 1) & 1) * s1 + (i & 1) * s0, SRC_AUX));
    __builtin_amdgcn_sched_barrier(0);
}
DEV float cvt_src(const f32x4 (&F)[8], bool isx, u32 (&D)[16]) {
    if (isx) {
#pragma unroll
        for (int u = 0; u < 8; ++u) { D[2 * u] = cvt2(F[u].x, F[u].y); D[2 * u + 1] = cvt2(F[u].z, F[u].w); }
    } else {
#pragma unroll
        for (int f = 0; f < 4; ++f)
#pragma unroll
            for (int e = 0; e < 4; ++e) D[4 * f + e] = cvt2(F[2 * e][f], F[2 * e + 1][f]);
    }
    float m = 0.f;
#pragma unroll
    for (int u = 0; u < 8; ++u) {
        m = fmaxf(fmaxf(m, fabsf(F[u].x)), fabsf(F[u].y));
        m = fmaxf(fmaxf(m, fabsf(F[u].z)), fabsf(F[u].w));
    }
    return m;
}
DEV void st_plain(const u32 (&D)[16], rsrc_t rs, u32 off) {
#pragma unroll
    for (int t = 0; t < 4; ++t) st128((u32x4){D[4 * t], D[4 * t + 1], D[4 * t + 2], D[4 * t + 3]}, rs, off + t * 1024);
}
DEV void shell_at(int t, int& a, int& b) {
    int s = 0;
    while (2 * (s + 1) * (s + 1) <= t) ++s;
    int r = t - 2 * s * s;
    if (r < 2 * s) { a = s; b = r; }
    else { r -= 2 * s; const int up = r > s; a = r - up * (s + 1); b = 2 * s + up; }
}
DEV u32x4 relu4(f32x4 v) {
    typedef int i32x4 __attribute__((ext_vector_type(4)));
    return __builtin_bit_cast(u32x4, __builtin_elementwise_max(__builtin_bit_cast(i32x4, v), (i32x4){0, 0, 0, 0}));
}
DEV u32 q4(float a, float b, float c, float d, float inv) {
    const f32x2 lo = (f32x2){a, b} * inv, hi = (f32x2){c, d} * inv;
    int r = __builtin_amdgcn_cvt_pk_fp8_f32(lo.x, lo.y, __builtin_bit_cast(int, lo.x), false);
    r = __builtin_amdgcn_cvt_pk_fp8_f32(hi.x, hi.y, r, true);
    return (u32)r;
}

#define XSLOT(s) (xs0 + ((s) >> 1) * SL + ((s) & 1))
#define WSLOT(s) (ws0 + ((s) >> 1) * SL + ((s) & 1) * 4)
#define SB() __builtin_amdgcn_sched_barrier(0)
template <int F>
DEV void ld_x(u32x4& d, u32x4 sd, u32 vo, u32 so) {
    asm volatile("buffer_load_dwordx4 %0, %1, %2, %3 offen offset:%4 sc0 sc1" : "+v"(d) : "v"(vo), "s"(sd), "s"(so), "i"(F * 1024));
}
template <int N>
DEV void wait_step(u32x4 (&x)[4]) {
    __builtin_amdgcn_s_waitcnt(0xC07F);
    asm volatile("s_waitcnt vmcnt(%4) lgkmcnt(0)\n\ts_barrier" : "+v"(x[0]), "+v"(x[1]), "+v"(x[2]), "+v"(x[3]) : "i"(N) : "memory");
}
DEV void mma(f32x4& c, u32x4 a, u32x4 b) { asm volatile("v_mfma_f32_16x16x32_bf16 %0, %1, %2, %0" : "+v"(c) : "v"(a), "v"(b)); }
template <int F>
DEV void mma_ldw(f32x4& c, u32x4 a, u32x4 b, u32 lds, u32x4 sd, u32 vo, u32 so) {
    asm volatile("s_mov_b32 m0, %3\n\tv_mfma_f32_16x16x32_bf16 %0, %1, %2, %0\n\tbuffer_load_dwordx4 %4, %5, %6 offen offset:%7 sc0 sc1 lds"
                 : "+v"(c) : "v"(a), "v"(b), "s"(lds), "v"(vo), "s"(sd), "s"(so), "i"(F * 1024) : "memory");
}
template <int S, int G>
DEV bool gemm_steps(f32x4 (&acc)[2][8], u32x4 (&X)[4][4], u32x4 (&a)[8], const u32x4* ring, u32 lds0, u32x4 sd, u32 xs0, u32 ws0,
                    int skip, int lane, u32 fo, u32& fv, u32 seq, u32* vote) {
    const u32 vo = lane * 16;
    u32x4 b[8];
    const int sl = (S + 3) >> 1;
    const u32 vx = (sl == skip || sl == skip + 6) ? 0x80000000u : vo;
    const u32 sx = O_IN + XSLOT(S + 3) * 4096, sw = O_IN + WSLOT(S + 4) * 4096, l = lds0 + S * 16384;
    constexpr bool LX = G < 2 || S == 0, LW = G < 2, END = G == 2 && S == 3;
    u32 vt = 0;
    if (G < 2 && S == 0) asm volatile("buffer_load_dword %0, %1, %2, 0 offen sc0 sc1" : "+v"(fv) : "v"(fo), "s"(sd));
#pragma unroll
    for (int i = 0; i < 8; ++i) {
        mma(acc[0][i], a[i], X[S][0]);
        SB();
        if (i < 6) b[i < 4 ? i + (i + 1) / 2 : i + 2] = ring[S * 1024 + (8 + (i < 4 ? i + (i + 1) / 2 : i + 2)) * 64 + lane];
        if (G < 2 && S == 3 && i == 6) vote[(fo != OOB && fv < seq) ? 0 : 8] = 1;
        SB();
        mma(acc[1][i], a[i], X[S][2]);
        SB();
        if (i == 0 || i == 2) b[i + i / 2 + 1] = ring[S * 1024 + (8 + i + i / 2 + 1) * 64 + lane];
        if (LX && i == 1) ld_x<0>(X[(S + 3) & 3][0], sd, vx, sx);
        if (LX && i == 3) ld_x<1>(X[(S + 3) & 3][1], sd, vx, sx);
        if (LX && i == 4) ld_x<2>(X[(S + 3) & 3][2], sd, vx, sx);
        if (LX && i == 6) ld_x<3>(X[(S + 3) & 3][3], sd, vx, sx);
        SB();
    }
    if (!END) wait_step<(G == 0 && S == 0 ? 8 : G == 0 && S == 1 ? 12 : G < 2 || S == 0 ? 16 : S == 1 ? 8 : 0)>(X[(S + 1) & 3]);
    if (G < 2 && S == 2) asm volatile("" : "+v"(fv));
    if (G < 2 && S == 3) vt = *vote;
    SB();
#pragma unroll
    for (int i = 0; i < 8; ++i) {
        mma(acc[0][i], b[i], X[S][1]);
        SB();
        if (!END) a[i] = ring[((S + 1) & 3) * 1024 + i * 64 + lane];
        SB();
        if (LW && i == 0) mma_ldw<0>(acc[1][i], b[i], X[S][3], l, sd, vo, sw);
        else if (LW && i == 2) mma_ldw<1>(acc[1][i], b[i], X[S][3], l, sd, vo, sw);
        else if (LW && i == 4) mma_ldw<2>(acc[1][i], b[i], X[S][3], l, sd, vo, sw);
        else if (LW && i == 6) mma_ldw<3>(acc[1][i], b[i], X[S][3], l, sd, vo, sw);
        else mma(acc[1][i], b[i], X[S][3]);
        SB();
    }
    if constexpr (S < 3) return gemm_steps<S + 1, G>(acc, X, a, ring, lds0, sd, xs0, ws0, skip, lane, fo, fv, seq, vote);
    else return rfl(vt) == 0;
}
DEV void first_w(u32 lds0, u32x4 sd, u32 ws0, int lane) {
#pragma unroll
    for (int s = 0; s < 4; ++s) {
        const u32 sw = O_IN + WSLOT(s) * 4096, l = lds0 + s * 16384, vo = lane * 16;
        asm volatile("s_mov_b32 m0, %0\n\ts_nop 0\n\tbuffer_load_dwordx4 %1, %2, %3 offen sc0 sc1 lds\n\t"
                     "buffer_load_dwordx4 %1, %2, %3 offen offset:1024 sc0 sc1 lds\n\tbuffer_load_dwordx4 %1, %2, %3 offen offset:2048 sc0 sc1 lds\n\t"
                     "buffer_load_dwordx4 %1, %2, %3 offen offset:3072 sc0 sc1 lds" :: "s"(l), "v"(vo), "s"(sd), "s"(sw) : "memory");
    }
}
DEV bool gemm_tile(f32x4 (&acc)[2][8], const u32x4* ring, u32 lds0, u32x4 sd, u32 xs0, u32 ws0, int gend, int skip, bool wfirst,
                   const char* my, const Wait& wt, u32* fail, u32* votes, int lane, u32 stb) {
    u32x4 X[4][4], a[8];
#pragma unroll
    for (int i = 0; i < 16; ++i) X[i >> 2][i & 3] = (u32x4){0, 0, 0, 0};
    const u32 f0 = O_INFLAG + ((lane & 16) ? WSLOT(lane & 15) : XSLOT(lane & 15)) * FS;
    const bool none = lane >= 32 || (lane & 15) >= 12 || (lane >> 1) == skip - 4;
    bool ok = true;
    u32 fv = 0;
    int g = 0;
    do {
        bool done = g ? none : lane >= 32 || (lane & 15) >= 8;
        const char* const fp = my + f0 + g * (2 * SL * FS);
        asm volatile("s_waitcnt lgkmcnt(0)\n\ts_barrier" ::: "memory");
        if (wfirst) {
            if (ok) ok = poll_part(fp, done, 0xffff0000ull, wt);
            first_w(lds0, sd, ws0, lane);
        }
        if (ok) ok = poll_part(fp, done, 0xffffull, wt);
        STAMP(20 + g);
#pragma unroll
        for (int s = 0; s < 3; ++s) {
            const u32 sx = O_IN + XSLOT(s) * 4096, vx = ((s >> 1) == skip || (s >> 1) == skip + 6) ? 0x80000000u : lane * 16;
            ld_x<0>(X[s][0], sd, vx, sx); ld_x<1>(X[s][1], sd, vx, sx); ld_x<2>(X[s][2], sd, vx, sx); ld_x<3>(X[s][3], sd, vx, sx);
        }
        if (!wfirst) {
            if (ok) ok = poll_part(fp, done, 0xffff0000ull, wt);
            STAMP(20 + g);
            first_w(lds0, sd, ws0, lane);
        } else {
            asm volatile("s_waitcnt vmcnt(8)" ::: "memory");
        }
        if (!ok && lane == 0) *fail = 1;
        wait_step<12>(X[0]);
        SB();
#pragma unroll
        for (int i = 0; i < 8; ++i) a[i] = ring[i * 64 + lane];
        SB();
#define NEXT_FO ((g == 0 || g == 3) && !none ? f0 + (g + 2) * (2 * SL * FS) : OOB)
#define ADVANCE() do { STAMP(30 + g); ++g; xs0 += 2 * SL; ws0 += 2 * SL; skip -= 2; } while (0)
        bool go = gemm_steps<0, 0>(acc, X, a, ring, lds0, sd, xs0, ws0, skip, lane, NEXT_FO, fv, wt.seq, votes + g);
        ADVANCE();
        while (g + 1 < gend && go) {
            go = gemm_steps<0, 1>(acc, X, a, ring, lds0, sd, xs0, ws0, skip, lane, NEXT_FO, fv, wt.seq, votes + g);
            ADVANCE();
        }
        gemm_steps<0, 2>(acc, X, a, ring, lds0, sd, xs0, ws0, skip, lane, OOB, fv, wt.seq, votes);
        ADVANCE();
    } while (g < gend);
    return ok;
}

__global__ void __launch_bounds__(256, 2) fused(const Params p) {
    __shared__ u32x4 ring[4096];
    __shared__ __attribute__((aligned(16))) u32 red[48];
    u32 ka0, ka1;
    asm volatile("s_load_dword %0, %2, 0x0\n\ts_load_dword %1, %2, 0x40" : "=&s"(ka0), "=&s"(ka1) : "s"(__builtin_amdgcn_kernarg_segment_ptr()));
    const int lane = threadIdx.x & 63;
    const int wv = (int)rfl(threadIdx.x >> 6);
    const int bid = blockIdx.x;
    int wg = bid < HW ? bid : bid < NG1 ? bid + NCOMM : bid < 256 ? bid - NG1 + HW : bid < B_IN ? bid : bid < B_IN + NIN ? GRID : bid - NIN;
    f32x4 F[8];
    if (wg < GRID) {
        for (int z = wg >= STAGP * 8 ? (wg >> 3) * STAG + STAG0 : 0; z > 0; --z) __builtin_amdgcn_s_sleep(1);
        ld_src(p, lane, wg * 4 + wv, F);
    }
    asm volatile("s_waitcnt lgkmcnt(0)" :: "s"(ka0), "s"(ka1));
    const int rank = p.rank & 7, rm = rank & 1, ri = (rank >> 1) & 1, rj = rank >> 2;
    char* const my = p.my;
    const rsrc_t rsm = mk_rsrc(my, WS_USED);

    Wait wt;
    wt.t0 = __builtin_amdgcn_s_memrealtime();
    wt.tmo = (gu32*)(my + O_TMO);
    gu32* const seqp = (gu32*)(my + O_SEQ + (bid * 4 + wv) * 4);
    const u32 seqv = __hip_atomic_load(seqp, __ATOMIC_RELAXED, __HIP_MEMORY_SCOPE_AGENT);
    STAMP_INIT();
    STAMP(0);
    STAMPC(60);
    if (threadIdx.x < 32) red[threadIdx.x] = 0;
    wt.seq = 0;
    u32* hx = red + 32;
    while (wg < GRID) {
        const Cv cv = cv_of(wg * 4 + wv);
        const int pos = cv.pos, dm = cv.dm, cls = cv.cls, blk = cv.blk, ch = cv.ch, pass = cv.pass, ks = cv.ks;
        u32 D[16];
        STAMP(62);
        const bool isx = ch < 16, own = isx && (ch & 1) * 4 + (blk >> 1) == rank, same = cls == (isx ? ri : rj);
        const int dc = same ? 0 : (isx ? 2 : 4), d0 = (dc ^ (isx ? 4 : 2)) | (dm ^ rm), d1 = dc | (dm ^ rm);
        const rsrc_t rs0 = mk_rsrc(p.rel[d0], WS_USED), rs1 = mk_rsrc(p.rel[d1], same ? 0u : WS_USED);
        const u32 cb8 = sto(ch) * 8 + blk;
        const u32 r0 = pass * 6 + d0 - 2, r1 = same ? 0 : pass * 6 + d1 - 2;
        const u32 ls = ks * SL + (same ? cb8 : isx ? 256 + (ch >> 1) * 2 + (blk & 1) : 144 + cb8);
        const int r32 = (pos & 3) * 8 + blk;
        const int item = (pos >> 2) * 30 + r32 - ((r32 >> 1) > rank ? 2 : 0);
        const bool local = same || own || !isx;
        const float m = cvt_src(F, isx, D);
        STAMP(63);
        u32 hpay = 0;
        if (!own) {
            Packed k;
            pack(D, m, k, isx ? 1 : 2);
            const u32 pk = (ks * 2 + cls) * 256 + cb8;
            if (__builtin_expect(k.lo & 0x80, 0)) st128((u32x4){k.X[0], k.X[1], k.X[2], k.X[3]}, rsm, O_PKOVF + pk * 1024 + lane * 16);
            const u32x4 l0 = {k.L[0], k.L[1], k.L[2], k.L[3]}, l1 = {k.L[4], k.L[5], k.L[6], k.L[7]}, nn = {k.N[0], k.N[1], k.N[2], k.N[3]};
            if (item < HEAD) {
                const u32 o0 = O_PKL + ((d0 - 2) * 64 + item) * BLK + lane * 16, o1 = O_PKL + ((same ? 0 : d1 - 2) * 64 + item) * BLK + lane * 16;
                st128(l0, rs0, o0); st128(l1, rs0, o0 + 1024); st128(nn, rs0, o0 + 2048);
                st128(l0, rs1, o1); st128(l1, rs1, o1 + 1024); st128(nn, rs1, o1 + 2048);
            } else {
                const u32 o = O_PK + pk * BLK + lane * 16;
                st128(l0, rsm, o); st128(l1, rsm, o + 1024); st128(nn, rsm, o + 2048);
            }
            hpay = lane ? k.hi : k.lo;
        }
        if (local) st_plain(D, rsm, O_IN + ls * 4096 + lane * 16);
        wt.seq = rfl(seqv) + 1;
        STAMP(61);
        WAIT_VM0();
        st_word_b(rsm, local && lane == 0, O_INFLAG + ls * FS, wt.seq);
        if (lane < 2) hx[wv * 2 + lane] = hpay;
        __syncthreads();
        if (wv == 3) {
            const u32 v = hx[lane & 7];
            st_gran_b(rs0, lane < 8, O_PKHDR + (r0 * 64 + (cb8 >> 2)) * 128 + lane * 8, v, wt.seq);
            st_gran_b(rs1, lane < 8, O_PKHDR + (r1 * 64 + (cb8 >> 2)) * 128 + lane * 8, v, wt.seq);
        }
        hx += 8;
        STAMP(1);
        wg = (wg >= 256 + NG2 && wg < 256 + NG2 + NIN) ? wg + GRID - NIN - 256 - NG2 : GRID;
        if (wg < GRID) ld_src(p, lane, wg * 4 + wv, F);
    }
    wt.seq = rfl(seqv) + 1;
    if (lane == 0) __hip_atomic_store(seqp, wt.seq, __ATOMIC_RELAXED, __HIP_MEMORY_SCOPE_AGENT);

    bool ok = true;
    float y[16 * KEEP];
    int nk = 0, kept = 0;

    const bool puller = bid >= NG1 && bid < 256;
    const int iw = ((bid & 255) - NG1) * 4 + wv, l = iw % 6, ip = iw / 6;
    if (bid >= B_IN ? ip < NPI : puller && ip < HEAD) {
        const int src = rank ^ (l + 2);
        char* const peer = p.rel[l + 2];
        const rsrc_t rs = mk_rsrc(puller ? my : peer, WS_USED);
        const char* const hb = my + O_PKHDR + l * 64 * 128 + (lane & 1) * 8;
        constexpr int D0 = HEAD + 2 * NPI, ND = 480 - D0;
        const u32 tbase = (wt.seq - 1) * (ND + NPI) - D0;
        int cntr = 2;
#define HOFF(b) (((b) & 252) * 32 + ((b) & 3) * 16 + (((b) >> 8) & 1) * (6 * 64 * 128))
#define PKIDX(b) (((2 * rm + (((b) >> 8) & 1)) * 2 + (((b) & 128) ? rj : ri)) * 256 + ((b) & 255))
        u32 i0 = puller ? ip : HEAD + ip;
        int b0 = item_at(i0, src);
        u32 tk = tbase + HEAD + NPI + ip, fo = OOB;
        u64 h0 = ld_gran(hb + HOFF(b0));
        WAIT_VM0();
        do {
            const u32 so = puller ? O_PKL + (l * 64 + i0) * BLK : O_PK + PKIDX(b0) * BLK;
            if (ok) ok = poll_gran(hb + HOFF(b0), true, h0, wt);
            Pk k;
            k.l0 = __builtin_amdgcn_raw_buffer_load_b128(rs, lane * 16, so, SYS);
            k.l1 = __builtin_amdgcn_raw_buffer_load_b128(rs, lane * 16 + 1024, so, SYS);
            k.n = __builtin_amdgcn_raw_buffer_load_b128(rs, lane * 16 + 2048, so, SYS);
            SB();
            STAMP(cntr); ++cntr;
            const u32 lo = (u32)__builtin_amdgcn_readlane((int)(u32)h0, 0), hi = (u32)__builtin_amdgcn_readlane((int)(u32)h0, 1);
            i0 = puller ? i0 + NCOMM * 4 / 6 : rfl(tk) - tbase;
            const int b1 = i0 < (puller ? HEAD : 480u) ? item_at(i0, src) : -1;
            h0 = ld_gran(hb + HOFF(b1));
            tk = __builtin_amdgcn_raw_ptr_buffer_atomic_add_i32(1, rsm, (b1 >= 0 && !puller && lane == 0) ? O_TICK + l * 128 : OOB, 0, 16);
            asm volatile("s_waitcnt vmcnt(5)" ::: "memory");
            st_word_b(rsm, ok, fo, wt.seq);
            const u32 slot = (4 + (b0 >> 8) * 6 + l) * SL + (b0 & 255), o = O_IN + slot * 4096 + lane * 16;
            fo = lane == 0 ? O_INFLAG + slot * FS : OOB;
            const u32x16 d = unpack(k, lo, hi, peer + O_PKOVF + PKIDX(b0) * 1024 + lane * 16, lane);
            asm volatile("" : "+v"(h0), "+v"(tk));
            st128(FRAG(d, 0), rsm, o); st128(FRAG(d, 1), rsm, o + 1024); st128(FRAG(d, 2), rsm, o + 2048); st128(FRAG(d, 3), rsm, o + 3072);
            SB();
            b0 = b1;
        } while (b0 >= 0);
        WAIT_VM0();
        st_word_b(rsm, ok, fo, wt.seq);
        STAMP(puller ? 19 : 30);
    }
    if (bid >= B_IN) { STAMP_DUMP(); return; }
    const int hi = bid < 256 ? bid - NG2 : bid - 256 + NG1 - NG2;
    if (!puller) {
        const bool mainT = bid < 256 + NG2;
        if (!mainT && bid < B_IN - 96) { STAMP_DUMP(); return; }
        const int xcd = bid & 7, k8 = (bid & 255) >> 3;
        __syncthreads();
        const u32 lds0 = (u32)(size_t)(__attribute__((address_space(3))) u32x4*)ring + wv * 4096;
        const u32x4 sd = {(u32)(size_t)my, (u32)((size_t)my >> 32) & 0xffffu, WS_USED, 0x00020000u};
        u32 xs0, ws0, ob, ho;
        int od;
        bool wfirst = false;
        if (mainT) {
            int a, b;
            shell_at(bid < 256 ? k8 : 31 - k8, a, b);
            const int cx = 4 * a + (xcd >> 1), n = 2 * b + (xcd & 1);
            xs0 = ((xcd >> 1) * 4 + a) * 8 + wv * 2;
            ws0 = (16 + (xcd & 1) * 8 + b) * 8 + wv;
            ob = (cx * 4 + wv) * 32 + 2 * n;
            ho = ((ri * 8 + (cx >> 1)) * 32 + rj * 16 + n) * 128 + rm * 32;
            od = ((cx & 1) * 4 + wv) ^ rank;
            wfirst = (cx >> 1) > (n >> 1);
        } else {
            const int e = k8 - (NG1 - 96) / 8, dq = (e >> 2) + 1, c = ((e >> 1) & 1) * 4 + wv, n = ((xcd >> 1) * 2 + (e & 1)) * 2 + (xcd & 1);
            xs0 = (dq & 1) ? 256 + c * 2 : sto(2 * c + rj) * 8 + (rank & 3) * 2;
            ws0 = ((dq & 2) ? 144 : 0) + sto(16 + n) * 8 + wv;
            const int rg = (ri ^ (dq & 1)) * 8 + c, n32 = (rj ^ (dq >> 1)) * 16 + n;
            ob = 2048 + rg * 64 + 2 * n32;
            ho = (rg * 32 + n32) * 128 + 64;
            od = 0;
        }
        f32x4 acc[2][8];
#pragma unroll
        for (int a = 0; a < 2; ++a)
#pragma unroll
            for (int f = 0; f < 8; ++f) acc[a][f] = (f32x4){0.f, 0.f, 0.f, 0.f};
        ok = gemm_tile(acc, ring, lds0, sd, xs0, ws0, mainT ? 8 : 2, od >= 2 ? od + 2 : -64, wfirst, my, wt, &red[4], &red[16], lane, stb);
        asm volatile("s_nop 15" ::: "memory");
        SB();
        u32 hpay = 0;
#pragma nounroll
        for (int h = 0; h < 2; ++h) {
            u32 D[16];
            float m = 0.f;
#pragma unroll
            for (int rf = 0; rf < 2; ++rf)
#pragma unroll
                for (int r = 0; r < 4; ++r) {
                    D[rf * 8 + r * 2] = cvt2(acc[rf][0][r], acc[rf][1][r]);
                    D[rf * 8 + r * 2 + 1] = cvt2(acc[rf][2][r], acc[rf][3][r]);
                    m = fmaxf(fmaxf(m, fabsf(acc[rf][0][r])), fabsf(acc[rf][1][r]));
                    m = fmaxf(fmaxf(m, fabsf(acc[rf][2][r])), fabsf(acc[rf][3][r]));
                }
            Packed k;
            pack(D, m, k, 4);
            if (__builtin_expect(k.lo & 0x80, 0)) st128((u32x4){k.X[0], k.X[1], k.X[2], k.X[3]}, rsm, O_OUTOVF + (ob + h) * 1024 + lane * 16);
#pragma unroll
            for (int t = 0; t < 4; ++t)
                st96((u32x3){k.L[2 * t], k.L[2 * t + 1], k.N[t]}, rsm, O_OUTDATA + (ob + h) * BLK + lane * 12 + t * 768);
            hpay = (lane >> 1) == h ? ((lane & 1) ? k.hi : k.lo) : hpay;
#pragma unroll
            for (int rf = 0; rf < 2; ++rf)
#pragma unroll
                for (int f = 0; f < 4; ++f) acc[rf][f] = acc[rf][4 + f];
        }
        const rsrc_t rso = mk_rsrc(p.rel[od & 7], WS_USED);
        ok = ok && red[4] == 0;
        WAIT_VM0();
        st_gran_b(rso, ok && lane < 4, O_OUTHDR + ho + lane * 8, hpay, wt.seq);
        STAMP(16);
        if (hi < 0 || hi >= HELD) { STAMP_DUMP(); return; }
    }

    u32 owed = OOB;
    int part = wv;
    int rgA = 0, cgA = 0;
    bool gate = false;
#pragma unroll
    for (int i = 0; i < 16 * KEEP; ++i) asm volatile("" : "=v"(y[i]));
#pragma nounroll
    for (int pass = 0; pass < 2; ++pass) {
    int bid = blockIdx.x, rank = p.rank & 7, wv = (int)rfl(threadIdx.x >> 6), lane = threadIdx.x & 63;
    asm volatile("" : "+s"(bid), "+s"(rank), "+s"(wv), "+v"(lane));
    const int rm = rank & 1, ri = (rank >> 1) & 1, rj = rank >> 2;
    if (puller) {
        const int pid = bid - NG1, dq = pid & 3, hf = wv & 1, pw = (pid >> 2) * 2 + (wv >> 1);
        rgA = (ri ^ (dq & 1)) * 8; cgA = (rj ^ (dq >> 1)) * 32;
        gate = dq == 0;
        char* amx = p.rel[lane & 7] + O_AMAX + (rank * NCOMM + pid) * AS;
        asm volatile("" : "+v"(amx));
        const u32 tko = O_TICK + (6 + dq * 2 + hf) * 128, tbase = (wt.seq - 1) * (256 - NP) - 2 * NP;
        const bool hact = lane < (dq ? 6 : 4);
        const char* const hbase = my + O_OUTHDR + (rgA * 32 + cgA / 2) * 128 + ((lane >> 1) & 3) * 32 + (lane & 1) * 8;
        const char* sp[3];
        sp[0] = p.rel[dq * 2 + rm]; sp[1] = p.rel[dq * 2 + (rm ^ 1)]; sp[2] = my;
#define OBLK(s, c, cg) ((s) < 2 ? (u32)((((c) * 2 + rj) * 4 + (rank & 3)) * 32 + (cg)) : (u32)(2048 + (rgA + (c)) * 64 + cgA + (cg)))
#define OHDR(c, cg) (((c) * 32 + ((cg) >> 1)) * 128 + ((cg) & 1) * 16)
        if (pass == 0) __syncthreads();
        float mx = 0.f;
        auto decode = [&](u32 t) -> int { int c, n; shell_at(t >> 1, c, n); return t * 256 + c * 32 + 2 * n + (t & 1); };
#define IC(i) (((i) >> 5) & 7)
#define IG(i) ((i) & 31)
        int ib = -1, ic = decode(pw), nkeep = 0;
        u32x3 nxt[6];
        f32x4 v[4];
        u32 b4[3], so[2], hcur = pass ? 0u : 0x10040100u, tk = tbase + NP + pw, fo = OOB;
        u64 fix = !pass, hn = 0;
#pragma unroll
        for (int s = 0; s < 6; ++s) { nxt[s] = (u32x3){0, 0, 0}; b4[s >> 1] = 0; }
        auto hand_over = [&]() -> u32 {
            const bool mem = ib >= 0 && ib < HELD * 256;
            const u32 o = mem ? O_Y + ((rgA + IC(ib)) * 64 + cgA + IG(ib)) * 8192 + hf * 4096 + lane * 16 : OOB;
            const u32x4 r0 = relu4(v[0]), r1 = relu4(v[1]), r2 = relu4(v[2]), r3 = relu4(v[3]);
            st128(r0, rsm, o); st128(r1, rsm, o + 1024); st128(r2, rsm, o + 2048); st128(r3, rsm, o + 3072);
            if (ib >= HELD * 256) {
                u32x4* const d = ring + wv * 1024 + nk * 256 + lane;
                d[0] = r0; d[64] = r1; d[128] = r2; d[192] = r3;
                kept = lane == nk ? ib : kept;
                ++nk;
            }
            return (mem && lane == 0) ? O_PROG + (dq * 256 + (ib >> 8)) * 128 + hf * 4 : OOB;
        };
        auto process = [&](const u32x3 (&cur)[6]) {
            u32 ha[6], hb[6];
#pragma unroll
            for (int s = 0; s < 6; ++s) dec_h(cur[s].z, b4[s >> 1], ha[s], hb[s]);
            if (__builtin_expect(fix != 0, 0)) {
#pragma unroll
                for (int s = 0; s < 3; ++s) {
                    const u32 lo = (u32)__builtin_amdgcn_readlane((int)hcur, 2 * s);
                    const u32 hi = (u32)__builtin_amdgcn_readlane((int)hcur, 2 * s + 1);
                    if (lo & 0x80) {
                        const char* pv = sp[s] + O_OUTOVF + OBLK(s, IC(ib), IG(ib)) * 1024 + lane * 16 + hf * 8;
                        u32x2 x;
                        asm volatile("global_load_dwordx2 %0, %1, off sc0 sc1\n\ts_waitcnt vmcnt(0)" : "=&v"(x) : "v"(pv) : "memory");
                        ha[2 * s] = cur[2 * s].z; hb[2 * s] = x.x; ha[2 * s + 1] = cur[2 * s + 1].z; hb[2 * s + 1] = x.y;
                    } else {
                        u64 pay = (((u64)hi << 32) | lo) >> 8;
                        while (pay != 0) {
                            const u32 e = (u32)pay & 0x3ffff, idx = (e >> 6) & 31, w = (idx >> 2) & 3;
                            pay >>= 18;
                            const u32 t = ((int)(idx >> 4) == hf && lane == (int)(e & 63)) ? (e >> 11) << (8 * (idx & 3)) : 0;
                            ha[2 * s] ^= w == 0 ? t : 0; hb[2 * s] ^= w == 1 ? t : 0;
                            ha[2 * s + 1] ^= w == 2 ? t : 0; hb[2 * s + 1] ^= w == 3 ? t : 0;
                        }
                    }
                }
            }
#pragma unroll
            for (int s = 0; s < 6; ++s) {
                f32x4 a, b;
#pragma unroll
                for (int j = 0; j < 4; ++j) {
                    const u32 sel = ((4u + j) << 24) | ((u32)j << 16) | 0x0c0cu;
                    a[j] = __builtin_bit_cast(float, __builtin_amdgcn_perm(ha[s], cur[s].x, sel));
                    b[j] = __builtin_bit_cast(float, __builtin_amdgcn_perm(hb[s], cur[s].y, sel));
                }
                v[(s & 1) * 2] = s >> 1 ? v[(s & 1) * 2] + a : a;
                v[(s & 1) * 2 + 1] = s >> 1 ? v[(s & 1) * 2 + 1] + b : b;
            }
#pragma unroll
            for (int j = 0; j < 4; ++j) mx = fmaxf(fmaxf(fmaxf(fmaxf(mx, v[j].x), v[j].y), v[j].z), v[j].w);
            asm volatile("" : "+v"(mx));
        };
        int cntr = 20;
        auto offsets = [&]() {
            so[0] = O_OUTDATA + OBLK(0, IC(ic), IG(ic)) * BLK + hf * 1536;
            so[1] = O_OUTDATA + OBLK(2, IC(ic), IG(ic)) * BLK + hf * 1536;
            asm volatile("" : "+s"(so[0]), "+s"(so[1]));
        };
        offsets();
        if (pass && dq) {
            const int l = 2 * dq - 2, b0 = item_at(480 - TRIG + pw, rank ^ (l + 2)), b1 = item_at(480 - TRIG + pw, rank ^ (l + 3));
            const int b = lane ? b1 : b0;
            if (ok) ok = poll_word(my + O_INFLAG + ((4 + (b >> 8) * 6 + l + (lane & 1)) * SL + (b & 255)) * FS, lane < 2, wt.seq, wt);
        }
        if (pass) do {
            asm volatile("" : "+v"(nxt[0]), "+v"(nxt[1]), "+v"(nxt[2]), "+v"(nxt[3]), "+v"(nxt[4]), "+v"(nxt[5]), "+v"(hn));
            u32x3 cur[6];
#pragma unroll
            for (int s = 0; s < 6; ++s) cur[s] = nxt[s];
            int id = -1;
            u32 hc;
            {
                if (ok) ok = poll_gran(hbase + OHDR(IC(ic), IG(ic)), hact, hn, wt);
#pragma unroll
                for (int s = 0; s < 6; ++s)
                    nxt[s] = __builtin_amdgcn_raw_buffer_load_b96(mk_rsrc(sp[s >> 1], (s >= 4 && !dq) ? 0u : WS_USED), lane * 12 + (s & 1) * 768,
                                                                  so[s >> 2], SYS);
                SB();
                hc = hact ? (u32)hn : 0u;
                asm volatile("" : "+v"(hc));
                const u32 t = rfl(tk) - tbase;
                if (nkeep < KEEP && t < 256u) {
                    id = decode(t);
                    nkeep += t >= HELD;
                }
                hn = ld_gran(hbase + OHDR(IC(id), IG(id)));
                tk = __builtin_amdgcn_raw_ptr_buffer_atomic_add_i32(1, rsm, (id >= 0 && nkeep < KEEP && lane == 0) ? tko : OOB, 0, 16);
            }
            STAMP(cntr); ++cntr;
            process(cur);
            {
                const u32 fn = hand_over();
                asm volatile("s_waitcnt vmcnt(12)" ::: "memory");
                st_word_b(rsm, ok, fo, wt.seq);
                fo = fn;
                ib = ic; ic = id;
                hcur = hc;
                const u32 bv = (hc & 0x7f) * 0x01010101u;
#pragma unroll
                for (int s = 0; s < 3; ++s) b4[s] = (u32)__builtin_amdgcn_readlane((int)bv, 2 * s);
                const u32 oth = (u32)__builtin_amdgcn_update_dpp(0, (int)hc, 0xB1, 0xf, 0xf, false);
                const u64 pay = ((u64)oth << 32) | hc;
                bool need = (hc & 0x80) != 0;
#pragma unroll
                for (int z = 0; z < 3; ++z) {
                    const u32 e = (u32)(pay >> (8 + 18 * z));
                    need |= ((e & 0x3f800) != 0) & ((int)((e >> 10) & 1) == hf);
                }
                fix = __ballot(need & !(lane & 1));
                offsets();
            }
        } while (ic >= 0);
        WAIT_VM0();
        STAMP(cntr);
        process(nxt);
        const u32 wm = ok ? wave_umax(__builtin_bit_cast(u32, mx)) : 0xffffffffu;
        u32 arrived, m;
        {
            const u32 la = (u32)(size_t)(__attribute__((address_space(3))) u32*)&red[2 * pass];
            asm volatile("s_mov_b64 exec, 1\n\tds_max_u32 %2, %3\n\tds_add_rtn_u32 %0, %2, %4 offset:4\n\tds_read_b32 %1, %2\n\t"
                         "s_mov_b64 exec, -1\n\ts_waitcnt lgkmcnt(0)" : "=&v"(arrived), "=&v"(m) : "v"(la), "v"(wm), "v"(1u) : "memory");
        }
        m = rfl(m);
        part = rfl(arrived);
        if (part == 3 && lane < 8 * pass && m != 0xffffffffu) st_gran(amx, m, wt.seq);
        STAMP(56);
        if (part == 3) __builtin_amdgcn_s_barrier();
        if (nkeep == KEEP && lane == 0) owed = tko;
        st_word_b(rsm, ok, fo, wt.seq);
        fo = hand_over();
        if (ib < HELD * 256) {
            WAIT_VM0();
            st_word_b(rsm, ok, fo, wt.seq);
        }
#pragma unroll
        for (int j = 0; j < 4 * KEEP; ++j) {
            const f32x4 a = __builtin_bit_cast(f32x4, ring[wv * 1024 + j * 64 + lane]);
            y[4 * j] = a.x; y[4 * j + 1] = a.y; y[4 * j + 2] = a.z; y[4 * j + 3] = a.w;
        }
        if (part != 3) __builtin_amdgcn_s_barrier();
    } else if (pass) {
        rgA = (ri ^ (wv & 1)) * 8; cgA = (rj ^ (wv >> 1)) * 32;
        gate = hi < HELD - 2 * NP;
        { int c, n; shell_at(hi >> 1, c, n); rgA += c; cgA += 2 * n + (hi & 1); }
        if (ok) ok = poll_word(my + O_PROG + (wv * 256 + hi) * 128 + (lane & 1) * 4, lane < 2, wt.seq, wt);
        const u32 so = O_Y + (rgA * 64 + cgA) * 8192;
#pragma unroll
        for (int u = 0; u < 8; ++u) {
            const f32x4 a = __builtin_bit_cast(f32x4, __builtin_amdgcn_raw_buffer_load_b128(rsm, lane * 16, so + u * 1024, SYS));
            y[4 * u] = a.x; y[4 * u + 1] = a.y; y[4 * u + 2] = a.z; y[4 * u + 3] = a.w;
        }
        WAIT_VM0();
        STAMP(17);
    }

    {
        u32 oo[KEEP];
        const int np = !pass ? KEEP : puller ? nk : 2;
#pragma unroll
        for (int r = 0; r < KEEP; ++r) {
            const int it = __builtin_amdgcn_readlane(kept, r);
            oo[r] = ((rgA + IC(it)) * 32 + (puller ? wv & 1 : r) * 16 + (lane & 15)) * 4096 + (cgA + IG(it)) * 64 + (lane >> 4) * 16;
            oo[r] = r < np ? oo[r] : OOB;
            asm volatile("" : "+v"(oo[r]));
        }
        const rsrc_t ro = mk_rsrc(p.out, pass ? 512u * 4096u : 0u);
        const char* a0 = my + O_AMAX + (part * 128 + lane) * AS;
        u64 v0 = 0, v1 = 0;
        bool d0 = part * 128 + lane >= NR * NCOMM, d1 = part * 128 + lane + 64 >= NR * NCOMM;
        if (pass && gate && ok && part < 3) ok = poll_word(my + O_PROG + ((part + 1) * 256 + HELD - 1) * 128, lane == 0, wt.seq, wt);
        while (pass && ok && part < 3) {
            if (!d0) v0 = ld_gran(a0);
            if (!d1) v1 = ld_gran(a0 + 64 * AS);
            d0 = d0 || (u32)(v0 >> 32) >= wt.seq;
            d1 = d1 || (u32)(v1 >> 32) >= wt.seq;
            if (__all(d0 && d1)) break;
            if (expired(wt)) ok = false;
            __builtin_amdgcn_s_sleep(1);
        }
        const u32 wm = wave_umax(umax((u32)v0, (u32)v1));
        if (lane == 0) red[8 + pass * 4 + wv] = ok ? wm : 0xffffffffu;
        __syncthreads();
        const u32x4 r4 = *(const u32x4*)&red[8 + pass * 4];
        const float am = __builtin_bit_cast(float, umax(umax(r4.x, r4.y), umax(r4.z, r4.w)));
        const float inv = am == 0.f ? 0.f : 448.0f * __builtin_amdgcn_rcpf(am);
        STAMP(57);
#pragma unroll
        for (int r = 0; r < KEEP; ++r) {
            if (r == 2 && np <= 2) break;
            u32x4 o;
#pragma unroll
            for (int j = 0; j < 4; ++j)
                o[j] = q4(y[16 * r + 4 * j], y[16 * r + 4 * j + 1], y[16 * r + 4 * j + 2], y[16 * r + 4 * j + 3], inv);
            __builtin_amdgcn_raw_buffer_store_b128(o, ro, oo[r], 0, OUT_AUX);
        }
        if (pass) __builtin_amdgcn_raw_ptr_buffer_atomic_add_i32(1, rsm, owed, 0, 16);
    }
    }
    STAMP(58);
    STAMPC(59);
    STAMP_DUMP();
}

extern "C" void kernel_launch(int rank, int nranks, ncclComm_t comm, hipStream_t stream,
                              void* const* d_in, const int* in_sizes, int n_in,
                              void* d_out, int out_size,
                              const int* in_dtypes, int out_dtype,
                              void* const* d_out_all,
                              void* const* d_ws_all, size_t ws_bytes) {
    (void)comm; (void)in_dtypes; (void)out_dtype; (void)d_out_all;
    if (nranks != NR || n_in != 2 || (unsigned)rank >= (unsigned)NR || ws_bytes < WS_USED ||
        in_sizes[0] != 4096 * 512 || in_sizes[1] != 512 * 4096 || out_size != 512 * 4096) {
        fprintf(stderr, "kernel_launch: unexpected problem shape\n");
        return;
    }
    Params p;
    p.x = (const float*)d_in[0];
    p.w = (const float*)d_in[1];
    p.out = (unsigned char*)d_out;
    p.my = (char*)d_ws_all[rank];
    for (int r = 0; r < NR; ++r) p.rel[r] = (char*)d_ws_all[r ^ rank];
    p.rank = rank;
    p.pad = 0;
    hipLaunchKernelGGL(fused, dim3(GRID), dim3(256), 0, stream, p);
    HIP_CHECK(hipGetLastError());
}
```

```cpp
#include <hip/hip_runtime.h>
#include <rccl/rccl.h>
#include <cstdio>

#define HIP_CHECK(expr) do { hipError_t _e = (expr); if (_e != hipSuccess) { \
    fprintf(stderr, "HIP error %s at %s:%d\n", hipGetErrorString(_e), __FILE__, __LINE__); \
    return; } } while (0)

typedef unsigned u32;
typedef unsigned long long u64;
typedef u32 u32x2 __attribute__((ext_vector_type(2)));
typedef u32 u32x3 __attribute__((ext_vector_type(3)));
typedef u32 u32x4 __attribute__((ext_vector_type(4)));
typedef u32 u32x8 __attribute__((ext_vector_type(8)));
typedef u32 u32x16 __attribute__((ext_vector_type(16)));
typedef float f32x2 __attribute__((ext_vector_type(2)));
typedef float f32x4 __attribute__((ext_vector_type(4)));
typedef __bf16 bf16x2 __attribute__((ext_vector_type(2)));
typedef __bf16 bf16x8 __attribute__((ext_vector_type(8)));
typedef __attribute__((address_space(1))) u32 gu32;
typedef __attribute__((address_space(1))) u64 gu64;
typedef __amdgpu_buffer_rsrc_t rsrc_t;

constexpr int HEAD = 36;
constexpr int HW = 40;
constexpr int STAGP = 11, STAG = 3;
constexpr u64 ORDER = 0x0128394a5b6c7defull;
constexpr int NPI = 26;
constexpr int NP = 24;
constexpr int TRIG = 64;
constexpr int NKEPT = 64;
#define TEST_BASE_SHIFT 0

constexpr int NR = 8;
constexpr int NCOMM = 2 * NP;
constexpr int GRID = 512;
constexpr int NG1 = 256 - NCOMM;
constexpr int NG2 = 256 - NG1;
constexpr int B_IN = 256 + NG1;
constexpr int NIN = (6 * NPI + 3) / 4;
constexpr int KEEP = 4;
constexpr int HELD = 256 - NKEPT;
static_assert(HELD <= NG1 && NG1 % 8 == 0 && NG2 + NIN + 96 <= NG1 && B_IN + NIN <= GRID && NKEPT <= NP * KEEP && 2 * NP <= HELD && HEAD <= 64, "roles");
constexpr u64 DEADLINE = 50000000ull;

constexpr u32 FS = 128, AS = 128;
constexpr u32 BLK = 3072;
constexpr u32 SL = 50 * 8;
constexpr u32 N_IN = 16 * SL, N_PK = 2048, N_OUT = 3072;
constexpr u32 O_SEQ = 0;
constexpr u32 O_TMO = O_SEQ + GRID * 4 * 4;
constexpr u32 O_TICK = O_TMO + 128;
constexpr u32 O_AMAX = O_TMO + 2048;
constexpr u32 O_PROG = O_AMAX + 512 * 128;
constexpr u32 O_PKHDR = O_PROG + 1024 * 128;
constexpr u32 O_OUTHDR = O_PKHDR + 12 * 64 * 128;
constexpr u32 O_INFLAG = O_OUTHDR + 512 * 128;
constexpr u32 O_IN = ((O_INFLAG + N_IN * FS + 4095) / 4096) * 4096;
constexpr u32 O_PK = O_IN + N_IN * 4096;
constexpr u32 O_PKL = O_PK + N_PK * BLK;
constexpr u32 O_OUTDATA = O_PKL + 6 * 64 * BLK;
constexpr u32 O_Y = O_OUTDATA + N_OUT * BLK;
constexpr u32 O_PKOVF = O_Y + 1024 * 8192;
constexpr u32 O_OUTOVF = O_PKOVF + N_PK * 1024;
constexpr u32 WS_USED = O_OUTOVF + N_OUT * 1024;
constexpr u32 OOB = 0xfffff000u;
static_assert(O_AMAX % 128 == 0 && O_PROG % 128 == 0 && O_INFLAG % 128 == 0 && O_PKHDR % 128 == 0 && O_OUTHDR % 128 == 0, "align");
static_assert(NR * NCOMM <= 512 && GRID * 4 == 2048, "shape");

constexpr int SYS = 17, NT = 2;

struct Params {
    const float* x;
    const float* w;
    unsigned char* out;
    char* my;
    int rank;
    int pad;
    char* rel[NR];
};

#define DEV __device__ __forceinline__
#define CFENCE() asm volatile("" ::: "memory")
#define WAIT_VM0() do { __builtin_amdgcn_s_waitcnt(0x0F70); asm volatile("s_waitcnt vmcnt(0)" ::: "memory"); } while (0)

DEV rsrc_t mk_rsrc(const void* base, u32 bytes) { return __builtin_amdgcn_make_buffer_rsrc((void*)base, 0, bytes, 0x00020000); }
DEV u32 rfl(u32 v) { return __builtin_amdgcn_readfirstlane(v); }
DEV u32 cvt2(float a, float b) { f32x2 v = {a, b}; return __builtin_bit_cast(u32, __builtin_convertvector(v, bf16x2)); }
DEV u32 umax(u32 a, u32 b) { return a > b ? a : b; }
DEV u32 wave_umax(u32 v) {
    v = umax(v, (u32)__builtin_amdgcn_update_dpp(0, (int)v, 0x111, 0xf, 0xf, true));
    v = umax(v, (u32)__builtin_amdgcn_update_dpp(0, (int)v, 0x112, 0xf, 0xf, true));
    v = umax(v, (u32)__builtin_amdgcn_update_dpp(0, (int)v, 0x114, 0xf, 0xf, true));
    v = umax(v, (u32)__builtin_amdgcn_update_dpp(0, (int)v, 0x118, 0xf, 0xf, true));
    v = umax(v, (u32)__builtin_amdgcn_update_dpp(0, (int)v, 0x142, 0xa, 0xf, false));
    v = umax(v, (u32)__builtin_amdgcn_update_dpp(0, (int)v, 0x143, 0xc, 0xf, false));
    return (u32)__builtin_amdgcn_readlane((int)v, 63);
}

struct Wait {
    u32 seq;
    u64 t0;
    gu32* tmo;
};
DEV bool expired(const Wait& wt) {
    const u64 dt = __builtin_amdgcn_s_memrealtime() - wt.t0;
    if (dt < 100000ull) return false;
    if (dt > DEADLINE || rfl(__hip_atomic_load(wt.tmo, __ATOMIC_RELAXED, __HIP_MEMORY_SCOPE_AGENT)) == wt.seq) {
        __hip_atomic_store(wt.tmo, wt.seq, __ATOMIC_RELAXED, __HIP_MEMORY_SCOPE_AGENT);
        return true;
    }
    return false;
}
DEV u64 ld_gran(const char* p) { return __hip_atomic_load((gu64*)p, __ATOMIC_RELAXED, __HIP_MEMORY_SCOPE_SYSTEM); }
DEV void st_gran(char* p, u32 payload, u32 seq) {
    __hip_atomic_store((gu64*)p, ((u64)seq << 32) | payload, __ATOMIC_RELAXED, __HIP_MEMORY_SCOPE_SYSTEM);
}
DEV bool poll_gran(const char* p, bool act, u64& v, const Wait& wt) {
    bool ok = true;
    bool done = !act || (u32)(v >> 32) >= wt.seq;
    if (__builtin_expect(!__all(done), 0)) {
        for (;;) {
            if (!done) { v = ld_gran(p); done = (u32)(v >> 32) >= wt.seq; }
            if (__all(done)) break;
            if (expired(wt)) { ok = false; break; }
            __builtin_amdgcn_s_sleep(4);
        }
    }
    CFENCE();
    return ok;
}
DEV bool poll_word(const char* p, bool act, u32 need, const Wait& wt) {
    bool ok = true, done = !act;
    for (;;) {
        if (!done) done = __hip_atomic_load((gu32*)p, __ATOMIC_RELAXED, __HIP_MEMORY_SCOPE_SYSTEM) >= need;
        if (__all(done)) break;
        if (expired(wt)) { ok = false; break; }
        __builtin_amdgcn_s_sleep(4);
    }
    CFENCE();
    return ok;
}

DEV bool poll_part(const char* p, bool& done, u64 want, const Wait& wt) {
    bool ok = true;
    if ((__ballot(done) & want) != want) {
        for (;;) {
            if (!done) done = __hip_atomic_load((gu32*)p, __ATOMIC_RELAXED, __HIP_MEMORY_SCOPE_SYSTEM) >= wt.seq;
            if ((__ballot(done) & want) == want) break;
            if (expired(wt)) { ok = false; break; }
            __builtin_amdgcn_s_sleep(1);
        }
    }
    CFENCE();
    return ok;
}

struct Packed {
    u32 L[8], N[4], X[4];
    u32 lo, hi;
};
DEV void pack(const u32 (&D)[16], float amax, Packed& o, int kind) {
    const u32 top = wave_umax(__builtin_bit_cast(u32, amax)) >> 24;
    int b = (int)(top & 0x7f) - 7 - ((TEST_BASE_SHIFT & kind) ? 1 : 0);
    b = b < 0 ? 0 : b;
    const u32 base4 = (u32)b * 0x01010101u;
    u32 H[8], d[8], chk = 0;
#pragma unroll
    for (int q = 0; q < 8; ++q) {
        o.L[q] = __builtin_amdgcn_perm(D[2 * q + 1], D[2 * q], 0x06040200u);
        H[q] = __builtin_amdgcn_perm(D[2 * q + 1], D[2 * q], 0x07050301u);
        d[q] = H[q] - base4;
        chk |= d[q];
    }
    u64 pay = (u64)b;
    u64 m = __ballot((chk & 0x78787878u) != 0);
    if (__builtin_expect(m != 0, 0)) {
        u32x8 dl;
        u32 esc = 0;
#pragma unroll
        for (int q = 0; q < 8; ++q) {
            d[q] = (((H[q] | 0x80808080u) - base4) & 0x07070707u) | (H[q] & 0x80808080u);
            dl[q] = (d[q] + base4) ^ H[q];
            esc |= ((dl[q] + 0x7f7f7f7fu) & 0x80808080u) >> (7 - q);
        }
        int cnt = 0;
        while (m != 0 && cnt <= 3) {
            const int l = __builtin_ctzll(m);
            m &= m - 1;
            u32 bits = (u32)__builtin_amdgcn_readlane((int)esc, l);
            while (bits != 0 && cnt <= 3) {
                const int pos = __builtin_ctz(bits);
                bits &= bits - 1;
                const u32 e = ((u32)__builtin_amdgcn_readlane((int)dl[pos & 7], l) >> (pos & 24)) & 0x7f;
                pay |= (u64)((u32)l | ((u32)(4 * (pos & 7) + (pos >> 3)) << 6) | (e << 11)) << (8 + 18 * cnt);
                ++cnt;
            }
        }
        if (cnt > 3) {
            pay = 0x80;
#pragma unroll
            for (int t = 0; t < 4; ++t) { d[2 * t] = H[2 * t]; d[2 * t + 1] = 0; }
        }
    }
#pragma unroll
    for (int t = 0; t < 4; ++t) {
        o.N[t] = d[2 * t] | __builtin_amdgcn_alignbit(d[2 * t + 1], d[2 * t + 1], 28);
        o.X[t] = H[2 * t + 1];
    }
    o.lo = (u32)pay;
    o.hi = (u32)(pay >> 32);
}
DEV void dec_h(u32 n, u32 base4, u32& ha, u32& hb) {
    ha = (n & 0x87878787u) + base4;
    hb = (__builtin_amdgcn_alignbit(n, n, 4) & 0x87878787u) + base4;
}
DEV u32x4 mk_frag(u32 la, u32 lb, u32 ha, u32 hb) {
    u32x4 r;
    r.x = __builtin_amdgcn_perm(ha, la, 0x05010400u);
    r.y = __builtin_amdgcn_perm(ha, la, 0x07030602u);
    r.z = __builtin_amdgcn_perm(hb, lb, 0x05010400u);
    r.w = __builtin_amdgcn_perm(hb, lb, 0x07030602u);
    return r;
}
struct Pk { u32x4 l0, l1, n; };
DEV u32x16 cat4(u32x4 a, u32x4 b, u32x4 c, u32x4 d) {
    const u32x8 l = __builtin_shufflevector(a, b, 0, 1, 2, 3, 4, 5, 6, 7), h = __builtin_shufflevector(c, d, 0, 1, 2, 3, 4, 5, 6, 7);
    return __builtin_shufflevector(l, h, 0, 1, 2, 3, 4, 5, 6, 7, 8, 9, 10, 11, 12, 13, 14, 15);
}
#define FRAG(v, t) __builtin_shufflevector(v, v, 4 * (t), 4 * (t) + 1, 4 * (t) + 2, 4 * (t) + 3)
DEV u32x16 unpack(const Pk& k, u32 lo, u32 hi, const char* ovf, int lane) {
    const u32 base4 = (lo & 0x7f) * 0x01010101u;
    u32x4 ha, hb;
#pragma unroll
    for (int t = 0; t < 4; ++t) { u32 a, b; dec_h(k.n[t], base4, a, b); ha[t] = a; hb[t] = b; }
    if (__builtin_expect(((lo >> 7) | hi) != 0, 0)) {
        if (lo & 0x80) {
            ha = k.n;
            asm volatile("global_load_dwordx4 %0, %1, off sc0 sc1\n\ts_waitcnt vmcnt(0)" : "=&v"(hb) : "v"(ovf) : "memory");
        } else {
            u64 pay = (((u64)hi << 32) | lo) >> 8;
            while (pay != 0) {
                const u32 e = (u32)pay & 0x3ffff, idx = (e >> 6) & 31;
                pay >>= 18;
                const u32 t = lane == (int)(e & 63) ? (e >> 11) << (8 * (idx & 3)) : 0;
#pragma unroll
                for (u32 z = 0; z < 4; ++z) {
                    ha[z] ^= (idx >> 2) == 2 * z ? t : 0;
                    hb[z] ^= (idx >> 2) == 2 * z + 1 ? t : 0;
                }
            }
        }
    }
    return cat4(mk_frag(k.l0.x, k.l0.y, ha.x, hb.x), mk_frag(k.l0.z, k.l0.w, ha.y, hb.y),
                mk_frag(k.l1.x, k.l1.y, ha.z, hb.z), mk_frag(k.l1.z, k.l1.w, ha.w, hb.w));
}
DEV void st128(u32x4 v, rsrc_t rs, u32 off) { __builtin_amdgcn_raw_buffer_store_b128(v, rs, off, 0, SYS); }
DEV void st96(u32x3 v, rsrc_t rs, u32 off) { __builtin_amdgcn_raw_buffer_store_b96(v, rs, off, 0, SYS); }
DEV void st_gran_b(rsrc_t rs, bool act, u32 off, u32 payload, u32 seq) {
    const u32x2 v = {payload, seq};
    __builtin_amdgcn_raw_buffer_store_b64(v, rs, act ? off : OOB, 0, SYS);
}

DEV void st_word_b(rsrc_t rs, bool act, u32 off, u32 v) { __builtin_amdgcn_raw_buffer_store_b32(v, rs, act ? off : OOB, 0, SYS); }

DEV int shell_of(int e) { return (int)(ORDER >> (60 - 4 * e)) & 15; }
DEV int sto(int ch) { return ch < 16 ? (ch & 3) * 4 + (ch >> 2) : 16 + (ch & 1) * 8 + ((ch - 16) >> 1); }
DEV int item_at(int i, int src) {
    const int e = i / 30, ps = shell_of(e);
    int r = i - e * 30;
    if ((r >> 1) >= src) r += 2;
    return (ps >> 3) * 256 + sto((r >> 4) * 16 + (ps & 7) * 2 + ((r >> 3) & 1)) * 8 + (r & 7);
}
struct Cv { int pos, dm, cls, blk, ch, pass, ks; };
DEV Cv cv_of(int u) {
    Cv c;
    c.pos = u >> 5;
    u += c.pos * 8;
    c.dm = (u >> 4) & 1; c.cls = (u >> 3) & 1; c.blk = u & 7;
    const int ps = shell_of(c.pos >> 2);
    c.ch = (c.pos & 2) * 8 + (ps & 7) * 2 + (c.pos & 1); c.pass = ps >> 3; c.ks = 2 * c.dm + c.pass;
    return c;
}
DEV void ld_src(const Params& p, int lane, int u, f32x4 (&F)[8]) {
    const Cv c = cv_of(u);
    const int fr = lane & 15, fq = lane >> 4;
    const bool isx = c.ch < 16;
    const int owner = (c.ch & 1) * 4 + (c.blk >> 1);
    u32 so = isx ? ((owner * 512 + (c.cls * 8 + (c.ch >> 1)) * 32) * 512 + c.ks * 128 + (c.blk & 1) * 64) * 4
                 : ((c.ks * 128 + (c.blk >> 1) * 32) * 4096 + c.cls * 2048 + (c.ch - 16) * 128 + (c.blk & 1) * 64) * 4;
    u32 s2 = isx ? 16 * 2048 : 16 * 16384, s1 = isx ? 128 : 2 * 16384, s0 = isx ? 64 : 16384;
    u32 vo = isx ? fr * 2048 + fq * 16 : fq * 4 * 16384 + fr * 16;
    asm volatile("" : "+s"(so), "+s"(s2), "+s"(s1), "+s"(s0), "+v"(vo));
    __builtin_amdgcn_sched_barrier(0);
    const rsrc_t rs = mk_rsrc(isx ? p.x : p.w, 4096u * 512u * 4u);
#pragma unroll
    for (int i = 0; i < 8; ++i)
        F[i] = __builtin_bit_cast(f32x4, __builtin_amdgcn_raw_buffer_load_b128(
                   rs, vo, so + (i >> 2) * s2 + ((i >> 1) & 1) * s1 + (i & 1) * s0, NT));
    __builtin_amdgcn_sched_barrier(0);
}
DEV float cvt_src(const f32x4 (&F)[8], bool isx, u32 (&D)[16]) {
    if (isx) {
#pragma unroll
        for (int u = 0; u < 8; ++u) { D[2 * u] = cvt2(F[u].x, F[u].y); D[2 * u + 1] = cvt2(F[u].z, F[u].w); }
    } else {
#pragma unroll
        for (int f = 0; f < 4; ++f)
#pragma unroll
            for (int e = 0; e < 4; ++e) D[4 * f + e] = cvt2(F[2 * e][f], F[2 * e + 1][f]);
    }
    float m = 0.f;
#pragma unroll
    for (int u = 0; u < 8; ++u) {
        m = fmaxf(fmaxf(m, fabsf(F[u].x)), fabsf(F[u].y));
        m = fmaxf(fmaxf(m, fabsf(F[u].z)), fabsf(F[u].w));
    }
    return m;
}
DEV void st_plain(const u32 (&D)[16], rsrc_t rs, u32 off) {
#pragma unroll
    for (int t = 0; t < 4; ++t) st128((u32x4){D[4 * t], D[4 * t + 1], D[4 * t + 2], D[4 * t + 3]}, rs, off + t * 1024);
}
DEV void shell_at(int t, int& a, int& b) {
    int s = 0;
    while (2 * (s + 1) * (s + 1) <= t) ++s;
    int r = t - 2 * s * s;
    if (r < 2 * s) { a = s; b = r; }
    else { r -= 2 * s; const int up = r > s; a = r - up * (s + 1); b = 2 * s + up; }
}
DEV u32x4 relu4(f32x4 v) {
    typedef int i32x4 __attribute__((ext_vector_type(4)));
    return __builtin_bit_cast(u32x4, __builtin_elementwise_max(__builtin_bit_cast(i32x4, v), (i32x4){0, 0, 0, 0}));
}
DEV u32 q4(float a, float b, float c, float d, float inv) {
    const f32x2 lo = (f32x2){a, b} * inv, hi = (f32x2){c, d} * inv;
    int r = __builtin_amdgcn_cvt_pk_fp8_f32(lo.x, lo.y, __builtin_bit_cast(int, lo.x), false);
    r = __builtin_amdgcn_cvt_pk_fp8_f32(hi.x, hi.y, r, true);
    return (u32)r;
}

#define XSLOT(s) (xs0 + ((s) >> 1) * SL + ((s) & 1))
#define WSLOT(s) (ws0 + ((s) >> 1) * SL + ((s) & 1) * 4)
#define SB() __builtin_amdgcn_sched_barrier(0)
template <int F>
DEV void ld_x(u32x4& d, u32x4 sd, u32 vo, u32 so) {
    asm volatile("buffer_load_dwordx4 %0, %1, %2, %3 offen offset:%4 sc0 sc1" : "+v"(d) : "v"(vo), "s"(sd), "s"(so), "i"(F * 1024));
}
template <int N>
DEV void wait_step(u32x4 (&x)[4]) {
    __builtin_amdgcn_s_waitcnt(0xC07F);
    asm volatile("s_waitcnt vmcnt(%4) lgkmcnt(0)\n\ts_barrier" : "+v"(x[0]), "+v"(x[1]), "+v"(x[2]), "+v"(x[3]) : "i"(N) : "memory");
}
DEV void mma(f32x4& c, u32x4 a, u32x4 b) { asm volatile("v_mfma_f32_16x16x32_bf16 %0, %1, %2, %0" : "+v"(c) : "v"(a), "v"(b)); }
template <int F>
DEV void mma_ldw(f32x4& c, u32x4 a, u32x4 b, u32 lds, u32x4 sd, u32 vo, u32 so) {
    asm volatile("s_mov_b32 m0, %3\n\tv_mfma_f32_16x16x32_bf16 %0, %1, %2, %0\n\tbuffer_load_dwordx4 %4, %5, %6 offen offset:%7 sc0 sc1 lds"
                 : "+v"(c) : "v"(a), "v"(b), "s"(lds), "v"(vo), "s"(sd), "s"(so), "i"(F * 1024) : "memory");
}
template <int S, int G>
DEV bool gemm_steps(f32x4 (&acc)[2][8], u32x4 (&X)[4][4], u32x4 (&a)[8], const u32x4* ring, u32 lds0, u32x4 sd, u32 xs0, u32 ws0,
                    int skip, int lane, u32 fo, u32& fv, u32 seq, u32* vote) {
    const u32 vo = lane * 16;
    u32x4 b[8];
    const int sl = (S + 3) >> 1;
    const u32 vx = (sl == skip || sl == skip + 6) ? 0x80000000u : vo;
    const u32 sx = O_IN + XSLOT(S + 3) * 4096, sw = O_IN + WSLOT(S + 4) * 4096, l = lds0 + S * 16384;
    constexpr bool LX = G < 2 || S == 0, LW = G < 2, END = G == 2 && S == 3;
    u32 vt = 0;
    if (G < 2 && S == 0) asm volatile("buffer_load_dword %0, %1, %2, 0 offen sc0 sc1" : "+v"(fv) : "v"(fo), "s"(sd));
#pragma unroll
    for (int i = 0; i < 8; ++i) {
        mma(acc[0][i], a[i], X[S][0]);
        SB();
        if (i < 6) b[i < 4 ? i + (i + 1) / 2 : i + 2] = ring[S * 1024 + (8 + (i < 4 ? i + (i + 1) / 2 : i + 2)) * 64 + lane];
        if (G < 2 && S == 3 && i == 6) vote[(fo != OOB && fv < seq) ? 0 : 8] = 1;
        SB();
        mma(acc[1][i], a[i], X[S][2]);
        SB();
        if (i == 0 || i == 2) b[i + i / 2 + 1] = ring[S * 1024 + (8 + i + i / 2 + 1) * 64 + lane];
        if (LX && i == 1) ld_x<0>(X[(S + 3) & 3][0], sd, vx, sx);
        if (LX && i == 3) ld_x<1>(X[(S + 3) & 3][1], sd, vx, sx);
        if (LX && i == 4) ld_x<2>(X[(S + 3) & 3][2], sd, vx, sx);
        if (LX && i == 6) ld_x<3>(X[(S + 3) & 3][3], sd, vx, sx);
        SB();
    }
    if (!END) wait_step<(G == 0 && S == 0 ? 8 : G == 0 && S == 1 ? 12 : G < 2 || S == 0 ? 16 : S == 1 ? 8 : 0)>(X[(S + 1) & 3]);
    if (G < 2 && S == 2) asm volatile("" : "+v"(fv));
    if (G < 2 && S == 3) vt = *vote;
    SB();
#pragma unroll
    for (int i = 0; i < 8; ++i) {
        mma(acc[0][i], b[i], X[S][1]);
        SB();
        if (!END) a[i] = ring[((S + 1) & 3) * 1024 + i * 64 + lane];
        SB();
        if (LW && i == 0) mma_ldw<0>(acc[1][i], b[i], X[S][3], l, sd, vo, sw);
        else if (LW && i == 2) mma_ldw<1>(acc[1][i], b[i], X[S][3], l, sd, vo, sw);
        else if (LW && i == 4) mma_ldw<2>(acc[1][i], b[i], X[S][3], l, sd, vo, sw);
        else if (LW && i == 6) mma_ldw<3>(acc[1][i], b[i], X[S][3], l, sd, vo, sw);
        else mma(acc[1][i], b[i], X[S][3]);
        SB();
    }
    if constexpr (S < 3) return gemm_steps<S + 1, G>(acc, X, a, ring, lds0, sd, xs0, ws0, skip, lane, fo, fv, seq, vote);
    else return rfl(vt) == 0;
}
DEV void first_w(u32 lds0, u32x4 sd, u32 ws0, int lane) {
#pragma unroll
    for (int s = 0; s < 4; ++s) {
        const u32 sw = O_IN + WSLOT(s) * 4096, l = lds0 + s * 16384, vo = lane * 16;
        asm volatile("s_mov_b32 m0, %0\n\ts_nop 0\n\tbuffer_load_dwordx4 %1, %2, %3 offen sc0 sc1 lds\n\t"
                     "buffer_load_dwordx4 %1, %2, %3 offen offset:1024 sc0 sc1 lds\n\tbuffer_load_dwordx4 %1, %2, %3 offen offset:2048 sc0 sc1 lds\n\t"
                     "buffer_load_dwordx4 %1, %2, %3 offen offset:3072 sc0 sc1 lds" :: "s"(l), "v"(vo), "s"(sd), "s"(sw) : "memory");
    }
}
DEV bool gemm_tile(f32x4 (&acc)[2][8], const u32x4* ring, u32 lds0, u32x4 sd, u32 xs0, u32 ws0, int gend, int skip, bool wfirst,
                   const char* my, const Wait& wt, u32* fail, u32* votes, int lane) {
    u32x4 X[4][4], a[8];
#pragma unroll
    for (int i = 0; i < 16; ++i) X[i >> 2][i & 3] = (u32x4){0, 0, 0, 0};
    const u32 f0 = O_INFLAG + ((lane & 16) ? WSLOT(lane & 15) : XSLOT(lane & 15)) * FS;
    const bool none = lane >= 32 || (lane & 15) >= 12 || (lane >> 1) == skip - 4;
    bool ok = true;
    u32 fv = 0;
    int g = 0;
    do {
        bool done = g ? none : lane >= 32 || (lane & 15) >= 8;
        const char* const fp = my + f0 + g * (2 * SL * FS);
        asm volatile("s_waitcnt lgkmcnt(0)\n\ts_barrier" ::: "memory");
        if (wfirst) {
            if (ok) ok = poll_part(fp, done, 0xffff0000ull, wt);
            first_w(lds0, sd, ws0, lane);
        }
        if (ok) ok = poll_part(fp, done, 0xffffull, wt);
#pragma unroll
        for (int s = 0; s < 3; ++s) {
            const u32 sx = O_IN + XSLOT(s) * 4096, vx = ((s >> 1) == skip || (s >> 1) == skip + 6) ? 0x80000000u : lane * 16;
            ld_x<0>(X[s][0], sd, vx, sx); ld_x<1>(X[s][1], sd, vx, sx); ld_x<2>(X[s][2], sd, vx, sx); ld_x<3>(X[s][3], sd, vx, sx);
        }
        if (!wfirst) {
            if (ok) ok = poll_part(fp, done, 0xffff0000ull, wt);
            first_w(lds0, sd, ws0, lane);
        } else {
            asm volatile("s_waitcnt vmcnt(8)" ::: "memory");
        }
        if (!ok && lane == 0) *fail = 1;
        wait_step<12>(X[0]);
        SB();
#pragma unroll
        for (int i = 0; i < 8; ++i) a[i] = ring[i * 64 + lane];
        SB();
#define NEXT_FO ((g == 0 || g == 3) && !none ? f0 + (g + 2) * (2 * SL * FS) : OOB)
#define ADVANCE() do { ++g; xs0 += 2 * SL; ws0 += 2 * SL; skip -= 2; } while (0)
        bool go = gemm_steps<0, 0>(acc, X, a, ring, lds0, sd, xs0, ws0, skip, lane, NEXT_FO, fv, wt.seq, votes + g);
        ADVANCE();
        while (g + 1 < gend && go) {
            go = gemm_steps<0, 1>(acc, X, a, ring, lds0, sd, xs0, ws0, skip, lane, NEXT_FO, fv, wt.seq, votes + g);
            ADVANCE();
        }
        gemm_steps<0, 2>(acc, X, a, ring, lds0, sd, xs0, ws0, skip, lane, OOB, fv, wt.seq, votes);
        ADVANCE();
    } while (g < gend);
    return ok;
}

__global__ void __launch_bounds__(256, 2) fused(const Params p) {
    __shared__ u32x4 ring[4096];
    __shared__ __attribute__((aligned(16))) u32 red[48];
    u32 ka0, ka1;
    asm volatile("s_load_dword %0, %2, 0x0\n\ts_load_dword %1, %2, 0x40" : "=&s"(ka0), "=&s"(ka1) : "s"(__builtin_amdgcn_kernarg_segment_ptr()));
    const int lane = threadIdx.x & 63;
    const int wv = (int)rfl(threadIdx.x >> 6);
    const int bid = blockIdx.x;
    int wg = bid < HW ? bid : bid < NG1 ? bid + NCOMM : bid < 256 ? bid - NG1 + HW : bid < B_IN ? bid : bid < B_IN + NIN ? GRID : bid - NIN;
    f32x4 F[8];
    if (wg < GRID) {
        for (int z = wg >= STAGP * 8 ? (wg >> 3) * STAG : 0; z > 0; --z) __builtin_amdgcn_s_sleep(1);
        ld_src(p, lane, wg * 4 + wv, F);
    }
    asm volatile("s_waitcnt lgkmcnt(0)" :: "s"(ka0), "s"(ka1));
    const int rank = p.rank & 7, rm = rank & 1, ri = (rank >> 1) & 1, rj = rank >> 2;
    char* const my = p.my;
    const rsrc_t rsm = mk_rsrc(my, WS_USED);

    Wait wt;
    wt.t0 = __builtin_amdgcn_s_memrealtime();
    wt.tmo = (gu32*)(my + O_TMO);
    gu32* const seqp = (gu32*)(my + O_SEQ + (bid * 4 + wv) * 4);
    const u32 seqv = __hip_atomic_load(seqp, __ATOMIC_RELAXED, __HIP_MEMORY_SCOPE_AGENT);
    if (threadIdx.x < 32) red[threadIdx.x] = 0;
    wt.seq = 0;
    u32* hx = red + 32;
    while (wg < GRID) {
        const Cv cv = cv_of(wg * 4 + wv);
        const int pos = cv.pos, dm = cv.dm, cls = cv.cls, blk = cv.blk, ch = cv.ch, pass = cv.pass, ks = cv.ks;
        u32 D[16];
        const bool isx = ch < 16, own = isx && (ch & 1) * 4 + (blk >> 1) == rank, same = cls == (isx ? ri : rj);
        const int dc = same ? 0 : (isx ? 2 : 4), d0 = (dc ^ (isx ? 4 : 2)) | (dm ^ rm), d1 = dc | (dm ^ rm);
        const rsrc_t rs0 = mk_rsrc(p.rel[d0], WS_USED), rs1 = mk_rsrc(p.rel[d1], same ? 0u : WS_USED);
        const u32 cb8 = sto(ch) * 8 + blk;
        const u32 r0 = pass * 6 + d0 - 2, r1 = same ? 0 : pass * 6 + d1 - 2;
        const u32 ls = ks * SL + (same ? cb8 : isx ? 256 + (ch >> 1) * 2 + (blk & 1) : 144 + cb8);
        const int r32 = (pos & 3) * 8 + blk;
        const int item = (pos >> 2) * 30 + r32 - ((r32 >> 1) > rank ? 2 : 0);
        const bool local = same || own || !isx;
        const float m = cvt_src(F, isx, D);
        u32 hpay = 0;
        if (!own) {
            Packed k;
            pack(D, m, k, isx ? 1 : 2);
            const u32 pk = (ks * 2 + cls) * 256 + cb8;
            if (__builtin_expect(k.lo & 0x80, 0)) st128((u32x4){k.X[0], k.X[1], k.X[2], k.X[3]}, rsm, O_PKOVF + pk * 1024 + lane * 16);
            const u32x4 l0 = {k.L[0], k.L[1], k.L[2], k.L[3]}, l1 = {k.L[4], k.L[5], k.L[6], k.L[7]}, nn = {k.N[0], k.N[1], k.N[2], k.N[3]};
            if (item < HEAD) {
                const u32 o0 = O_PKL + ((d0 - 2) * 64 + item) * BLK + lane * 16, o1 = O_PKL + ((same ? 0 : d1 - 2) * 64 + item) * BLK + lane * 16;
                st128(l0, rs0, o0); st128(l1, rs0, o0 + 1024); st128(nn, rs0, o0 + 2048);
                st128(l0, rs1, o1); st128(l1, rs1, o1 + 1024); st128(nn, rs1, o1 + 2048);
            } else {
                const u32 o = O_PK + pk * BLK + lane * 16;
                st128(l0, rsm, o); st128(l1, rsm, o + 1024); st128(nn, rsm, o + 2048);
            }
            hpay = lane ? k.hi : k.lo;
        }
        if (local) st_plain(D, rsm, O_IN + ls * 4096 + lane * 16);
        wt.seq = rfl(seqv) + 1;
        WAIT_VM0();
        st_word_b(rsm, local && lane == 0, O_INFLAG + ls * FS, wt.seq);
        if (lane < 2) hx[wv * 2 + lane] = hpay;
        __syncthreads();
        if (wv == 3) {
            const u32 v = hx[lane & 7];
            st_gran_b(rs0, lane < 8, O_PKHDR + (r0 * 64 + (cb8 >> 2)) * 128 + lane * 8, v, wt.seq);
            st_gran_b(rs1, lane < 8, O_PKHDR + (r1 * 64 + (cb8 >> 2)) * 128 + lane * 8, v, wt.seq);
        }
        hx += 8;
        wg = (wg >= 256 + NG2 && wg < 256 + NG2 + NIN) ? wg + GRID - NIN - 256 - NG2 : GRID;
        if (wg < GRID) ld_src(p, lane, wg * 4 + wv, F);
    }
    wt.seq = rfl(seqv) + 1;
    if (lane == 0) __hip_atomic_store(seqp, wt.seq, __ATOMIC_RELAXED, __HIP_MEMORY_SCOPE_AGENT);

    bool ok = true;
    float y[16 * KEEP];
    int nk = 0, kept = 0;

    const bool puller = bid >= NG1 && bid < 256;
    const int iw = ((bid & 255) - NG1) * 4 + wv, l = iw % 6, ip = iw / 6;
    if (bid >= B_IN ? ip < NPI : puller && ip < HEAD && ip < NCOMM * 4 / 6) {
        const int src = rank ^ (l + 2);
        char* const peer = p.rel[l + 2];
        const rsrc_t rs = mk_rsrc(puller ? my : peer, WS_USED);
        const char* const hb = my + O_PKHDR + l * 64 * 128 + (lane & 1) * 8;
        constexpr int D0 = HEAD + 2 * NPI, ND = 480 - D0;
        const u32 tbase = (wt.seq - 1) * (ND + NPI) - D0;
#define HOFF(b) (((b) & 252) * 32 + ((b) & 3) * 16 + (((b) >> 8) & 1) * (6 * 64 * 128))
#define PKIDX(b) (((2 * rm + (((b) >> 8) & 1)) * 2 + (((b) & 128) ? rj : ri)) * 256 + ((b) & 255))
        u32 i0 = puller ? ip : HEAD + ip;
        int b0 = item_at(i0, src);
        u32 tk = tbase + HEAD + NPI + ip, fo = OOB;
        u64 h0 = ld_gran(hb + HOFF(b0));
        WAIT_VM0();
        do {
            const u32 so = puller ? O_PKL + (l * 64 + i0) * BLK : O_PK + PKIDX(b0) * BLK;
            if (ok) ok = poll_gran(hb + HOFF(b0), true, h0, wt);
            Pk k;
            k.l0 = __builtin_amdgcn_raw_buffer_load_b128(rs, lane * 16, so, SYS);
            k.l1 = __builtin_amdgcn_raw_buffer_load_b128(rs, lane * 16 + 1024, so, SYS);
            k.n = __builtin_amdgcn_raw_buffer_load_b128(rs, lane * 16 + 2048, so, SYS);
            SB();
            const u32 lo = (u32)__builtin_amdgcn_readlane((int)(u32)h0, 0), hi = (u32)__builtin_amdgcn_readlane((int)(u32)h0, 1);
            i0 = puller ? i0 + NCOMM * 4 / 6 : rfl(tk) - tbase;
            const int b1 = i0 < (puller ? HEAD : 480u) ? item_at(i0, src) : -1;
            h0 = ld_gran(hb + HOFF(b1));
            tk = __builtin_amdgcn_raw_ptr_buffer_atomic_add_i32(1, rsm, (b1 >= 0 && !puller && lane == 0) ? O_TICK + l * 128 : OOB, 0, 16);
            asm volatile("s_waitcnt vmcnt(5)" ::: "memory");
            st_word_b(rsm, ok, fo, wt.seq);
            const u32 slot = (4 + (b0 >> 8) * 6 + l) * SL + (b0 & 255), o = O_IN + slot * 4096 + lane * 16;
            fo = lane == 0 ? O_INFLAG + slot * FS : OOB;
            const u32x16 d = unpack(k, lo, hi, peer + O_PKOVF + PKIDX(b0) * 1024 + lane * 16, lane);
            asm volatile("" : "+v"(h0), "+v"(tk));
            st128(FRAG(d, 0), rsm, o); st128(FRAG(d, 1), rsm, o + 1024); st128(FRAG(d, 2), rsm, o + 2048); st128(FRAG(d, 3), rsm, o + 3072);
            SB();
            b0 = b1;
        } while (b0 >= 0);
        WAIT_VM0();
        st_word_b(rsm, ok, fo, wt.seq);
    }
    if (bid >= B_IN) return;
    const int hi = bid < 256 ? bid - NG2 : bid - 256 + NG1 - NG2;
    if (!puller) {
        const bool mainT = bid < 256 + NG2;
        if (!mainT && bid < B_IN - 96) return;
        const int xcd = bid & 7, k8 = (bid & 255) >> 3;
        __syncthreads();
        const u32 lds0 = (u32)(size_t)(__attribute__((address_space(3))) u32x4*)ring + wv * 4096;
        const u32x4 sd = {(u32)(size_t)my, (u32)((size_t)my >> 32) & 0xffffu, WS_USED, 0x00020000u};
        u32 xs0, ws0, ob, ho;
        int od;
        bool wfirst = false;
        if (mainT) {
            int a, b;
            shell_at(bid < 256 ? k8 : 31 - k8, a, b);
            const int cx = 4 * a + (xcd >> 1), n = 2 * b + (xcd & 1);
            xs0 = ((xcd >> 1) * 4 + a) * 8 + wv * 2;
            ws0 = (16 + (xcd & 1) * 8 + b) * 8 + wv;
            ob = (cx * 4 + wv) * 32 + 2 * n;
            ho = ((ri * 8 + (cx >> 1)) * 32 + rj * 16 + n) * 128 + rm * 32;
            od = ((cx & 1) * 4 + wv) ^ rank;
            wfirst = (cx >> 1) > (n >> 1);
        } else {
            const int e = k8 - (NG1 - 96) / 8, dq = (e >> 2) + 1, c = ((e >> 1) & 1) * 4 + wv, n = ((xcd >> 1) * 2 + (e & 1)) * 2 + (xcd & 1);
            xs0 = (dq & 1) ? 256 + c * 2 : sto(2 * c + rj) * 8 + (rank & 3) * 2;
            ws0 = ((dq & 2) ? 144 : 0) + sto(16 + n) * 8 + wv;
            const int rg = (ri ^ (dq & 1)) * 8 + c, n32 = (rj ^ (dq >> 1)) * 16 + n;
            ob = 2048 + rg * 64 + 2 * n32;
            ho = (rg * 32 + n32) * 128 + 64;
            od = 0;
        }
        f32x4 acc[2][8];
#pragma nounroll
        for (int pass = 0; pass < 2; ++pass) {
        const float z0 = pass ? 0.f : lane == 5 ? 0.f : 1.f, z1 = pass ? 0.f : 1.f;
#pragma unroll
        for (int a = 0; a < 2; ++a)
#pragma unroll
            for (int f = 0; f < 8; ++f) acc[a][f] = (f32x4){a + f ? z1 : z0, z1, z1, z1};
        if (pass) ok = gemm_tile(acc, ring, lds0, sd, xs0, ws0, mainT ? 8 : 2, od >= 2 ? od + 2 : -64, wfirst, my, wt, &red[4], &red[16], lane);
        asm volatile("s_nop 15" ::: "memory");
        SB();
        const rsrc_t rsm = mk_rsrc(my, pass ? WS_USED : 0u);
        u32 hpay = 0;
#pragma nounroll
        for (int h = 0; h < 2; ++h) {
            u32 D[16];
            float m = 0.f;
#pragma unroll
            for (int rf = 0; rf < 2; ++rf)
#pragma unroll
                for (int r = 0; r < 4; ++r) {
                    D[rf * 8 + r * 2] = cvt2(acc[rf][0][r], acc[rf][1][r]);
                    D[rf * 8 + r * 2 + 1] = cvt2(acc[rf][2][r], acc[rf][3][r]);
                    m = fmaxf(fmaxf(m, fabsf(acc[rf][0][r])), fabsf(acc[rf][1][r]));
                    m = fmaxf(fmaxf(m, fabsf(acc[rf][2][r])), fabsf(acc[rf][3][r]));
                }
            Packed k;
            pack(D, m, k, 4);
            if (__builtin_expect(k.lo & 0x80, 0)) st128((u32x4){k.X[0], k.X[1], k.X[2], k.X[3]}, rsm, O_OUTOVF + (ob + h) * 1024 + lane * 16);
#pragma unroll
            for (int t = 0; t < 4; ++t)
                st96((u32x3){k.L[2 * t], k.L[2 * t + 1], k.N[t]}, rsm, O_OUTDATA + (ob + h) * BLK + lane * 12 + t * 768);
            hpay = (lane >> 1) == h ? ((lane & 1) ? k.hi : k.lo) : hpay;
#pragma unroll
            for (int rf = 0; rf < 2; ++rf)
#pragma unroll
                for (int f = 0; f < 4; ++f) acc[rf][f] = acc[rf][4 + f];
        }
        const rsrc_t rso = mk_rsrc(p.rel[od & 7], pass ? WS_USED : 0u);
        ok = ok && red[4] == 0;
        WAIT_VM0();
        st_gran_b(rso, ok && lane < 4, O_OUTHDR + ho + lane * 8, hpay, wt.seq);
        }
        if (hi < 0 || hi >= HELD) return;
    }

    u32 owed = OOB;
    int part = wv;
    int rgA = 0, cgA = 0;
    bool gate = false;
#pragma unroll
    for (int i = 0; i < 16 * KEEP; ++i) asm volatile("" : "=v"(y[i]));
#pragma nounroll
    for (int pass = 0; pass < 2; ++pass) {
    int bid = blockIdx.x, rank = p.rank & 7, wv = (int)rfl(threadIdx.x >> 6), lane = threadIdx.x & 63;
    asm volatile("" : "+s"(bid), "+s"(rank), "+s"(wv), "+v"(lane));
    const int rm = rank & 1, ri = (rank >> 1) & 1, rj = rank >> 2;
    if (puller) {
        const int pid = bid - NG1, dq = pid & 3, hf = wv & 1, pw = (pid >> 2) * 2 + (wv >> 1);
        rgA = (ri ^ (dq & 1)) * 8; cgA = (rj ^ (dq >> 1)) * 32;
        char* amx = p.rel[lane & 7] + O_AMAX + (rank * NCOMM + pid) * AS;
        asm volatile("" : "+v"(amx));
        const u32 tko = O_TICK + (6 + dq * 2 + hf) * 128, tbase = (wt.seq - 1) * (256 - NP) - 2 * NP;
        const bool hact = lane < (dq ? 6 : 4);
        const char* const hbase = my + O_OUTHDR + (rgA * 32 + cgA / 2) * 128 + ((lane >> 1) & 3) * 32 + (lane & 1) * 8;
        const char* sp[3];
        sp[0] = p.rel[dq * 2 + rm]; sp[1] = p.rel[dq * 2 + (rm ^ 1)]; sp[2] = my;
#define OBLK(s, c, cg) ((s) < 2 ? (u32)((((c) * 2 + rj) * 4 + (rank & 3)) * 32 + (cg)) : (u32)(2048 + (rgA + (c)) * 64 + cgA + (cg)))
#define OHDR(c, cg) (((c) * 32 + ((cg) >> 1)) * 128 + ((cg) & 1) * 16)
        if (pass == 0) __syncthreads();
        float mx = 0.f;
        auto decode = [&](u32 t) -> int { int c, n; shell_at(t >> 1, c, n); return t * 256 + c * 32 + 2 * n + (t & 1); };
#define IC(i) (((i) >> 5) & 7)
#define IG(i) ((i) & 31)
        int ib = -1, ic = decode(pw), nkeep = 0;
        u32x3 nxt[6];
        f32x4 v[4];
        u32 b4[3], so[2], hcur = pass ? 0u : 0x10040100u, tk = tbase + NP + pw, fo = OOB;
        u64 fix = !pass, hn = 0;
#pragma unroll
        for (int s = 0; s < 6; ++s) { nxt[s] = (u32x3){0, 0, 0}; b4[s >> 1] = 0; }
        auto hand_over = [&]() -> u32 {
            const bool mem = ib >= 0 && ib < HELD * 256;
            const u32 o = mem ? O_Y + ((rgA + IC(ib)) * 64 + cgA + IG(ib)) * 8192 + hf * 4096 + lane * 16 : OOB;
            const u32x4 r0 = relu4(v[0]), r1 = relu4(v[1]), r2 = relu4(v[2]), r3 = relu4(v[3]);
            st128(r0, rsm, o); st128(r1, rsm, o + 1024); st128(r2, rsm, o + 2048); st128(r3, rsm, o + 3072);
            if (ib >= HELD * 256) {
                u32x4* const d = ring + wv * 1024 + nk * 256 + lane;
                d[0] = r0; d[64] = r1; d[128] = r2; d[192] = r3;
                kept = lane == nk ? ib : kept;
                ++nk;
            }
            return (mem && lane == 0) ? O_PROG + (dq * 256 + (ib >> 8)) * 128 + hf * 4 : OOB;
        };
        auto process = [&](const u32x3 (&cur)[6]) {
            u32 ha[6], hb[6];
#pragma unroll
            for (int s = 0; s < 6; ++s) dec_h(cur[s].z, b4[s >> 1], ha[s], hb[s]);
            if (__builtin_expect(fix != 0, 0)) {
#pragma unroll
                for (int s = 0; s < 3; ++s) {
                    const u32 lo = (u32)__builtin_amdgcn_readlane((int)hcur, 2 * s);
                    const u32 hi = (u32)__builtin_amdgcn_readlane((int)hcur, 2 * s + 1);
                    if (lo & 0x80) {
                        const char* pv = sp[s] + O_OUTOVF + OBLK(s, IC(ib), IG(ib)) * 1024 + lane * 16 + hf * 8;
                        u32x2 x;
                        asm volatile("global_load_dwordx2 %0, %1, off sc0 sc1\n\ts_waitcnt vmcnt(0)" : "=&v"(x) : "v"(pv) : "memory");
                        ha[2 * s] = cur[2 * s].z; hb[2 * s] = x.x; ha[2 * s + 1] = cur[2 * s + 1].z; hb[2 * s + 1] = x.y;
                    } else {
                        u64 pay = (((u64)hi << 32) | lo) >> 8;
                        while (pay != 0) {
                            const u32 e = (u32)pay & 0x3ffff, idx = (e >> 6) & 31, w = (idx >> 2) & 3;
                            pay >>= 18;
                            const u32 t = ((int)(idx >> 4) == hf && lane == (int)(e & 63)) ? (e >> 11) << (8 * (idx & 3)) : 0;
                            ha[2 * s] ^= w == 0 ? t : 0; hb[2 * s] ^= w == 1 ? t : 0;
                            ha[2 * s + 1] ^= w == 2 ? t : 0; hb[2 * s + 1] ^= w == 3 ? t : 0;
                        }
                    }
                }
            }
#pragma unroll
            for (int s = 0; s < 6; ++s) {
                f32x4 a, b;
#pragma unroll
                for (int j = 0; j < 4; ++j) {
                    const u32 sel = ((4u + j) << 24) | ((u32)j << 16) | 0x0c0cu;
                    a[j] = __builtin_bit_cast(float, __builtin_amdgcn_perm(ha[s], cur[s].x, sel));
                    b[j] = __builtin_bit_cast(float, __builtin_amdgcn_perm(hb[s], cur[s].y, sel));
                }
                v[(s & 1) * 2] = s >> 1 ? v[(s & 1) * 2] + a : a;
                v[(s & 1) * 2 + 1] = s >> 1 ? v[(s & 1) * 2 + 1] + b : b;
            }
#pragma unroll
            for (int j = 0; j < 4; ++j) mx = fmaxf(fmaxf(fmaxf(fmaxf(mx, v[j].x), v[j].y), v[j].z), v[j].w);
            asm volatile("" : "+v"(mx));
        };
        auto offsets = [&]() {
            so[0] = O_OUTDATA + OBLK(0, IC(ic), IG(ic)) * BLK + hf * 1536;
            so[1] = O_OUTDATA + OBLK(2, IC(ic), IG(ic)) * BLK + hf * 1536;
            asm volatile("" : "+s"(so[0]), "+s"(so[1]));
        };
        offsets();
        if (pass) {
            const int l = dq ? 2 * dq - 2 : 2 * ((pid >> 2) % 3), b0 = item_at(480 - TRIG + pw, rank ^ (l + 2)), b1 = item_at(480 - TRIG + pw, rank ^ (l + 3));
            const int b = lane ? b1 : b0;
            if (ok) ok = poll_word(my + O_INFLAG + ((4 + (b >> 8) * 6 + l + (lane & 1)) * SL + (b & 255)) * FS, lane < 2, wt.seq, wt);
        }
        if (pass) do {
            asm volatile("" : "+v"(nxt[0]), "+v"(nxt[1]), "+v"(nxt[2]), "+v"(nxt[3]), "+v"(nxt[4]), "+v"(nxt[5]), "+v"(hn));
            u32x3 cur[6];
#pragma unroll
            for (int s = 0; s < 6; ++s) cur[s] = nxt[s];
            int id = -1;
            u32 hc;
            {
                if (ok) ok = poll_gran(hbase + OHDR(IC(ic), IG(ic)), hact, hn, wt);
#pragma unroll
                for (int s = 0; s < 6; ++s)
                    nxt[s] = __builtin_amdgcn_raw_buffer_load_b96(mk_rsrc(sp[s >> 1], (s >= 4 && !dq) ? 0u : WS_USED), lane * 12 + (s & 1) * 768,
                                                                  so[s >> 2], SYS);
                SB();
                hc = hact ? (u32)hn : 0u;
                asm volatile("" : "+v"(hc));
                const u32 t = rfl(tk) - tbase;
                if (nkeep < KEEP && t < 256u) {
                    id = decode(t);
                    nkeep += t >= HELD;
                }
                hn = ld_gran(hbase + OHDR(IC(id), IG(id)));
                tk = __builtin_amdgcn_raw_ptr_buffer_atomic_add_i32(1, rsm, (id >= 0 && nkeep < KEEP && lane == 0) ? tko : OOB, 0, 16);
            }
            process(cur);
            {
                const u32 fn = hand_over();
                asm volatile("s_waitcnt vmcnt(12)" ::: "memory");
                st_word_b(rsm, ok, fo, wt.seq);
                fo = fn;
                ib = ic; ic = id;
                hcur = hc;
                const u32 bv = (hc & 0x7f) * 0x01010101u;
#pragma unroll
                for (int s = 0; s < 3; ++s) b4[s] = (u32)__builtin_amdgcn_readlane((int)bv, 2 * s);
                const u32 oth = (u32)__builtin_amdgcn_update_dpp(0, (int)hc, 0xB1, 0xf, 0xf, false);
                const u64 pay = ((u64)oth << 32) | hc;
                bool need = (hc & 0x80) != 0;
#pragma unroll
                for (int z = 0; z < 3; ++z) {
                    const u32 e = (u32)(pay >> (8 + 18 * z));
                    need |= ((e & 0x3f800) != 0) & ((int)((e >> 10) & 1) == hf);
                }
                fix = __ballot(need & !(lane & 1));
                offsets();
            }
        } while (ic >= 0);
        WAIT_VM0();
        process(nxt);
        const u32 wm = ok ? wave_umax(__builtin_bit_cast(u32, mx)) : 0xffffffffu;
        u32 arrived, m;
        {
            const u32 la = (u32)(size_t)(__attribute__((address_space(3))) u32*)&red[2 * pass];
            asm volatile("s_mov_b64 exec, 1\n\tds_max_u32 %2, %3\n\tds_add_rtn_u32 %0, %2, %4 offset:4\n\tds_read_b32 %1, %2\n\t"
                         "s_mov_b64 exec, -1\n\ts_waitcnt lgkmcnt(0)" : "=&v"(arrived), "=&v"(m) : "v"(la), "v"(wm), "v"(1u) : "memory");
        }
        m = rfl(m);
        part = rfl(arrived);
        if (part == 3 && lane < 8 * pass && m != 0xffffffffu) st_gran(amx, m, wt.seq);
        if (part == 3) __builtin_amdgcn_s_barrier();
        if (nkeep == KEEP && lane == 0) owed = tko;
        st_word_b(rsm, ok, fo, wt.seq);
        fo = hand_over();
        if (ib < HELD * 256) {
            WAIT_VM0();
            st_word_b(rsm, ok, fo, wt.seq);
        }
#pragma unroll
        for (int j = 0; j < 4 * KEEP; ++j) {
            const f32x4 a = __builtin_bit_cast(f32x4, ring[wv * 1024 + j * 64 + lane]);
            y[4 * j] = a.x; y[4 * j + 1] = a.y; y[4 * j + 2] = a.z; y[4 * j + 3] = a.w;
        }
        if (part != 3) __builtin_amdgcn_s_barrier();
    } else if (pass) {
        rgA = (ri ^ (wv & 1)) * 8; cgA = (rj ^ (wv >> 1)) * 32;
        gate = hi < HELD - 2 * NP;
        { int c, n; shell_at(hi >> 1, c, n); rgA += c; cgA += 2 * n + (hi & 1); }
        if (ok) ok = poll_word(my + O_PROG + (wv * 256 + hi) * 128 + (lane & 1) * 4, lane < 2, wt.seq, wt);
        const u32 so = O_Y + (rgA * 64 + cgA) * 8192;
#pragma unroll
        for (int u = 0; u < 8; ++u) {
            const f32x4 a = __builtin_bit_cast(f32x4, __builtin_amdgcn_raw_buffer_load_b128(rsm, lane * 16, so + u * 1024, SYS));
            y[4 * u] = a.x; y[4 * u + 1] = a.y; y[4 * u + 2] = a.z; y[4 * u + 3] = a.w;
        }
        WAIT_VM0();
    }

    {
        u32 oo[KEEP];
        const int np = !pass ? KEEP : puller ? nk : 2;
#pragma unroll
        for (int r = 0; r < KEEP; ++r) {
            const int it = __builtin_amdgcn_readlane(kept, r);
            oo[r] = ((rgA + IC(it)) * 32 + (puller ? wv & 1 : r) * 16 + (lane & 15)) * 4096 + (cgA + IG(it)) * 64 + (lane >> 4) * 16;
            oo[r] = r < np ? oo[r] : OOB;
            asm volatile("" : "+v"(oo[r]));
        }
        const rsrc_t ro = mk_rsrc(p.out, pass ? 512u * 4096u : 0u);
        constexpr int GL = (NR * NCOMM + 191) / 192;
        const char* a0 = my + O_AMAX + (part * 64 * GL + lane) * AS;
        u64 gv[GL];
        bool gd[GL];
#pragma unroll
        for (int z = 0; z < GL; ++z) { gv[z] = 0; gd[z] = (part * GL + z) * 64 + lane >= NR * NCOMM; }
        if (pass && gate && ok && part < 3) ok = poll_word(my + O_PROG + ((part + 1) * 256 + HELD - 1) * 128, lane == 0, wt.seq, wt);
        while (pass && ok && part < 3) {
            bool all = true;
#pragma unroll
            for (int z = 0; z < GL; ++z) if (!gd[z]) gv[z] = ld_gran(a0 + z * 64 * AS);
#pragma unroll
            for (int z = 0; z < GL; ++z) { gd[z] = gd[z] || (u32)(gv[z] >> 32) >= wt.seq; all = all && gd[z]; }
            if (__all(all)) break;
            if (expired(wt)) ok = false;
            __builtin_amdgcn_s_sleep(1);
        }
        u32 gm = 0;
#pragma unroll
        for (int z = 0; z < GL; ++z) gm = umax(gm, (u32)gv[z]);
        const u32 wm = wave_umax(gm);
        if (lane == 0) red[8 + pass * 4 + wv] = ok ? wm : 0xffffffffu;
        __syncthreads();
        const u32x4 r4 = *(const u32x4*)&red[8 + pass * 4];
        const float am = __builtin_bit_cast(float, umax(umax(r4.x, r4.y), umax(r4.z, r4.w)));
        const float inv = am == 0.f ? 0.f : 448.0f * __builtin_amdgcn_rcpf(am);
#pragma unroll
        for (int r = 0; r < KEEP; ++r) {
            if (r == 2 && np <= 2) break;
            u32x4 o;
#pragma unroll
            for (int j = 0; j < 4; ++j)
                o[j] = q4(y[16 * r + 4 * j], y[16 * r + 4 * j + 1], y[16 * r + 4 * j + 2], y[16 * r + 4 * j + 3], inv);
            __builtin_amdgcn_raw_buffer_store_b128(o, ro, oo[r], 0, SYS);
        }
        if (pass) __builtin_amdgcn_raw_ptr_buffer_atomic_add_i32(1, rsm, owed, 0, 16);
    }
    }
}

extern "C" void kernel_launch(int rank, int nranks, ncclComm_t comm, hipStream_t stream,
                              void* const* d_in, const int* in_sizes, int n_in,
                              void* d_out, int out_size,
                              const int* in_dtypes, int out_dtype,
                              void* const* d_out_all,
                              void* const* d_ws_all, size_t ws_bytes) {
    (void)comm; (void)in_dtypes; (void)out_dtype; (void)d_out_all;
    if (nranks != NR || n_in != 2 || (unsigned)rank >= (unsigned)NR || ws_bytes < WS_USED ||
        in_sizes[0] != 4096 * 512 || in_sizes[1] != 512 * 4096 || out_size != 512 * 4096) {
        fprintf(stderr, "kernel_launch: unexpected problem shape\n");
        return;
    }
    Params p;
    p.x = (const float*)d_in[0];
    p.w = (const float*)d_in[1];
    p.out = (unsigned char*)d_out;
    p.my = (char*)d_ws_all[rank];
    for (int r = 0; r < NR; ++r) p.rel[r] = (char*)d_ws_all[r ^ rank];
    p.rank = rank;
    p.pad = 0;
    hipLaunchKernelGGL(fused, dim3(GRID), dim3(256), 0, stream, p);
    HIP_CHECK(hipGetLastError());
}
```
